```python
import numpy as np

kernel_name = "gpu_gemm_ar_m2048_k2048_n2048_bf16_none"


ABSMAX_RTOL = 0.008

COMPUTE_DTYPE = "bfloat16"
EPILOGUE = "none"
IN_QUANT = None
OUT_QUANT = None
IN_DTYPES = {"x": "bfloat16", "w_mat": "bfloat16"}
WORLD_SIZE = 8

def _to_bf16(x):
    u = np.ascontiguousarray(x, dtype=np.float32).view(np.uint32)
    return ((u + 0x7FFF + ((u >> 16) & 1)) & 0xFFFF0000).view(np.float32)

def setup_inputs():

    x = _to_bf16(np.random.randn(2048, 2048).astype(np.float32))
    w_mat = _to_bf16((np.random.randn(2048, 2048) / np.sqrt(2048)).astype(np.float32))
    return {"x": x, "w_mat": w_mat}

def _epilogue(y):
    return y

def reference(x, w_mat):
    y = (x @ w_mat).astype(np.float32)
    return _epilogue(y)
```

```cpp
#include <hip/hip_runtime.h>
#include <rccl/rccl.h>
#include <cstdio>
#include <cstdint>

namespace {

typedef unsigned u32x4 __attribute__((ext_vector_type(4)));
typedef float f32x4 __attribute__((ext_vector_type(4)));
typedef float f32x16 __attribute__((ext_vector_type(16)));
typedef __bf16 bf16x8 __attribute__((ext_vector_type(8)));
typedef __bf16 bf16x2 __attribute__((ext_vector_type(2)));
typedef float f32x2 __attribute__((ext_vector_type(2)));

constexpr int NR = 8;
constexpr unsigned UNIT = 16384;
constexpr unsigned SLICE = 128 * UNIT;
constexpr unsigned OUT_BYTES = 2048u * 2048u * 2u;
constexpr unsigned IN_BYTES = 2048u * 256u * 2u;
constexpr size_t OFF_CNT = 0;
constexpr size_t OFF_TMO = 16384;
constexpr size_t OFF_FS = 32768;
constexpr size_t OFF_FL = OFF_FS + 131072;
constexpr size_t OFF_FO = OFF_FL + 131072;
constexpr size_t OFF_DBG = 524288;
constexpr size_t OFF_DATA = 1u << 20;
constexpr size_t WS_NEED = OFF_DATA + (size_t)NR * SLICE;
constexpr int NGEMM = 128;
#define NW 16
constexpr int NMOVE = 7 * NW;
constexpr unsigned long long DEADLINE = 100000000ull;
constexpr int AUX_SYS = 17, AUX_DEV = 16;

#define STAMP(slot) do {} while (0)

struct Params {
  const char* x; const char* w;
  char* out[NR]; char* ws[NR];
  int rank;
};

__device__ __forceinline__ unsigned ld_sys(const unsigned* p) { return __hip_atomic_load(p, __ATOMIC_RELAXED, __HIP_MEMORY_SCOPE_SYSTEM); }
__device__ __forceinline__ unsigned ld_dev(const unsigned* p) { return __hip_atomic_load(p, __ATOMIC_RELAXED, __HIP_MEMORY_SCOPE_AGENT); }
__device__ __forceinline__ void st_sys(unsigned* p, unsigned v) { __hip_atomic_store(p, v, __ATOMIC_RELAXED, __HIP_MEMORY_SCOPE_SYSTEM); }
__device__ __forceinline__ void st_dev(unsigned* p, unsigned v) { __hip_atomic_store(p, v, __ATOMIC_RELAXED, __HIP_MEMORY_SCOPE_AGENT); }
#define WAIT_VM(n) asm volatile("s_waitcnt vmcnt(" #n ")" ::: "memory")

template <bool SYS>
__device__ __forceinline__ bool wait_flags(const unsigned* p, unsigned call, unsigned* tmo) {
  const unsigned long long t0 = __builtin_amdgcn_s_memrealtime();
  for (;;) {
    const unsigned v = SYS ? ld_sys(p) : ld_dev(p);
    if (__all(v >= call)) break;
    const unsigned long long dt = __builtin_amdgcn_s_memrealtime() - t0;
    if (dt > 10000ull) {
      if (__builtin_amdgcn_readfirstlane(ld_dev(tmo)) == call || dt > DEADLINE) { st_dev(tmo, call); return false; }
      __builtin_amdgcn_s_sleep(32);
    }
    __builtin_amdgcn_s_sleep(1);
  }
  __builtin_amdgcn_fence(__ATOMIC_ACQUIRE, "wavefront");
  return true;
}

__device__ __forceinline__ __amdgpu_buffer_rsrc_t rsrc(const void* base, unsigned bytes) {
  return __builtin_amdgcn_make_buffer_rsrc((void*)base, 0, bytes, 0x00020000);
}

struct Item { const char* sb; char* db; unsigned cap, off, tstr; const unsigned* flag; unsigned* done; };

__global__ void __launch_bounds__(512, 2) fused_gemm_ar(const Params P) {
  __shared__ f32x4 lds[8192];
  const int lane = threadIdx.x & 63;
  const int wave = __builtin_amdgcn_readfirstlane(threadIdx.x >> 6);
  const int bid = blockIdx.x;
  const unsigned rank = (unsigned)P.rank & 7u;
  char* const myws = P.ws[rank];
  unsigned* const tmo = (unsigned*)(myws + OFF_TMO);
  unsigned* const cntp = (unsigned*)(myws + OFF_CNT) + bid * 8 + wave;

  if (bid < NGEMM) {
    STAMP(0);
    const unsigned cnt_raw = ld_dev(cntp);
    char* const peer_ws = P.ws[lane & 7];
    const auto rdat = rsrc(myws + OFF_DATA, NR * SLICE);
    const unsigned u = bid;
    const unsigned ubase = rank * SLICE + u * UNIT;
    bool need;
    if (u < 64) {
      const unsigned b = u >> 1, h = u & 1;
      const auto rx = rsrc(P.x, IN_BYTES);
      const unsigned row = 8 * wave + (lane & 7);
      u32x4 v[2];
#pragma unroll
      for (int q = 0; q < 2; ++q) v[q] = __builtin_amdgcn_raw_buffer_load_b128(rx, (64 * b + row) * 512 + h * 256 + (8 * q + (lane >> 3)) * 16, 0, 0);
#pragma unroll
      for (int q = 0; q < 2; ++q) {
        const unsigned c = 8 * q + (lane >> 3);
        __builtin_amdgcn_raw_buffer_store_b128(v[q], rdat, ubase + ((c >> 1) * 2 + (row >> 5)) * 1024 + ((row & 31) + 32 * (c & 1)) * 16, 0, AUX_SYS);
      }
      need = ((unsigned)(lane & 7) >> 2) == (b >> 4);
    } else {
      const unsigned c = (u - 64) >> 1, h = u & 1;
      const auto rw = rsrc(P.w, IN_BYTES);
      unsigned d[8];
#pragma unroll
      for (int j = 0; j < 8; ++j) d[j] = __builtin_amdgcn_raw_buffer_load_b32(rw, (128 * h + 16 * wave + 8 * (lane >> 5) + j) * 4096 + c * 128 + (lane & 31) * 4, 0, 0);
      u32x4 lo, hi;
#pragma unroll
      for (int j = 0; j < 4; ++j) {
        lo[j] = __builtin_amdgcn_perm(d[2 * j + 1], d[2 * j], 0x05040100u);
        hi[j] = __builtin_amdgcn_perm(d[2 * j + 1], d[2 * j], 0x07060302u);
      }
      __builtin_amdgcn_raw_buffer_store_b128(lo, rdat, ubase + (wave * 2 + 0) * 1024 + lane * 16, 0, AUX_SYS);
      __builtin_amdgcn_raw_buffer_store_b128(hi, rdat, ubase + (wave * 2 + 1) * 1024 + lane * 16, 0, AUX_SYS);
      need = ((unsigned)(lane & 3)) == (c >> 3);
    }
    const unsigned call = __builtin_amdgcn_readfirstlane(cnt_raw) + 1;
    WAIT_VM(0);
    __builtin_amdgcn_s_barrier();
    if (wave == 0) {
      if (lane < 8) { if (need && (unsigned)lane != rank) st_sys((unsigned*)(peer_ws + OFF_FS + (rank * 128 + u) * 128), call); }
      else if (lane < 12) st_dev((unsigned*)(myws + OFF_FL + (rank * 128 + u) * 128) + (lane - 8), call);
    }
    if (lane == 0) st_dev(cntp, call);
    STAMP(1);

    const unsigned xcd = bid & 7, idx = bid >> 3;
    const unsigned ta_ = (idx >> 2) * 4 + (xcd >> 1), tb_ = (xcd & 1) * 4 + (idx & 3);
    const unsigned bi = rank >> 2, bj = rank & 3;
    const unsigned s = wave;
    f32x16 acc[2][2];
#pragma unroll
    for (int i = 0; i < 2; ++i)
#pragma unroll
      for (int j = 0; j < 2; ++j)
#pragma unroll
        for (int r = 0; r < 16; ++r) acc[i][j][r] = 0.f;
    bool ok = true;
#pragma unroll 1
    for (int h = 0; h < 2; ++h) {
      const unsigned ux = (16 * bi + ta_) * 2 + h, uw = 64 + (8 * bj + tb_) * 2 + h;
      u32x4 wf[16], xf[16];
      ok = ok && wait_flags<false>((const unsigned*)(myws + OFF_FL + (s * 128 + uw) * 128) + (lane & 3), call, tmo);
#pragma unroll
      for (int f = 0; f < 16; ++f) wf[f] = __builtin_amdgcn_raw_buffer_load_b128(rdat, s * SLICE + uw * UNIT + f * 1024 + lane * 16, 0, AUX_DEV);
      ok = ok && wait_flags<false>((const unsigned*)(myws + OFF_FL + (s * 128 + ux) * 128) + (lane & 3), call, tmo);
      STAMP(2 + h);
#pragma unroll
      for (int f = 0; f < 16; ++f) xf[f] = __builtin_amdgcn_raw_buffer_load_b128(rdat, s * SLICE + ux * UNIT + f * 1024 + lane * 16, 0, AUX_DEV);
      __builtin_amdgcn_sched_barrier(0);
#pragma unroll
      for (int ks = 0; ks < 8; ++ks)
#pragma unroll
        for (int i = 0; i < 2; ++i)
#pragma unroll
          for (int j = 0; j < 2; ++j)
            acc[i][j] = __builtin_amdgcn_mfma_f32_32x32x16_bf16(__builtin_bit_cast(bf16x8, wf[ks * 2 + j]), __builtin_bit_cast(bf16x8, xf[ks * 2 + i]), acc[i][j], 0, 0, 0);
    }
    const unsigned tm = ld_dev(tmo);
    STAMP(4);
    {
      const unsigned m = lane & 31, hh = lane >> 5;
#pragma unroll
      for (int i = 0; i < 2; ++i)
#pragma unroll
        for (int g = 0; g < 4; ++g)
#pragma unroll
          for (int j = 0; j < 2; ++j) {
            f32x4 v = {acc[i][j][4 * g], acc[i][j][4 * g + 1], acc[i][j][4 * g + 2], acc[i][j][4 * g + 3]};
            if (!ok) v = f32x4{__builtin_nanf(""), __builtin_nanf(""), __builtin_nanf(""), __builtin_nanf("")};
            lds[wave * 1024 + ((i * 4 + g) * 2 + j) * 64 + hh * 32 + ((m + 4 * (2 * (g & 1) + hh)) & 31)] = v;
          }
    }
    __syncthreads();
    {
      const unsigned row = 8 * wave + (lane >> 3), c = lane & 7, g = c >> 1, hh = c & 1;
      const unsigned slot = (((row >> 5) * 4 + g) * 2) * 64 + hh * 32 + (((row & 31) + 4 * (c & 3)) & 31);
      f32x4 r0[8], r1[8];
#pragma unroll
      for (int w = 0; w < 8; ++w) { r0[w] = lds[w * 1024 + slot]; r1[w] = lds[w * 1024 + slot + 64]; }
      __builtin_amdgcn_sched_barrier(0);
      f32x4 e0 = r0[0], e1 = r1[0];
#pragma unroll
      for (int w = 1; w < 8; ++w) { e0 += r0[w]; e1 += r1[w]; }
      u32x4 o;
#pragma unroll
      for (int i = 0; i < 4; ++i) {
        bf16x2 t = __builtin_convertvector(f32x2{e0[i], e1[i]}, bf16x2);
        o[i] = __builtin_bit_cast(unsigned, t);
      }
      if (tm == call) o = u32x4{0x7fc07fc0u, 0x7fc07fc0u, 0x7fc07fc0u, 0x7fc07fc0u};
      const auto rout = rsrc(P.out[rank], OUT_BYTES);
      __builtin_amdgcn_raw_buffer_store_b128(o, rout, (1024 * bi + 64 * ta_ + row) * 4096 + (512 * bj + 64 * tb_) * 2 + c * 16, 0, AUX_SYS);
    }
    WAIT_VM(0);
    __builtin_amdgcn_s_barrier();
    if (wave == 0 && lane < 8 && (unsigned)lane != rank) st_sys((unsigned*)(peer_ws + OFF_FO + (rank * 128 + ta_ * 8 + tb_) * 128), call);
    STAMP(5);
    return;
  }

  if (wave != 0) return;
  STAMP(0);
  const unsigned mv = bid - NGEMM;
  const unsigned p = (rank ^ (mv % 7 + 1)) & 7u, j0 = mv / 7;
  const unsigned call = __builtin_amdgcn_readfirstlane(ld_dev(cntp)) + 1;
  if (lane == 0) st_dev(cntp, call);
  const char* const pws = P.ws[p];
  const char* const pout = P.out[p];
  char* const myout = P.out[rank];
  constexpr unsigned NI_IN = 192, NI = 448;
  const unsigned lo_in = lane * 16, lo_out = (lane >> 3) * 4096 + (lane & 7) * 16;
  auto decode = [&](unsigned n) {
    Item it;
    if (n < NI_IN) {
      const unsigned ui = n >> 2, q = n & 3;
      const unsigned u = ui < 16 ? 64 + 16 * (rank & 3) + ui : 32 * (rank >> 2) + (ui - 16);
      it.sb = pws + OFF_DATA; it.db = myws + OFF_DATA; it.cap = NR * SLICE;
      it.off = p * SLICE + u * UNIT + q * 4096; it.tstr = 1024;
      it.flag = (const unsigned*)(myws + OFF_FS + (p * 128 + u) * 128);
      it.done = (unsigned*)(myws + OFF_FL + (p * 128 + u) * 128) + q;
    } else {
      const unsigned m = n - NI_IN, t = m >> 1, hf = m & 1;
      it.sb = pout; it.db = myout; it.cap = OUT_BYTES;
      it.off = (1024 * (p >> 2) + 64 * (t >> 3) + 32 * hf) * 4096 + (512 * (p & 3) + 64 * (t & 7)) * 2; it.tstr = 8 * 4096;
      it.flag = (const unsigned*)(myws + OFF_FO + (p * 128 + t) * 128);
      it.done = nullptr;
    }
    return it;
  };
  u32x4 v[4];
  auto issue = [&](const Item& it, bool good) {
    const auto rs = rsrc(it.sb, good ? it.cap : 0u);
    const unsigned lo = it.done ? lo_in : lo_out;
#pragma unroll
    for (int t = 0; t < 4; ++t) v[t] = __builtin_amdgcn_raw_buffer_load_b128(rs, it.off + t * it.tstr + lo, 0, AUX_SYS);
  };
  unsigned n = j0;
  Item cur = decode(n);
  bool ok = wait_flags<true>(cur.flag, call, tmo);
  STAMP(1);
  issue(cur, ok);
  int dbgi = 2;
  for (;;) {
    const unsigned n2 = n + NW;
    const bool has_next = n2 < NI;
    const Item nxt = decode(has_next ? n2 : n);
    const unsigned f = ld_sys(nxt.flag);
    const unsigned tm = ld_dev(tmo);
    if (__builtin_amdgcn_readfirstlane(tm) == call) ok = false;
    if (!ok) {
#pragma unroll
      for (int t = 0; t < 4; ++t) v[t] = u32x4{0x7fc07fc0u, 0x7fc07fc0u, 0x7fc07fc0u, 0x7fc07fc0u};
    }
    {
      const auto rd = rsrc(cur.db, cur.cap);
      const unsigned lo = cur.done ? lo_in : lo_out;
#pragma unroll
      for (int t = 0; t < 4; ++t) __builtin_amdgcn_raw_buffer_store_b128(v[t], rd, cur.off + t * cur.tstr + lo, 0, AUX_SYS);
    }
    if (!has_next) break;
    if (__builtin_amdgcn_readfirstlane(f) >= call) {
      __builtin_amdgcn_fence(__ATOMIC_ACQUIRE, "wavefront");
      issue(nxt, ok);
      if (cur.done) { WAIT_VM(4); if (lane == 0) st_dev(cur.done, call); }
    } else {
      if (cur.done) { WAIT_VM(0); if (lane == 0) st_dev(cur.done, call); }
      ok = ok && wait_flags<true>(nxt.flag, call, tmo);
      issue(nxt, ok);
    }
    cur = nxt; n = n2;
  }
  WAIT_VM(0);
  STAMP(31);
}

}

extern "C" void kernel_launch(int rank, int nranks, ncclComm_t comm, hipStream_t stream,
                              void* const* d_in, const int* in_sizes, int n_in,
                              void* d_out, int out_size, const int* in_dtypes, int out_dtype,
                              void* const* d_out_all, void* const* d_ws_all, size_t ws_bytes) {
  if (nranks != NR || (unsigned)rank >= (unsigned)NR || n_in < 2 || ws_bytes < WS_NEED || out_size != 2048 * 2048 ||
      in_sizes[0] != 2048 * 256 || in_sizes[1] != 256 * 2048) { fprintf(stderr, "unexpected problem shape\n"); return; }
  Params P;
  P.x = (const char*)d_in[0]; P.w = (const char*)d_in[1];
  for (int p = 0; p < NR; ++p) { P.out[p] = (char*)d_out_all[p]; P.ws[p] = (char*)d_ws_all[p]; }
  P.rank = rank;
  hipLaunchKernelGGL(fused_gemm_ar, dim3(NGEMM + NMOVE), dim3(512), 0, stream, P);
}
```

```cpp
#include <hip/hip_runtime.h>
#include <rccl/rccl.h>
#include <cstdio>
#include <cstdint>

namespace {

typedef unsigned u32x4 __attribute__((ext_vector_type(4)));
typedef float f32x4 __attribute__((ext_vector_type(4)));
typedef float f32x16 __attribute__((ext_vector_type(16)));
typedef __bf16 bf16x8 __attribute__((ext_vector_type(8)));
typedef __bf16 bf16x2 __attribute__((ext_vector_type(2)));
typedef float f32x2 __attribute__((ext_vector_type(2)));
typedef unsigned long long u64;

constexpr int NR = 8;
constexpr unsigned UNIT = 16384;
constexpr unsigned SLICE = 128 * UNIT;
constexpr unsigned OUT_BYTES = 2048u * 2048u * 2u;
constexpr unsigned IN_BYTES = 2048u * 256u * 2u;
constexpr unsigned KB = 1024;
constexpr unsigned OFF_CNT = 0;
constexpr unsigned OFF_TMO = 16 * KB;
constexpr unsigned OFF_LC = 17 * KB;
constexpr unsigned OFF_PLAN = 18 * KB;
constexpr unsigned OFF_FS = 32 * KB;
constexpr unsigned OFF_FL = 160 * KB;
constexpr unsigned OFF_FO = 288 * KB;
constexpr unsigned OFF_FOG = 416 * KB;
constexpr unsigned OFF_FFG = 424 * KB;
constexpr unsigned OFF_DUM = 448 * KB;
constexpr unsigned OFF_DBG = 1024 * KB;
constexpr unsigned OFF_DATA = 2048 * KB;
constexpr size_t WS_NEED = OFF_DATA + (size_t)NR * SLICE;
constexpr int NGEMM = 128;
#define NW 16
#define NPUSH 4
#define T0 50
#define TSNAP 48
#define RPOLL 1
#define GAUX 16
constexpr int NMOVE = 7 * NW;
constexpr unsigned N_IN = (48 - NPUSH) * 4;
constexpr unsigned L_A = N_IN + 256 - 72;
constexpr u64 DEADLINE = 100000000ull;
constexpr int AUX_SYS = 17;
constexpr unsigned NANPK = 0x7fc07fc0u;

#define STAMP(slot) do {} while (0)

struct Params {
  const char* x; const char* w;
  char* myws; char* myout;
  unsigned rank, pad;
  char* out[NR]; char* ws[NR];
  unsigned inv[32];
  unsigned tab[64];
};

__device__ __forceinline__ unsigned ld_sys(const void* p) { return __hip_atomic_load((const unsigned*)p, __ATOMIC_RELAXED, __HIP_MEMORY_SCOPE_SYSTEM); }
__device__ __forceinline__ unsigned ld_dev(const void* p) { return __hip_atomic_load((const unsigned*)p, __ATOMIC_RELAXED, __HIP_MEMORY_SCOPE_AGENT); }
__device__ __forceinline__ void st_sys(void* p, unsigned v) { __hip_atomic_store((unsigned*)p, v, __ATOMIC_RELAXED, __HIP_MEMORY_SCOPE_SYSTEM); }
__device__ __forceinline__ void st_dev(void* p, unsigned v) { __hip_atomic_store((unsigned*)p, v, __ATOMIC_RELAXED, __HIP_MEMORY_SCOPE_AGENT); }
__device__ __forceinline__ unsigned rfl(unsigned v) { return __builtin_amdgcn_readfirstlane(v); }
__device__ __forceinline__ unsigned rdl(unsigned v, unsigned l) { return __builtin_amdgcn_readlane(v, l); }
#define WAIT_VM(n) asm volatile("s_waitcnt vmcnt(" #n ")" ::: "memory")

__device__ __forceinline__ bool spin_ok(u64 t0, unsigned call, unsigned* tmo) {
  const u64 dt = __builtin_amdgcn_s_memrealtime() - t0;
  if (dt > 10000ull) {
    if (rfl(ld_dev(tmo)) == call || dt > DEADLINE) { st_dev(tmo, call); return false; }
    __builtin_amdgcn_s_sleep(32);
  }
  __builtin_amdgcn_s_sleep(1);
  return true;
}
template <bool SYS>
__device__ __forceinline__ u64 wait_flags(const void* p, unsigned call, unsigned* tmo, u64 ma, u64 mb) {
  const u64 t0 = __builtin_amdgcn_s_memrealtime();
  u64 m;
  for (;;) {
    m = __ballot((SYS ? ld_sys(p) : ld_dev(p)) >= call);
    if ((m & ma) == ma || (m & mb) == mb) break;
    if (!spin_ok(t0, call, tmo)) return 0;
  }
  __builtin_amdgcn_fence(__ATOMIC_ACQUIRE, "wavefront");
  return m;
}

__device__ __forceinline__ __amdgpu_buffer_rsrc_t rsrc(const void* base, unsigned bytes) {
  return __builtin_amdgcn_make_buffer_rsrc((void*)base, 0, bytes, 0x00020000);
}

template <int N>
__device__ __forceinline__ void stage_pieces(const Params& P, unsigned u, unsigned e0, unsigned lane, __amdgpu_buffer_rsrc_t rd, unsigned ub) {
  const unsigned h = u & 1;
  u32x4 a[N], b[N];
  if (u < 64) {
    const auto rx = rsrc(P.x, IN_BYTES);
    const unsigned c = lane >> 3;
#pragma unroll
    for (int i = 0; i < N; ++i) {
      const unsigned o = (64 * (u >> 1) + 8 * (e0 + i) + (lane & 7)) * 512 + h * 256 + c * 16;
      a[i] = __builtin_amdgcn_raw_buffer_load_b128(rx, o, 0, 0);
      b[i] = __builtin_amdgcn_raw_buffer_load_b128(rx, o + 128, 0, 0);
    }
#pragma unroll
    for (int i = 0; i < N; ++i) {
      const unsigned row = 8 * (e0 + i) + (lane & 7);
      const unsigned o = ub + ((c >> 1) * 2 + (row >> 5)) * 1024 + ((row & 31) + 32 * (c & 1)) * 16;
      __builtin_amdgcn_raw_buffer_store_b128(a[i], rd, o, 0, AUX_SYS);
      __builtin_amdgcn_raw_buffer_store_b128(b[i], rd, o + 8192, 0, AUX_SYS);
    }
  } else {
    const auto rw = rsrc(P.w, IN_BYTES);
    unsigned d[N][8];
#pragma unroll
    for (int i = 0; i < N; ++i)
#pragma unroll
      for (int j = 0; j < 8; ++j)
        d[i][j] = __builtin_amdgcn_raw_buffer_load_b32(rw, (128 * h + 16 * (e0 + i) + 8 * (lane >> 5) + j) * 4096 + ((u - 64) >> 1) * 128 + (lane & 31) * 4, 0, 0);
    __builtin_amdgcn_sched_barrier(0);
#pragma unroll
    for (int i = 0; i < N; ++i) {
#pragma unroll
      for (int j = 0; j < 4; ++j) {
        a[i][j] = __builtin_amdgcn_perm(d[i][2 * j + 1], d[i][2 * j], 0x05040100u);
        b[i][j] = __builtin_amdgcn_perm(d[i][2 * j + 1], d[i][2 * j], 0x07060302u);
      }
      const unsigned o = ub + (e0 + i) * 2048 + lane * 16;
      __builtin_amdgcn_raw_buffer_store_b128(a[i], rd, o, 0, AUX_SYS);
      __builtin_amdgcn_raw_buffer_store_b128(b[i], rd, o + 1024, 0, AUX_SYS);
    }
  }
}
__device__ __forceinline__ unsigned unit_of(unsigned code, unsigned h, unsigned r) {
  return code < 16 ? (16 * (r >> 2) + code) * 2 + h : 64 + (8 * (r & 3) + (code - 16)) * 2 + h;
}

struct Item { unsigned off, fa, noff, nmask; bool tile; };

__global__ void __launch_bounds__(512, 2) fused_gemm_ar(const Params P) {
  __shared__ f32x4 lds[8192];
  const int lane = threadIdx.x & 63;
  const int wave = __builtin_amdgcn_readfirstlane(threadIdx.x >> 6);
  const int bid = blockIdx.x;
  const unsigned rank = P.rank & 7u;
  char* const myws = P.myws;
  unsigned* const tmo = (unsigned*)(myws + OFF_TMO);
  unsigned* const cntp = (unsigned*)(myws + OFF_CNT) + bid * 8 + wave;

  if (bid < NGEMM) {
    STAMP(0);
    const auto rdat = rsrc(myws + OFF_DATA, NR * SLICE);
    const unsigned u = bid;
    const unsigned cnt_raw = ld_dev(cntp);
    char* const peer_ws = P.ws[lane & 7];
    __builtin_amdgcn_sched_barrier(0);
    stage_pieces<1>(P, u, wave, lane, rdat, rank * SLICE + u * UNIT);
    const unsigned call = rfl(cnt_raw) + 1;
    {
      const bool need = lane < 8 ? ((u < 64 ? ((unsigned)lane >> 2) == (u >> 5) : ((unsigned)lane & 3) == ((u - 64) >> 4)) && (unsigned)lane != rank) : lane < 12;
      char* const fp = lane < 8 ? peer_ws + OFF_FS + (rank * 128 + u) * 128 : myws + OFF_FL + (rank * 128 + u) * 128 + (lane & 3) * 4;
      WAIT_VM(0);
      __builtin_amdgcn_s_barrier();
      if (wave == 0 && need) st_sys(fp, call);
    }
    if (lane == 0) st_dev(cntp, call);
    STAMP(1);

    const unsigned ta_ = ((bid & 7) >> 1) + 4 * (bid >> 5), tb_ = (bid & 1) + 2 * ((bid >> 3) & 3);
    const unsigned ti_ = (P.inv[ta_ * 2 + (tb_ >> 2)] >> (8 * (tb_ & 3))) & 127u;
    const unsigned dbase = wave * SLICE + lane * 16;
    f32x16 acc[2][2];
#pragma unroll
    for (int i = 0; i < 2; ++i)
#pragma unroll
      for (int j = 0; j < 2; ++j)
#pragma unroll
        for (int r = 0; r < 16; ++r) acc[i][j][r] = 0.f;
    bool ok = true;
#pragma unroll 1
    for (int h = 0; h < 2; ++h) {
      const unsigned ux = unit_of(ta_, h, rank), uw = unit_of(16 + tb_, h, rank);
      u32x4 wf[16], xf[16];
      auto loadx = [&]() {
#pragma unroll
        for (int f = 0; f < 16; ++f) xf[f] = __builtin_amdgcn_raw_buffer_load_b128(rdat, dbase + ux * UNIT + f * 1024, 0, GAUX);
      };
      auto loadw = [&]() {
#pragma unroll
        for (int f = 0; f < 16; ++f) wf[f] = __builtin_amdgcn_raw_buffer_load_b128(rdat, dbase + uw * UNIT + f * 1024, 0, GAUX);
      };
      const char* const fl = myws + OFF_FL + (wave * 128 + ((lane & 4) ? uw : ux)) * 128 + (lane & 3) * 4;
      const u64 m = ok ? wait_flags<GAUX == 17>(fl, call, tmo, 0x0f, 0xf0) : 0;
      ok = m != 0;
      if ((m & 0xf0) == 0xf0) {
        loadw();
        if ((m & 0x0f) != 0x0f) ok = wait_flags<GAUX == 17>(fl, call, tmo, 0x0f, 0x0f) != 0;
        STAMP(2 + h);
        loadx();
      } else {
        loadx();
        if (ok) ok = wait_flags<GAUX == 17>(fl, call, tmo, 0xf0, 0xf0) != 0;
        STAMP(2 + h);
        loadw();
      }
      __builtin_amdgcn_sched_barrier(0);
#pragma unroll
      for (int ks = 0; ks < 8; ++ks)
#pragma unroll
        for (int i = 0; i < 2; ++i)
#pragma unroll
          for (int j = 0; j < 2; ++j)
            acc[i][j] = __builtin_amdgcn_mfma_f32_32x32x16_bf16(__builtin_bit_cast(bf16x8, wf[ks * 2 + j]), __builtin_bit_cast(bf16x8, xf[ks * 2 + i]), acc[i][j], 0, 0, 0);
    }
    const unsigned tm = ld_dev(tmo);
    STAMP(4);
    if (!ok) {
#pragma unroll
      for (int i = 0; i < 2; ++i)
#pragma unroll
        for (int j = 0; j < 2; ++j)
#pragma unroll
          for (int r = 0; r < 16; ++r) acc[i][j][r] = __builtin_nanf("");
    }
    {
      const unsigned m = lane & 31, hh = lane >> 5;
#pragma unroll
      for (int i = 0; i < 2; ++i)
#pragma unroll
        for (int g = 0; g < 4; ++g)
#pragma unroll
          for (int j = 0; j < 2; ++j)
            lds[wave * 1024 + ((i * 4 + g) * 2 + j) * 64 + hh * 32 + ((m + 4 * (2 * (g & 1) + hh)) & 31)] =
                f32x4{acc[i][j][4 * g], acc[i][j][4 * g + 1], acc[i][j][4 * g + 2], acc[i][j][4 * g + 3]};
    }
    __syncthreads();
    {
      const unsigned row = 8 * wave + (lane >> 3), c = lane & 7, g = c >> 1, hh = c & 1;
      const unsigned slot = (((row >> 5) * 4 + g) * 2) * 64 + hh * 32 + (((row & 31) + 4 * (c & 3)) & 31);
      f32x4 r0[8], r1[8];
#pragma unroll
      for (int w = 0; w < 8; ++w) { r0[w] = lds[w * 1024 + slot]; r1[w] = lds[w * 1024 + slot + 64]; }
      __builtin_amdgcn_sched_barrier(0);
      f32x4 e0 = r0[0], e1 = r1[0];
#pragma unroll
      for (int w = 1; w < 8; ++w) { e0 += r0[w]; e1 += r1[w]; }
      u32x4 o;
#pragma unroll
      for (int i = 0; i < 4; ++i) o[i] = __builtin_bit_cast(unsigned, __builtin_convertvector(f32x2{e0[i], e1[i]}, bf16x2));
      if (tm == call) o = u32x4{NANPK, NANPK, NANPK, NANPK};
      __builtin_amdgcn_raw_buffer_store_b128(o, rsrc(P.myout, OUT_BYTES), (1024 * (rank >> 2) + 64 * ta_ + row) * 4096 + (512 * (rank & 3) + 64 * tb_) * 2 + c * 16, 0, AUX_SYS);
    }
    {
      const unsigned si = ti_ - T0;
      const unsigned fo = si < 42 ? OFF_FOG + (rank * 8 + si % 7) * 128 + (si / 7) * 8 : OFF_FO + (rank * 128 + ti_) * 128;
      WAIT_VM(0);
      __builtin_amdgcn_s_barrier();
      if (wave == 0 && lane < 8 && (unsigned)lane != rank)
        __hip_atomic_store((u64*)(peer_ws + fo), (u64)call << 32 | call, __ATOMIC_RELAXED, __HIP_MEMORY_SCOPE_SYSTEM);
    }
    STAMP(5);
    return;
  }

  const unsigned mv = bid - NGEMM;
  const unsigned k = mv % 7, j0 = mv / 7;
  const unsigned p = (rank ^ (k + 1)) & 7u;
  if (wave != 0) {
    if (wave > 4 || j0 >= NPUSH) return;
    const unsigned u = unit_of(P.tab[j0 >> 1] & 31u, j0 & 1, p);
    const unsigned cnt_raw = ld_dev(cntp);
    char* const pws = P.ws[p];
    stage_pieces<2>(P, u, 2 * wave - 2, lane, rsrc(pws + OFF_DATA, NR * SLICE), rank * SLICE + u * UNIT);
    const unsigned call = rfl(cnt_raw) + 1;
    WAIT_VM(0);
    if (lane == 0) { st_sys(pws + OFF_FL + (rank * 128 + u) * 128 + (wave - 1) * 4, call); st_dev(cntp, call); }
    STAMP(1);
    return;
  }

  STAMP(0);
  const unsigned tab = P.tab[lane];
  char* const peer_ws = P.ws[lane & 7];
  const unsigned call = rfl(ld_dev(cntp)) + 1;
  if (lane == 0) st_dev(cntp, call);
  const char* const pdat = P.ws[p] + OFF_DATA;
  const char* const pout = P.out[p];
  char* const myout = P.myout;
  if (lane == 0) __hip_atomic_fetch_max((u64*)(myws + OFF_LC + k * 128), (u64)call << 32, __ATOMIC_RELAXED, __HIP_MEMORY_SCOPE_AGENT);
  const unsigned lo_lin = lane * 16, lo_tile = (lane >> 3) * 4096 + (lane & 7) * 16;
  char* const dummy = myws + OFF_DUM + mv * 128;

  auto outoff = [&](unsigned o, unsigned ti, unsigned half) {
    const unsigned t = (rdl(tab, ti & 63) >> (ti < 64 ? 8 : 16)) & 127u;
    return (1024 * (o >> 2) + 64 * (t >> 3) + 32 * half) * 4096 + (512 * (o & 3) + 64 * (t & 7)) * 2;
  };
  auto decode = [&](unsigned n) {
    Item it;
    if (n < N_IN) {
      const unsigned ui = NPUSH + (n >> 2), q = n & 3;
      const unsigned u = unit_of(rdl(tab, ui >> 1) & 31u, ui & 1, rank);
      it.off = p * SLICE + u * UNIT + q * 4096; it.tile = false;
      it.fa = OFF_FS + (p * 128 + u) * 128;
      it.noff = OFF_FL + (p * 128 + u) * 128 + q * 4; it.nmask = 1u << rank;
    } else {
      const unsigned y = n - N_IN, s = y - 2 * T0;
      it.tile = true;
      if (s < 12) {
        it.off = outoff(p, T0 + (s >> 1) * 7 + k, s & 1);
        it.fa = OFF_FOG + (p * 8 + k) * 128 + s * 4;
        it.noff = OFF_FFG + (p * 8 + k) * 128 + s * 4; it.nmask = 0xffu & ~(1u << rank) & ~(1u << p);
      } else {
        const unsigned ti = (y < 2 * T0 ? y : y + 72) >> 1;
        it.off = outoff(p, ti, y & 1);
        it.fa = OFF_FO + (p * 128 + ti) * 128;
        it.noff = 0; it.nmask = 0;
      }
    }
    return it;
  };
  auto notify = [&](unsigned noff, unsigned nmask) {
    st_sys((lane < 8 && ((nmask >> lane) & 1)) ? peer_ws + noff : dummy, call);
  };
  auto claim = [&]() -> unsigned {
    unsigned old = 0, off = OFF_LC + k * 128;
    asm volatile("" : "+v"(off));
    if (lane == 0) old = __hip_atomic_fetch_add((unsigned*)(myws + off), 1u, __ATOMIC_RELAXED, __HIP_MEMORY_SCOPE_AGENT);
    return old;
  };
  u32x4 v[4];
  bool ok = true;
  unsigned cap = 0;
  auto issue = [&](const Item& it) {
    const auto rs = rsrc(it.tile ? pout : pdat, ok ? (it.tile ? OUT_BYTES : NR * SLICE) & cap : 0u);
    const unsigned o0 = it.off + (it.tile ? lo_tile : lo_lin), ts = it.tile ? 8 * 4096 : 1024;
#pragma unroll
    for (int t = 0; t < 4; ++t) v[t] = __builtin_amdgcn_raw_buffer_load_b128(rs, o0 + t * ts, 0, AUX_SYS);
    __builtin_amdgcn_sched_barrier(0);
  };
  auto put = [&](const Item& it) {
    if (!ok) {
#pragma unroll
      for (int t = 0; t < 4; ++t) v[t] = u32x4{NANPK, NANPK, NANPK, NANPK};
    }
    const auto rd = rsrc(it.tile ? myout : myws + OFF_DATA, (it.tile ? OUT_BYTES : NR * SLICE) & cap);
    const unsigned o0 = it.off + (it.tile ? lo_tile : lo_lin), ts = it.tile ? 8 * 4096 : 1024;
#pragma unroll
    for (int t = 0; t < 4; ++t) __builtin_amdgcn_raw_buffer_store_b128(v[t], rd, o0 + t * ts, 0, AUX_SYS);
  };

  const unsigned ep = call & 0x7fffu;
  auto publish = [&](u64 seen) -> u64 {
    const u64 w = __hip_atomic_load((const u64*)(myws + OFF_LC + (lane < 7 ? lane : k) * 128), __ATOMIC_RELAXED, __HIP_MEMORY_SCOPE_AGENT);
    const unsigned prog = (unsigned)(w >> 32) == call ? (unsigned)w : 0u;
    unsigned mx = 0;
#pragma unroll
    for (int i = 0; i < 7; ++i) { const unsigned x = rdl(prog, i); mx = x > mx ? x : mx; }
    u64 val = (u64)ep << 49;
#pragma unroll
    for (int i = 0; i < 7; ++i) { const unsigned d = mx - rdl(prog, i); val |= (u64)(d > 127 ? 127u : d) << (7 * i); }
    for (;;) {
      u64 old = seen;
      if (lane == 0) __hip_atomic_compare_exchange_strong((u64*)(myws + OFF_PLAN), &old, val, __ATOMIC_RELAXED, __ATOMIC_RELAXED, __HIP_MEMORY_SCOPE_AGENT);
      old = (u64)rfl((unsigned)(old >> 32)) << 32 | rfl((unsigned)old);
      if (old == seen) return val;
      if ((unsigned)(old >> 49) == ep) return old;
      seen = old;
    }
  };
  const unsigned kk_l = (lane % 6) < k ? lane % 6 : lane % 6 + 1;
  const unsigned q_l = (rank ^ (kk_l + 1)) & 7u;
  const unsigned line_l = (lane < 6 ? p : q_l) * 8 + ((p ^ q_l) - 1);
  unsigned seg_end = 0, seg_beg = 0, L = L_A;
  bool planned = false;
  auto apply = [&](u64 plan) {
    const int dk = (int)(plan >> (7 * k)) & 127, dq = (int)(plan >> (7 * kk_l)) & 127;
    const int dl = k < kk_l ? dk - dq : dq - dk;
    int sl = (171 + 7 * 32 - 2 * dl + 4) / 14 - 16;
    sl = sl < 0 ? 0 : sl > 24 ? 24 : sl;
    const int sh = k < kk_l ? sl : 24 - sl;
    const unsigned len = lane < 6 ? (sh < 12 ? sh : 12) : lane < 12 ? (sh > 12 ? sh - 12 : 0) : 0;
    unsigned run = 0;
#pragma unroll
    for (int i = 0; i < 12; ++i) { if (lane == i) seg_beg = run; run += rdl(len, i); }
    seg_end = lane < 12 ? seg_beg + len : 0xffffu;
    L = L_A + run;
    planned = true;
  };
  auto decode_b = [&](unsigned e) {
    const unsigned l = __builtin_popcountll(__ballot(seg_end <= e));
    const unsigned sub = e - rdl(seg_beg, l), ln = rdl(line_l, l);
    const unsigned s = l < 6 ? sub : 11 - sub;
    Item it;
    it.tile = true; it.off = outoff(ln >> 3, T0 + (s >> 1) * 7 + (ln & 7), s & 1);
    it.fa = (l < 6 ? OFF_FOG : OFF_FFG) + ln * 128 + s * 4; it.noff = 0; it.nmask = 0;
    return it;
  };

  Item cur; cur.off = 0; cur.fa = 0; cur.noff = 0; cur.nmask = 0; cur.tile = false;
  unsigned pnoff = 0, pnmask = 0;
  unsigned tk = j0 - NW;
  u64 plan = 0;
  for (;;) {
    issue(cur);
    const unsigned idx = rfl(tk) + NW;
    if (!planned && idx >= L_A - TSNAP) {
      if ((unsigned)(plan >> 49) != ep && (idx == L_A - TSNAP || idx >= L_A)) plan = publish(plan);
      if ((unsigned)(plan >> 49) == ep) apply(plan);
    }
    const bool last = idx >= L;
    const Item nxt = idx >= L_A && !last ? decode_b(idx - L_A) : decode(last ? 0u : idx);
    const char* nfl = myws + ((lane & 2) ? OFF_PLAN + (lane & 1) * 4 : (lane & 1) ? OFF_TMO : nxt.fa);
    const unsigned f = ld_sys(nfl);
    WAIT_VM(5);
    notify(pnoff, pnmask);
    __builtin_amdgcn_sched_barrier(0);
    if (rdl(f, 1) == call) ok = false;
    const bool ready = rfl(f) >= call;
    if (!last) tk = claim();
    put(cur);
    pnoff = cur.noff; pnmask = cur.nmask;
    if (last) break;
    if (!planned) plan = (u64)rdl(f, 3) << 32 | rdl(f, 2);
    if (!ready) {
      WAIT_VM(0);
      notify(pnoff, pnmask);
      pnmask = 0;
      if (RPOLL && cap == 0) nfl = P.ws[p] + (nxt.fa - OFF_FS + OFF_FL);
      if (ok) ok = wait_flags<true>(nfl, call, tmo, 1, 1) != 0;
    }
    __builtin_amdgcn_fence(__ATOMIC_ACQUIRE, "wavefront");
    cur = nxt; cap = 0xffffffffu;
  }
  if (pnmask) { WAIT_VM(0); notify(pnoff, pnmask); }
}

}

extern "C" void kernel_launch(int rank, int nranks, ncclComm_t comm, hipStream_t stream,
                              void* const* d_in, const int* in_sizes, int n_in,
                              void* d_out, int out_size, const int* in_dtypes, int out_dtype,
                              void* const* d_out_all, void* const* d_ws_all, size_t ws_bytes) {
  if (nranks != NR || (unsigned)rank >= (unsigned)NR || n_in < 2 || ws_bytes < WS_NEED || out_size != 2048 * 2048 ||
      in_sizes[0] != 2048 * 256 || in_sizes[1] != 256 * 2048) { fprintf(stderr, "unexpected problem shape\n"); return; }
  Params P;
  P.x = (const char*)d_in[0]; P.w = (const char*)d_in[1];
  for (int p = 0; p < NR; ++p) { P.out[p] = (char*)d_out_all[p]; P.ws[p] = (char*)d_ws_all[p]; }
  P.myws = P.ws[rank]; P.myout = P.out[rank];
  P.rank = rank; P.pad = 0;
  const unsigned char bo[24] = {16, 0, 17, 1, 2, 18, 3, 19, 4, 20, 5, 21, 6, 22, 7, 23, 8, 9, 10, 11, 12, 13, 14, 15};
  unsigned char ord[128]; int no = 0, na = 0, nb = 0;
  for (int n = 0; n < 24; ++n) {
    if (bo[n] < 16) { for (int b = 0; b < nb; ++b) ord[no++] = na * 8 + b; ++na; }
    else { for (int a = 0; a < na; ++a) ord[no++] = a * 8 + nb; ++nb; }
  }
  for (int i = 0; i < 32; ++i) P.inv[i] = 0;
  for (int i = 0; i < 128; ++i) P.inv[ord[i] >> 2] |= (unsigned)i << (8 * (ord[i] & 3));
  for (int l = 0; l < 64; ++l) P.tab[l] = bo[l < 24 ? l : 0] | (unsigned)ord[l] << 8 | (unsigned)ord[64 + l] << 16;
  hipLaunchKernelGGL(fused_gemm_ar, dim3(NGEMM + NMOVE), dim3(512), 0, stream, P);
}
```

```cpp
#include <hip/hip_runtime.h>
#include <rccl/rccl.h>
#include <cstdio>
#include <cstdint>

namespace {

typedef unsigned u32x4 __attribute__((ext_vector_type(4)));
typedef float f32x4 __attribute__((ext_vector_type(4)));
typedef float f32x16 __attribute__((ext_vector_type(16)));
typedef __bf16 bf16x8 __attribute__((ext_vector_type(8)));
typedef __bf16 bf16x2 __attribute__((ext_vector_type(2)));
typedef float f32x2 __attribute__((ext_vector_type(2)));
typedef unsigned long long u64;

#define NW 16
#define SH 2
#define SHQ 536
#define NPUSH 4
#define T0 60
#define OWNPOS 120
#define LSL 16
#define NCRIT 8
#define TSNAP 48
#define GAUX 17
#define GNAP 1
#define PB0 96
#define DLEAD 64
#define RPOLL 1
#define PSL 6
#define NBW 4
#define C0 (2 * NW + NW / 2)

constexpr int NR = 8;
constexpr unsigned UNIT = 16384;
constexpr unsigned SLICE = 128 * UNIT;
constexpr unsigned OUT_BYTES = 2048u * 2048u * 2u;
constexpr unsigned IN_BYTES = 2048u * 256u * 2u;
constexpr unsigned KB = 1024;
constexpr unsigned OFF_CNT = 0;
constexpr unsigned OFF_TMO = 16 * KB;
constexpr unsigned OFF_LC = 17 * KB;
constexpr unsigned OFF_DEC = 18 * KB + 256;
constexpr unsigned OFF_WV = 19 * KB;
constexpr unsigned OFF_FS = 32 * KB;
constexpr unsigned OFF_FL = 160 * KB;
constexpr unsigned OFF_FO = 288 * KB;
constexpr unsigned OFF_FC = 416 * KB;
constexpr unsigned OFF_FF = 424 * KB;
constexpr unsigned OFF_FG = 432 * KB;
constexpr unsigned OFF_TM2 = 448 * KB;
constexpr unsigned OFF_SW = 512 * KB;
constexpr unsigned OFF_DBG = 1024 * KB;
constexpr unsigned OFF_DATA = 2048 * KB;
constexpr size_t WS_NEED = OFF_DATA + (size_t)NR * SLICE;
constexpr int NGEMM = 128, NMOVE = 7 * NW;
constexpr unsigned FO1 = 32, FOG = 4;
constexpr unsigned N1 = (2 * (24 - SH) - NPUSH) * 4;
constexpr unsigned TBL = 704;
constexpr u64 DEADLINE = 100000000ull;
constexpr int AUX_SYS = 17;
constexpr unsigned NANPK = 0x7fc07fc0u;
static_assert(NGEMM + NMOVE + 1 <= 256 && NMOVE <= 128 && N1 + 16 * SH + 16 * (16 + SH) - 72 + 144 < TBL, "layout");

#define STAMP(slot) do {} while (0)

struct Params {
  const char* x; const char* w;
  char* myws; char* myout;
  unsigned rank, pad;
  char* ws[NR];
  char* lws[8]; char* lout[8];
  unsigned task[128];
  unsigned wfirst[8];
  unsigned inv[16];
  unsigned tab[64];
};

__device__ __forceinline__ unsigned ld_sys(const void* p) { return __hip_atomic_load((const unsigned*)p, __ATOMIC_RELAXED, __HIP_MEMORY_SCOPE_SYSTEM); }
__device__ __forceinline__ unsigned ld_dev(const void* p) { return __hip_atomic_load((const unsigned*)p, __ATOMIC_RELAXED, __HIP_MEMORY_SCOPE_AGENT); }
__device__ __forceinline__ u64 ld_dev64(const void* p) { return __hip_atomic_load((const u64*)p, __ATOMIC_RELAXED, __HIP_MEMORY_SCOPE_AGENT); }
__device__ __forceinline__ void st_sys(void* p, unsigned v) { __hip_atomic_store((unsigned*)p, v, __ATOMIC_RELAXED, __HIP_MEMORY_SCOPE_SYSTEM); }
__device__ __forceinline__ void st_dev(void* p, unsigned v) { __hip_atomic_store((unsigned*)p, v, __ATOMIC_RELAXED, __HIP_MEMORY_SCOPE_AGENT); }
__device__ __forceinline__ unsigned rfl(unsigned v) { return __builtin_amdgcn_readfirstlane(v); }
__device__ __forceinline__ u64 rfl64(u64 v) { return (u64)rfl((unsigned)(v >> 32)) << 32 | rfl((unsigned)v); }
__device__ __forceinline__ unsigned rdl(unsigned v, unsigned l) { return __builtin_amdgcn_readlane(v, l); }
__device__ __forceinline__ unsigned pick(unsigned v, unsigned l) { return __builtin_amdgcn_ds_bpermute(l * 4, v); }
#define STR_(x) #x
#define STR(x) STR_(x)
#define WAIT_VM(n) asm volatile("s_waitcnt vmcnt(" #n ")" ::: "memory")

template <int NAP>
__device__ __forceinline__ bool spin_ok(u64 t0, unsigned call, char* myws) {
  const u64 dt = __builtin_amdgcn_s_memrealtime() - t0;
  if (dt > 10000ull) {
    if (rfl(ld_dev(myws + OFF_TMO)) == call || dt > DEADLINE) {
      char* const c = myws + OFF_TM2 + (threadIdx.x & 63) * 128;
      st_dev(c, call); st_dev(c + 64 * 128, call); st_dev(myws + OFF_TMO, call);
      return false;
    }
    __builtin_amdgcn_s_sleep(32);
  }
  __builtin_amdgcn_s_sleep(NAP);
  return true;
}
template <bool SYS>
__device__ __forceinline__ bool wait_flags(const void* p, unsigned call, char* myws, u64 mask) {
  const u64 t0 = __builtin_amdgcn_s_memrealtime();
  while ((__ballot((SYS ? ld_sys(p) : ld_dev(p)) >= call) & mask) != mask)
    if (!spin_ok<SYS ? 1 : GNAP>(t0, call, myws)) return false;
  __builtin_amdgcn_fence(__ATOMIC_ACQUIRE, "wavefront");
  return true;
}
__device__ __forceinline__ unsigned wait_word(unsigned off, unsigned call, char* myws) {
  const u64 t0 = __builtin_amdgcn_s_memrealtime();
  for (;;) {
    const u64 v = rfl64(ld_dev64(myws + off));
    if ((unsigned)(v >> 32) == call) return (unsigned)v;
    if (!spin_ok<8>(t0, call, myws)) return 0;
  }
}

__device__ __forceinline__ __amdgpu_buffer_rsrc_t rsrc(const void* base, unsigned bytes) {
  return __builtin_amdgcn_make_buffer_rsrc((void*)base, 0, bytes, 0x00020000);
}

template <class F>
__device__ __forceinline__ void stage_piece(const Params& P, unsigned u, unsigned e, unsigned lane, char* base, unsigned ub, u32x4* sc, F&& after_loads) {
  const unsigned h = u & 1;
  const bool isx = u < 64;
  const unsigned o = isx ? (64 * (u >> 1) + 32 * (e & 1) + (lane >> 2)) * 512 + h * 256 + (e >> 1) * 64 + (lane & 3) * 16
                         : (128 * h + 16 * e + (lane >> 3)) * 4096 + ((u - 64) >> 1) * 128 + (lane & 7) * 16;
  const auto rs = rsrc(isx ? P.x : P.w, IN_BYTES);
  u32x4 a = __builtin_amdgcn_raw_buffer_load_b128(rs, o, 0, 0);
  u32x4 b = __builtin_amdgcn_raw_buffer_load_b128(rs, o + (isx ? 16 * 512 : 8 * 4096), 0, 0);
  __builtin_amdgcn_sched_barrier(0);
  after_loads();
  __builtin_amdgcn_sched_barrier(0);
  unsigned oa, ob;
  if (isx) {
    const unsigned wi = (lane & 3) * 33 + (lane >> 2);
    sc[wi] = a; sc[wi + 16] = b;
    const unsigned ri = (lane >> 5) * 33 + (lane & 31);
    a = sc[ri]; b = sc[ri + 66];
    oa = ub + ((e & 6) * 2 + (e & 1)) * 1024 + lane * 16; ob = oa + 2048;
  } else {
    sc[lane] = a; sc[lane + 64] = b;
    const unsigned* const sd = (const unsigned*)sc + (lane >> 5) * 256 + (lane & 31);
    unsigned d[8];
#pragma unroll
    for (int j = 0; j < 8; ++j) d[j] = sd[j * 32];
#pragma unroll
    for (int j = 0; j < 4; ++j) {
      a[j] = __builtin_amdgcn_perm(d[2 * j + 1], d[2 * j], 0x05040100u);
      b[j] = __builtin_amdgcn_perm(d[2 * j + 1], d[2 * j], 0x07060302u);
    }
    oa = ub + e * 2048 + lane * 16; ob = oa + 1024;
  }
  const auto rd = rsrc(base + OFF_DATA, NR * SLICE);
  __builtin_amdgcn_raw_buffer_store_b128(a, rd, oa, 0, AUX_SYS);
  __builtin_amdgcn_raw_buffer_store_b128(b, rd, ob, 0, AUX_SYS);
}
__host__ __device__ __forceinline__ unsigned gband(unsigned a, unsigned r) { return (r & 4) ? 31 - a : a; }
__host__ __device__ __forceinline__ unsigned unit_of(unsigned code, unsigned h, unsigned r) {
  return code < 32 ? gband(code, r) * 2 + h : 64 + (8 * (r & 3) + (code - 32)) * 2 + h;
}
__device__ __forceinline__ unsigned fo_of(unsigned ti) { return ti < FO1 ? ti : FO1 + (ti - FO1) / FOG; }
__device__ __forceinline__ unsigned bands_of(unsigned dec, unsigned r) {
  const int d = (int)(((dec >> (4 * (r & 3))) & 15u) ^ 8u) - 8;
  return (r & 4) ? 16 + d : 16 - d;
}

template <bool B> struct BoolC { static constexpr bool value = B; };

__global__ void __launch_bounds__(512, 2) fused_gemm_ar(const Params P) {
  __shared__ f32x4 lds[8192];
  const int lane = threadIdx.x & 63;
  const int wave = __builtin_amdgcn_readfirstlane(threadIdx.x >> 6);
  const int bid = blockIdx.x;
  const unsigned rank = P.rank & 7u;
  char* const myws = P.myws;
  char* const myout = P.myout;
  const unsigned task = P.task[bid & 127];
  const unsigned mv = (bid - NGEMM) & 127, k = mv % 7, j0 = mv / 7;
  asm volatile("" :: "s"(P.x), "s"(P.w), "s"(myws), "s"(myout), "s"(rank), "s"(task));
  unsigned* const cntp = (unsigned*)(myws + OFF_CNT) + bid * 8 + wave;

  if (bid < NGEMM) {
    unsigned cnt_raw;
    char* peer_ws;
    {
      u32x4* const scr = (u32x4*)lds + wave * 1024;
      if ((task >> 19) & 1) {
        const unsigned u = (task >> 8) & 127u;
        char* const dws = P.ws[(task >> 16) & 7u];
        stage_piece(P, u, wave, lane, dws, rank * SLICE + u * UNIT, scr, [&]() { cnt_raw = ld_dev(cntp); });
        WAIT_VM(0);
        __builtin_amdgcn_s_barrier();
        if (wave == 0 && lane < 4) st_sys(dws + OFF_FL + (rank * 128 + u) * 128 + lane * 4, rfl(cnt_raw) + 1);
        STAMP(6);
      } else if ((task >> 20) & 1) __builtin_amdgcn_s_sleep(LSL);
      const unsigned u = task & 127u;
      stage_piece(P, u, wave, lane, myws, rank * SLICE + u * UNIT, scr, [&]() { cnt_raw = ld_dev(cntp); peer_ws = P.ws[lane & 7]; });
      const bool need = lane < 8 ? ((u < 64 ? gband(u >> 1, lane) < 16 + SH : ((unsigned)lane & 3) == ((u - 64) >> 4)) && (unsigned)lane != rank) : lane < 12;
      char* const fp = lane < 8 ? peer_ws + OFF_FS + (rank * 128 + u) * 128 : myws + OFF_FL + (rank * 128 + u) * 128 + (lane & 3) * 4;
      WAIT_VM(0);
      if (RPOLL && !((task >> 19) & 3) && lane < 8 && need)
        st_sys(myws + OFF_SW + (lane * 128 + u) * 128 + wave * 4, rfl(cnt_raw) + 1);
      __builtin_amdgcn_s_barrier();
      if (wave == 0 && need) st_sys(fp, rfl(cnt_raw) + 1);
    }
    const unsigned call = rfl(cnt_raw) + 1;
    if (lane == 0) st_dev(cntp, call);
    STAMP(1);

    unsigned ta_ = ((bid & 7) >> 1) + 4 * (bid >> 5), tb_ = (bid & 1) + 2 * ((bid >> 3) & 3);
#pragma unroll 1
    for (;;) {
    const unsigned ti_ = ta_ < 8 ? (P.inv[(ta_ * 2 + (tb_ >> 2)) & 15] >> (8 * (tb_ & 3))) & 127u : ta_ * 8 + tb_;
    const bool wfirst = ta_ >= 8 || ((P.wfirst[ta_ & 7] >> tb_) & 1);
    const unsigned ue = wfirst ? unit_of(32 + tb_, 0, rank) : unit_of(ta_, 0, rank);
    const unsigned ul = wfirst ? unit_of(ta_, 0, rank) : unit_of(32 + tb_, 0, rank);
    if (SH && ta_ >= 16 - SH && ta_ >= bands_of(wait_word(OFF_DEC, call, myws), rank)) return;
    const auto rdat = rsrc(myws + OFF_DATA, NR * SLICE);
    const unsigned rot = (bid >> 5) + (bid >> 3);
    const unsigned dbase = wave * SLICE + lane * 16;
    auto ld = [&](unsigned u, int f) {
      return __builtin_amdgcn_raw_buffer_load_b128(rdat, dbase + (f & 3) * 1024, u * UNIT + ((((f >> 2) + rot) & 3) << 12), GAUX);
    };
    f32x16 acc[2][2];
    u32x4* const stash = (u32x4*)lds + wave * 1024 + lane;
    bool ok = wait_flags<false>(myws + OFF_FL + (wave * 128 + ue + ((lane >> 2) & 1)) * 128 + (lane & 3) * 4, call, myws, 0xff);
    if (ti_ < 128) STAMP(2);
    auto body = [&](auto WF) {
      constexpr bool W1 = decltype(WF)::value;
      u32x4 r1[16], r2[16];
#pragma unroll
      for (int f = 0; f < 16; ++f) r2[f] = ld(ue + 1, f);
#pragma unroll
      for (int f = 0; f < 16; ++f) r1[f] = ld(ue, f);
      __builtin_amdgcn_sched_barrier(0);
#pragma unroll
      for (int f = 0; f < 16; ++f) stash[f * 64] = r2[f];
      if (ok) ok = wait_flags<false>(myws + OFF_FL + (wave * 128 + ul + ((lane >> 2) & 1)) * 128 + (lane & 3) * 4, call, myws, 0xff);
      if (ti_ < 128) STAMP(3);
#pragma unroll
      for (int f = 0; f < 16; ++f) r2[f] = ld(ul, f);
      __builtin_amdgcn_sched_barrier(0);
#pragma unroll
      for (int h = 0; h < 2; ++h)
#pragma unroll
        for (int ks = 0; ks < 8; ++ks) {
#pragma unroll
          for (int i = 0; i < 2; ++i)
#pragma unroll
            for (int j = 0; j < 2; ++j) {
              const bf16x8 wv = __builtin_bit_cast(bf16x8, W1 ? r1[ks * 2 + j] : r2[ks * 2 + j]);
              const bf16x8 xv = __builtin_bit_cast(bf16x8, W1 ? r2[ks * 2 + i] : r1[ks * 2 + i]);
              acc[i][j] = __builtin_amdgcn_mfma_f32_32x32x16_bf16(wv, xv, (h | ks) ? acc[i][j] : f32x16{}, 0, 0, 0);
            }
          if (h == 0) {
#pragma unroll
            for (int t = 0; t < 2; ++t) { r2[ks * 2 + t] = ld(ul + 1, ks * 2 + t); r1[ks * 2 + t] = stash[(ks * 2 + t) * 64]; }
            __builtin_amdgcn_sched_barrier(0);
          }
        }
    };
    if (wfirst) body(BoolC<true>{}); else { asm volatile("; row band first" ::: "memory"); body(BoolC<false>{}); }
    const unsigned tm = ld_dev(myws + OFF_TMO);
    if (ti_ < 128) STAMP(4);
    if (!ok) {
#pragma unroll
      for (int i = 0; i < 2; ++i)
#pragma unroll
        for (int j = 0; j < 2; ++j)
#pragma unroll
          for (int r = 0; r < 16; ++r) acc[i][j][r] = __builtin_nanf("");
    }
    {
      const unsigned m = lane & 31, hh = lane >> 5;
#pragma unroll
      for (int i = 0; i < 2; ++i)
#pragma unroll
        for (int g = 0; g < 4; ++g)
#pragma unroll
          for (int j = 0; j < 2; ++j)
            lds[wave * 1024 + ((i * 4 + g) * 2 + j) * 64 + hh * 32 + ((m + 4 * (2 * (g & 1) + hh)) & 31)] =
                f32x4{acc[i][j][4 * g], acc[i][j][4 * g + 1], acc[i][j][4 * g + 2], acc[i][j][4 * g + 3]};
    }
    __syncthreads();
    {
      const unsigned row = 8 * wave + (lane >> 3), c = lane & 7, g = c >> 1, hh = c & 1;
      const unsigned slot = (((row >> 5) * 4 + g) * 2) * 64 + hh * 32 + (((row & 31) + 4 * (c & 3)) & 31);
      f32x4 r0[8], r1[8];
#pragma unroll
      for (int w = 0; w < 8; ++w) { r0[w] = lds[w * 1024 + slot]; r1[w] = lds[w * 1024 + slot + 64]; }
      __builtin_amdgcn_sched_barrier(0);
      f32x4 e0 = r0[0], e1 = r1[0];
#pragma unroll
      for (int w = 1; w < 8; ++w) { e0 += r0[w]; e1 += r1[w]; }
      u32x4 o;
#pragma unroll
      for (int i = 0; i < 4; ++i) o[i] = __builtin_bit_cast(unsigned, __builtin_convertvector(f32x2{e0[i], e1[i]}, bf16x2));
      if (tm == call) o = u32x4{NANPK, NANPK, NANPK, NANPK};
      __builtin_amdgcn_raw_buffer_store_b128(o, rsrc(myout, OUT_BYTES), (64 * gband(ta_, rank) + row) * 4096 + (512 * (rank & 3) + 64 * tb_) * 2 + c * 16, 0, AUX_SYS);
    }
    WAIT_VM(0);
    __builtin_amdgcn_s_barrier();
    if (wave == 0) {
      bool tell = true;
      if (ti_ >= FO1) {
        unsigned old = 0, off = OFF_FC + ((ti_ - FO1) / FOG) * 128;
        asm volatile("" : "+v"(off));
        if (lane == 0) old = __hip_atomic_fetch_add((unsigned*)(myws + off), 1u, __ATOMIC_RELAXED, __HIP_MEMORY_SCOPE_AGENT);
        tell = rfl(old) % FOG == FOG - 1;
      }
      if (tell && lane < 8 && (unsigned)lane != rank) st_sys(peer_ws + OFF_FO + (rank * 128 + fo_of(ti_)) * 128, call);
    }
    if (ti_ < 128) STAMP(5);
    if (ti_ >= 8 * SH) return;
    ta_ = 16 + (ti_ >> 3); tb_ = ti_ & 7;
    }
  }

  const u64 t_in = __builtin_amdgcn_s_memrealtime();
  auto list_len = [&](unsigned nr, unsigned np) { return N1 + 8 * (nr - (16 - SH)) + 16 * np - 72; };
  if (bid == NGEMM + NMOVE) {
    if (wave) return;
    char* const peer_ws = P.ws[lane & 7];
    const unsigned call = rfl(ld_dev(cntp)) + 1;
    if (lane == 0) st_dev(cntp, call);
    const unsigned kl = lane < 7 ? lane : 0;
    auto drawn = [&]() { const u64 w = ld_dev64(myws + OFF_LC + kl * 128); return (unsigned)(w >> 32) == call ? (unsigned)w + NW : 0u; };
    auto post = [&](unsigned off, u64 v) {
      __hip_atomic_store((u64*)(myws + OFF_TM2 + lane * 128 + off), v, __ATOMIC_RELAXED, __HIP_MEMORY_SCOPE_AGENT);
      __hip_atomic_store((u64*)(myws + OFF_TM2 + (64 + lane) * 128 + off), v, __ATOMIC_RELAXED, __HIP_MEMORY_SCOPE_AGENT);
    };
    bool alive = true;
    unsigned dec = 0;
    if (SH) {
      unsigned ts = 0;
      for (;;) {
        const bool go = drawn() >= C0;
        const u64 dt = __builtin_amdgcn_s_memrealtime() - t_in;
        if (go && !ts) ts = dt > 100000 ? 100000u : (unsigned)dt + 1;
        if (__all(ts != 0) || !(alive = spin_ok<2>(t_in, call, myws))) break;
      }
      unsigned sum = 0;
#pragma unroll
      for (int i = 0; i < 7; ++i) sum += rdl(ts, i);
      if (lane < 8) __hip_atomic_store((u64*)(peer_ws + OFF_WV + rank * 128), (u64)call << 32 | sum, __ATOMIC_RELAXED, __HIP_MEMORY_SCOPE_SYSTEM);
      u64 v;
      for (;;) {
        v = __hip_atomic_load((const u64*)(myws + OFF_WV + (lane & 7) * 128), __ATOMIC_RELAXED, __HIP_MEMORY_SCOPE_SYSTEM);
        if (__all((unsigned)(v >> 32) == call) || !(alive = alive && spin_ok<2>(t_in, call, myws))) break;
      }
#pragma unroll
      for (int i = 0; i < 4; ++i) {
        const int df = (int)rdl((unsigned)v, i) - (int)rdl((unsigned)v, i + 4);
        int d = (df + (df < 0 ? -SHQ / 2 : SHQ / 2)) / SHQ;
        d = d < -SH ? -SH : d > SH ? SH : d;
        dec |= ((unsigned)d & 15u) << (4 * i);
      }
      if (!alive) dec = 0;
      post(16, (u64)call << 32 | dec);
      if (lane == 0) __hip_atomic_store((u64*)(myws + OFF_DEC), (u64)call << 32 | dec, __ATOMIC_RELAXED, __HIP_MEMORY_SCOPE_AGENT);
    }
    const unsigned la_l = list_len(bands_of(dec, rank), bands_of(dec, rank ^ (kl + 1)));
    const auto rff = rsrc(myws + OFF_FF, 7 * 128);
    unsigned todo = 0xffu;
    while (todo) {
      const unsigned left = la_l + 512 - drawn();
      const u32x4 ff = __builtin_amdgcn_raw_buffer_load_b128(rff, (lane >> 2) * 128 + ((lane & 3) == 3 ? 0 : lane & 3) * 16, 0, AUX_SYS);
      if ((todo & 128) && (__any(left <= 512 + TSNAP) || !alive)) {
        unsigned mn = 0xffffu;
#pragma unroll
        for (int i = 0; i < 7; ++i) { const unsigned x = rdl(left, i); mn = x < mn ? x : mn; }
        u64 val = (u64)(call & 0x7fffu) << 49;
#pragma unroll
        for (int i = 0; i < 7; ++i) { const unsigned d = rdl(left, i) - mn; val |= (u64)(d > 127 ? 127u : d) << (7 * i); }
        post(8, val);
        todo &= 127u;
      }
      const unsigned have = (unsigned)__ballot(ff[0] >= call && ff[1] >= call && ff[2] >= call && ff[3] >= call);
#pragma unroll
      for (int i = 0; i < 7; ++i)
        if (((todo >> i) & 1) && ((have >> (4 * i)) & 15u) == 15u) {
          if (lane < 8 && (unsigned)lane != rank) st_sys(peer_ws + OFF_FG + ((rank ^ (i + 1)) * 8 + i) * 128, call);
          todo &= ~(1u << i);
        }
      if (!alive) break;
      if (todo) alive = spin_ok<2>(t_in, call, myws);
    }
    return;
  }

  if (wave >= NBW) return;
  const char* const pws = P.lws[k];
  const char* const pout = P.lout[k];
  const unsigned p = (rank ^ (k + 1)) & 7u;
  const unsigned tab = P.tab[lane];
  unsigned cnt_raw = 0;
  if (wave < 2) cnt_raw = ld_dev(cntp);
  typedef __attribute__((address_space(3))) volatile u32x4 lds_u32x4;
  lds_u32x4* const tbl = (lds_u32x4*)lds;
  auto tile_entry = [&](unsigned o, unsigned ti, unsigned half, unsigned fa, unsigned noff) {
    const unsigned t0 = (pick(tab, ti & 63) >> 8) & 63u, t = ti < 64 ? t0 : ti;
    return u32x4{(64 * gband(t >> 3, o) + 32 * half) * 4096 + (512 * (o & 3) + 64 * (t & 7)) * 2, fa, noff, 2u};
  };
  auto entry = [&](unsigned n, unsigned nr, unsigned np) {
    const unsigned nin = N1 + 8 * (nr - (16 - SH));
    const unsigned ui = NPUSH + (n >> 2), u = unit_of(pick(tab, (ui >> 1) & 63) & 63u, ui & 1, rank) & 127u;
    const unsigned y = n - nin, s = y - OWNPOS, z = y < OWNPOS ? y : y - 12;
    const unsigned ti = s < 12 ? T0 + (s >> 1) * 7 + k : (z < 2 * T0 ? z : z + 84) >> 1;
    const u32x4 te = tile_entry(p, ti, y & 1, OFF_FO + (p * 128 + fo_of(ti)) * 128, s < 12 ? OFF_FF + k * 128 + s * 4 : 0u);
    if (n < nin) return u32x4{p * SLICE + u * UNIT + (n & 3) * 4096, OFF_FS + (p * 128 + u) * 128, OFF_FL + (p * 128 + u) * 128 + (n & 3) * 4, 1u};
    return n < list_len(nr, np) ? te : u32x4{0, 0, 0, 0};
  };
  typedef __attribute__((address_space(3))) volatile unsigned lds_u32;
  lds_u32* const box = (lds_u32*)(tbl + TBL);
#pragma unroll 1
  for (unsigned n0 = wave * 64; n0 < TBL; n0 += NBW * 64) {
    u32x4 e = {0, 0, 0, 0};
    if (n0 < (SH ? N1 : N1 + 184)) e = entry(n0 + lane, 16, 16);
    tbl[n0 + lane] = (SH == 0 || n0 + lane < N1) ? e : u32x4{0, 0, 0, 0};
  }
  const unsigned call = rfl(cnt_raw) + 1;
  unsigned fprev = lane == 3 ? (call + 1) << 17 : call - 1;
  if (threadIdx.x < 8) box[threadIdx.x] = fprev;
  if (wave < 2 && lane == 0) {
    st_dev(cntp, call);
    if (wave == 0) __hip_atomic_fetch_max((u64*)(myws + OFF_LC + k * 128), (u64)call << 32, __ATOMIC_RELAXED, __HIP_MEMORY_SCOPE_AGENT);
  }
  __syncthreads();
  if (wave > 1) return;

  if (wave == 1) {
    typedef __attribute__((address_space(3))) volatile u64 lds_u64;
    unsigned dec = 0;
    if (SH) {
      u64 v;
      while ((unsigned)((v = rfl64(*(lds_u64*)(box + 4))) >> 32) != call) if (!spin_ok<4>(t_in, call, myws)) { v = 0; break; }
      dec = (unsigned)v;
      const unsigned nr = bands_of(dec, rank), np = bands_of(dec, p);
#pragma unroll 1
      for (unsigned n = N1; n < N1 + 16 * SH + 16 * (16 + SH) - 72; n += 64) tbl[n + lane] = entry(n + lane, nr, np);
    }
    const unsigned la = list_len(bands_of(dec, rank), bands_of(dec, p));
    u64 plan;
    while ((unsigned)((plan = rfl64(*(lds_u64*)(box + 2))) >> 49) != (call & 0x7fffu)) if (!spin_ok<4>(t_in, call, myws)) { plan = 0; break; }
    const unsigned kk = (lane % 6) < k ? lane % 6 : lane % 6 + 1, q = (rank ^ (kk + 1)) & 7u;
    const unsigned line = (lane < 6 ? p : q) * 8 + ((p ^ q) - 1);
    const int dk = (int)(plan >> (7 * k)) & 127, dq = (int)(plan >> (7 * kk)) & 127;
    const int dl = k < kk ? dk - dq : dq - dk;
    int sl = (168 + 14 * 32 - 2 * dl + 2 * (int)((k + kk) % 7) + 1) / 14 - 32;
    sl = sl < 0 ? 0 : sl > 24 ? 24 : sl;
    const int sh = k < kk ? sl : 24 - sl;
    const unsigned len = lane < 6 ? (sh < 12 ? sh : 12) : lane < 12 ? (sh > 12 ? sh - 12 : 0) : 0;
#pragma unroll 1
    for (unsigned e0 = 0; la + e0 < TBL; e0 += 64) {
      const unsigned e = e0 + lane;
      unsigned l = 0, sub = e;
#pragma unroll
      for (int i = 0; i < 12; ++i) { const unsigned x = rdl(len, i); if (l == (unsigned)i && sub >= x) { sub -= x; ++l; } }
      const unsigned ln = pick(line, l), s = (l < 6 ? sub : 11 - sub) & 15u, ti = T0 + (s >> 1) * 7 + (ln & 7);
      const u32x4 te = tile_entry(ln >> 3, ti, s & 1, l < 6 ? OFF_FO + (p * 128 + fo_of(ti)) * 128 : OFF_FG + ln * 128, 0);
      tbl[la + e < TBL ? la + e : TBL - 1] = l < 12 ? te : u32x4{0, OFF_TMO, 0, 3u};
    }
    return;
  }

  auto desc = [](const void* b, unsigned bytes) { const u64 a = (u64)b; return u32x4{rfl((unsigned)a), rfl((unsigned)(a >> 32) & 0xffffu), rfl(bytes), 0x00020000u}; };
  const unsigned vcall = call, vone = 1, vlc = OFF_LC + k * 128;
  const unsigned lo_lin = lane * 16, lo_tile = (lane >> 3) * 4096 + (lane & 7) * 16;
  const unsigned tm_l = OFF_TM2 + mv * 128 + ((lane & 7) >= 2 && (lane & 7) < 6 ? (lane & 7) * 4 : 0);
  u32x4 w0 = {}, w1 = {}, w2 = {}, w3 = {};
  unsigned o0, o1, o2, o3, r0 = 0, r1 = 0, r2 = 0, r3 = 0;
  bool ok = true;
  unsigned kind, noff;
  unsigned nfo;
  auto fetch = [&](unsigned idx) {
    u32x4 e = tbl[idx < TBL ? idx : TBL - 1];
    while (rfl(e[3]) == 0) {
      const unsigned x = ld_sys(myws + tm_l);
      if (lane < 8) box[lane] = x;
      if (!spin_ok<2>(t_in, call, myws)) { e = u32x4{0, OFF_TMO, 0, 3u}; ok = false; break; }
      e = tbl[idx < TBL ? idx : TBL - 1];
    }
    kind = rfl(e[3]); noff = rfl(e[2]);
    const unsigned ts = kind == 2 ? 8 * 4096 : 1024;
    o0 = e[0] + (kind == 2 ? lo_tile : lo_lin); o1 = o0 + ts; o2 = o1 + ts; o3 = o2 + ts;
    nfo = (lane & 7) ? tm_l : e[1];
  };
  auto put = [&](unsigned kd) {
    const u32x4 rd = desc(kd == 2 ? myout : myws + OFF_DATA, kd == 2 ? OUT_BYTES : kd ? NR * SLICE : 0u);
    asm volatile("s_nop 4\n\tbuffer_store_dwordx4 %0, %4, %8, 0 offen sc0 sc1\n\tbuffer_store_dwordx4 %1, %5, %8, 0 offen sc0 sc1\n\t"
                 "buffer_store_dwordx4 %2, %6, %8, 0 offen sc0 sc1\n\tbuffer_store_dwordx4 %3, %7, %8, 0 offen sc0 sc1\n\ts_nop 1"
                 :: "v"(w0), "v"(w1), "v"(w2), "v"(w3), "v"(r0), "v"(r1), "v"(r2), "v"(r3), "s"(rd) : "memory");
  };
  auto notify = [&](unsigned off) {
    asm volatile("s_mov_b64 exec, 1\n\tglobal_store_dword %0, %1, %2 sc0 sc1\n\ts_mov_b64 exec, -1" :: "v"(off), "v"(vcall), "s"(myws) : "memory");
  };

  fetch(j0);
  {
    u64 bad = 1;
    if (RPOLL) {
      const unsigned ro = OFF_SW + rank * 128 * 128 + ((rfl(nfo) - OFF_FS) & (128 * 128 - 1)) + (j0 & 3) * 8 + (lane & 1) * 4;
      unsigned left = 64;
      asm volatile(
          "global_load_dword %1, %5, %6 sc0 sc1\n\ts_sleep " STR(PSL) "\n\t"
          "global_load_dword %2, %5, %6 sc0 sc1\n\ts_sleep " STR(PSL) "\n"
          "0:\n\t"
          "global_load_dword %3, %5, %6 sc0 sc1\n\t"
          "s_waitcnt vmcnt(2)\n\tv_cmp_gt_u32 %0, %7, %1\n\ts_cmp_eq_u64 %0, 0\n\ts_cbranch_scc1 1f\n\tglobal_load_dword %1, %5, %6 sc0 sc1\n\t"
          "s_waitcnt vmcnt(2)\n\tv_cmp_gt_u32 %0, %7, %2\n\ts_cmp_eq_u64 %0, 0\n\ts_cbranch_scc1 1f\n\tglobal_load_dword %2, %5, %6 sc0 sc1\n\t"
          "s_waitcnt vmcnt(2)\n\tv_cmp_gt_u32 %0, %7, %3\n\ts_cmp_eq_u64 %0, 0\n\ts_cbranch_scc1 1f\n\t"
          "s_sub_u32 %4, %4, 1\n\ts_cmp_lg_u32 %4, 0\n\ts_cbranch_scc1 0b\n"
          "1:"
          : "=&s"(bad), "=&v"(r1), "=&v"(r2), "=&v"(r3), "+s"(left) : "v"(ro), "s"(pws), "s"(call) : "memory", "scc");
    }
    if (bad) { ok = wait_flags<true>(myws + nfo, call, myws, 1); WAIT_VM(0); }
  }
  unsigned skind = 0, snoff = 0, pnoff = 0;
  for (;;) {
    u32x4 v0, v1, v2, v3;
    unsigned f, tk;
    asm volatile("s_mov_b64 exec, 1\n\tglobal_atomic_add %0, %1, %2, %3 sc0\n\ts_mov_b64 exec, -1" : "=&v"(tk) : "v"(vlc), "v"(vone), "s"(myws) : "memory");
    const bool first = snoff != 0;
    if (first) { put(skind); pnoff = snoff; skind = 0; snoff = 0; }
    {
      const u32x4 rs = desc(kind == 2 ? pout : pws + OFF_DATA, ok ? (kind == 2 ? OUT_BYTES : NR * SLICE) : 0u);
      asm volatile("s_nop 4\n\tbuffer_load_dwordx4 %0, %4, %8, 0 offen sc0 sc1\n\tbuffer_load_dwordx4 %1, %5, %8, 0 offen sc0 sc1\n\t"
                   "buffer_load_dwordx4 %2, %6, %8, 0 offen sc0 sc1\n\tbuffer_load_dwordx4 %3, %7, %8, 0 offen sc0 sc1"
                   : "=&v"(v0), "=&v"(v1), "=&v"(v2), "=&v"(v3) : "v"(o0), "v"(o1), "v"(o2), "v"(o3), "s"(rs) : "memory");
    }
    if (!first) put(skind);
    if (lane < 8) box[lane] = fprev;
    const unsigned ckind = kind, cnoff = noff, q0 = o0, q1 = o1, q2 = o2, q3 = o3;
    asm volatile("s_waitcnt vmcnt(8)" : "+v"(tk) : "v"(r1), "v"(r2), "v"(r3) : "memory");
    fetch(rfl(tk) + NW);
    asm volatile("global_load_dword %0, %1, %2 sc0 sc1" : "=&v"(f) : "v"(nfo), "s"(myws) : "memory");
    asm volatile("" : "+v"(o0), "+v"(o1), "+v"(o2), "+v"(o3));
    if (pnoff) { WAIT_VM(5); notify(pnoff); pnoff = 0; WAIT_VM(1); } else WAIT_VM(0);
    asm volatile("" : "+v"(v0), "+v"(v1), "+v"(v2), "+v"(v3), "+v"(f) :: "memory");
    if (rdl(f, 1) == call) ok = false;
    w0 = v0; w1 = v1; w2 = v2; w3 = v3;
    if (!ok) w0 = w1 = w2 = w3 = u32x4{NANPK, NANPK, NANPK, NANPK};
    r0 = q0; r1 = q1; r2 = q2; r3 = q3; skind = ckind; snoff = cnoff;
    fprev = f;
    if (kind == 3 || rfl(f) < call) {
      if (kind == 3 || rfl(ld_sys(myws + nfo)) < call) {
        put(skind);
        if (snoff) { WAIT_VM(0); notify(snoff); }
        skind = 0; snoff = 0;
        if (kind == 3) break;
        if (ok) ok = wait_flags<true>(myws + nfo, call, myws, 1);
      }
    }
  }
  if (!ok && lane == 0) st_sys(myout + gband(0, rank) * 64 * 4096 + (rank & 3) * 1024, NANPK);
}

}

extern "C" void kernel_launch(int rank, int nranks, ncclComm_t comm, hipStream_t stream,
                              void* const* d_in, const int* in_sizes, int n_in,
                              void* d_out, int out_size, const int* in_dtypes, int out_dtype,
                              void* const* d_out_all, void* const* d_ws_all, size_t ws_bytes) {
  if (nranks != NR || (unsigned)rank >= (unsigned)NR || n_in < 2 || ws_bytes < WS_NEED || out_size != 2048 * 2048 ||
      in_sizes[0] != 2048 * 256 || in_sizes[1] != 256 * 2048) { fprintf(stderr, "unexpected problem shape\n"); return; }
  Params P;
  P.x = (const char*)d_in[0]; P.w = (const char*)d_in[1];
  for (int p = 0; p < NR; ++p) P.ws[p] = (char*)d_ws_all[p];
  for (int k = 0; k < 8; ++k) { const int p = (rank ^ (k + 1)) & 7; P.lws[k] = (char*)d_ws_all[p]; P.lout[k] = (char*)d_out_all[p]; }
  P.myws = P.ws[rank]; P.myout = (char*)d_out_all[rank];
  P.rank = rank; P.pad = 0;
  const unsigned char bo[32] = {32, 0, 33, 1, 2, 34, 3, 35, 4, 36, 5, 37, 6, 38, 7, 39, 8, 9, 10, 11, 12, 13, 14, 15, 16, 17, 18, 19, 20, 21, 22, 23};
  unsigned char ord[64]; int no = 0, na = 0, nb = 0;
  for (int n = 0; n < 16; ++n) {
    if (bo[n] < 32) { P.wfirst[na] = (1u << nb) - 1; for (int b = 0; b < nb; ++b) ord[no++] = na * 8 + b; ++na; }
    else { for (int a = 0; a < na; ++a) ord[no++] = a * 8 + nb; ++nb; }
  }
  for (int i = 0; i < 16; ++i) P.inv[i] = 0;
  for (int i = 0; i < 64; ++i) { P.inv[ord[i] >> 2] |= (unsigned)i << (8 * (ord[i] & 3)); P.tab[i] = bo[i & 31] | (unsigned)ord[i] << 8; }
  int us[128], ps[128], nu = 0;
  for (int pos = 23; pos >= 0; --pos)
    for (int u = 0; u < 128; ++u)
      if ((u < 64 ? (u < 32 ? u >> 1 : 31 - (u >> 1)) : 32 + (((u - 64) >> 1) & 7)) == bo[pos]) { us[nu] = u; ps[nu++] = pos; }
  int i0 = 0, i1 = 16, np = 0;
  for (int bb = 0; bb < 128; ++bb) {
    const int b = (bb + PB0) & 127;
    const int i = (b & 7) == 0 ? i0++ : i1++;
    P.task[b] = us[i] | (2 * ps[i] >= NPUSH && 2 * ps[i] < NPUSH + NCRIT ? 0u : 1u << 20);
    if ((b & 7) != 0 && np < 7 * NPUSH) {
      const unsigned dst = rank ^ ((np + np / 7) % 7 + 1), j = np / 7;
      P.task[b] = us[i] | unit_of(bo[j >> 1], j & 1, dst) << 8 | dst << 16 | 1u << 19; ++np;
    }
  }
  hipLaunchKernelGGL(fused_gemm_ar, dim3(NGEMM + NMOVE + 1), dim3(512), 0, stream, P);
}
```

```cpp
#include <hip/hip_runtime.h>
#include <rccl/rccl.h>
#include <cstdio>
#include <cstdint>

namespace {

typedef unsigned u32x4 __attribute__((ext_vector_type(4)));
typedef float f32x4 __attribute__((ext_vector_type(4)));
typedef float f32x16 __attribute__((ext_vector_type(16)));
typedef __bf16 bf16x8 __attribute__((ext_vector_type(8)));
typedef __bf16 bf16x2 __attribute__((ext_vector_type(2)));
typedef float f32x2 __attribute__((ext_vector_type(2)));
typedef unsigned long long u64;

constexpr unsigned NW = 16;
constexpr int SH = 2;
constexpr int SHQ = 640;
constexpr unsigned NPUSH = 4;
constexpr unsigned NCRIT = 8;
constexpr unsigned T0 = 60;
constexpr unsigned OWNPOS = 120;
constexpr unsigned TSNAP = 48;
constexpr unsigned C0 = 2 * NW + NW / 2;
constexpr int NBW = 4;

constexpr int NR = 8;
constexpr unsigned UNIT = 16384;
constexpr unsigned SLICE = 128 * UNIT;
constexpr unsigned OUT_BYTES = 2048u * 2048u * 2u;
constexpr unsigned IN_BYTES = 2048u * 256u * 2u;
constexpr unsigned KB = 1024;
constexpr unsigned OFF_CNT = 0;
constexpr unsigned OFF_TMO = 16 * KB;
constexpr unsigned OFF_LC = 17 * KB;
constexpr unsigned OFF_DEC = 18 * KB + 256;
constexpr unsigned OFF_WV = 19 * KB;
constexpr unsigned OFF_FS = 32 * KB;
constexpr unsigned OFF_FL = 160 * KB;
constexpr unsigned OFF_FO = 288 * KB;
constexpr unsigned OFF_FC = 416 * KB;
constexpr unsigned OFF_FF = 424 * KB;
constexpr unsigned OFF_FG = 432 * KB;
constexpr unsigned OFF_TM2 = 448 * KB;
constexpr unsigned OFF_SW = 512 * KB;
constexpr unsigned OFF_DATA = 2048 * KB;
constexpr size_t WS_NEED = OFF_DATA + (size_t)NR * SLICE;
constexpr int NGEMM = 128, NMOVE = 7 * NW;
constexpr unsigned FO1 = 32, FOG = 4;
constexpr unsigned N1 = (2 * (24 - SH) - NPUSH) * 4;
constexpr unsigned TBL = 704;
constexpr u64 DEADLINE = 100000000ull;
constexpr int AUX_SYS = 17;
constexpr unsigned NANPK = 0x7fc07fc0u;
static_assert(NGEMM + NMOVE + 1 <= 256 && NMOVE <= 128 && N1 + 16 * SH + 16 * (16 + SH) - 72 + 144 < TBL, "layout");


struct Params {
  const char* x; const char* w;
  char* myws; char* myout;
  unsigned rank, pad;
  char* ws[NR];
  char* lws[8]; char* lout[8];
  unsigned task[128];
  unsigned wfirst[8];
  unsigned inv[16];
  unsigned tab[64];
};

__device__ __forceinline__ unsigned ld_sys(const void* p) { return __hip_atomic_load((const unsigned*)p, __ATOMIC_RELAXED, __HIP_MEMORY_SCOPE_SYSTEM); }
__device__ __forceinline__ unsigned ld_dev(const void* p) { return __hip_atomic_load((const unsigned*)p, __ATOMIC_RELAXED, __HIP_MEMORY_SCOPE_AGENT); }
__device__ __forceinline__ u64 ld_dev64(const void* p) { return __hip_atomic_load((const u64*)p, __ATOMIC_RELAXED, __HIP_MEMORY_SCOPE_AGENT); }
__device__ __forceinline__ void st_sys(void* p, unsigned v) { __hip_atomic_store((unsigned*)p, v, __ATOMIC_RELAXED, __HIP_MEMORY_SCOPE_SYSTEM); }
__device__ __forceinline__ void st_dev(void* p, unsigned v) { __hip_atomic_store((unsigned*)p, v, __ATOMIC_RELAXED, __HIP_MEMORY_SCOPE_AGENT); }
__device__ __forceinline__ unsigned rfl(unsigned v) { return __builtin_amdgcn_readfirstlane(v); }
__device__ __forceinline__ u64 rfl64(u64 v) { return (u64)rfl((unsigned)(v >> 32)) << 32 | rfl((unsigned)v); }
__device__ __forceinline__ unsigned rdl(unsigned v, unsigned l) { return __builtin_amdgcn_readlane(v, l); }
__device__ __forceinline__ unsigned pick(unsigned v, unsigned l) { return __builtin_amdgcn_ds_bpermute(l * 4, v); }
#define WAIT_VM(n) asm volatile("s_waitcnt vmcnt(" #n ")" ::: "memory")

template <int NAP>
__device__ __forceinline__ bool spin_ok(u64 t0, unsigned call, char* myws) {
  const u64 dt = __builtin_amdgcn_s_memrealtime() - t0;
  if (dt > 10000ull) {
    if (rfl(ld_dev(myws + OFF_TMO)) == call || dt > DEADLINE) {
      char* const c = myws + OFF_TM2 + (threadIdx.x & 63) * 128;
      st_dev(c, call); st_dev(c + 64 * 128, call); st_dev(myws + OFF_TMO, call);
      return false;
    }
    __builtin_amdgcn_s_sleep(32);
  }
  __builtin_amdgcn_s_sleep(NAP);
  return true;
}
__device__ __forceinline__ bool wait_flags(const void* p, unsigned call, char* myws, u64 mask) {
  const u64 t0 = __builtin_amdgcn_s_memrealtime();
  while ((__ballot(ld_sys(p) >= call) & mask) != mask)
    if (!spin_ok<1>(t0, call, myws)) return false;
  __builtin_amdgcn_fence(__ATOMIC_ACQUIRE, "wavefront");
  return true;
}
__device__ __forceinline__ unsigned wait_word(unsigned off, unsigned call, char* myws) {
  const u64 t0 = __builtin_amdgcn_s_memrealtime();
  for (;;) {
    const u64 v = rfl64(ld_dev64(myws + off));
    if ((unsigned)(v >> 32) == call) return (unsigned)v;
    if (!spin_ok<8>(t0, call, myws)) return 0;
  }
}

__device__ __forceinline__ __amdgpu_buffer_rsrc_t rsrc(const void* base, unsigned bytes) {
  return __builtin_amdgcn_make_buffer_rsrc((void*)base, 0, bytes, 0x00020000);
}

template <class F>
__device__ __forceinline__ void stage_piece(const Params& P, unsigned u, unsigned e, unsigned lane, char* base, unsigned ub, u32x4* sc, F&& after_loads) {
  const unsigned h = u & 1;
  const bool isx = u < 64;
  const unsigned o = isx ? (64 * (u >> 1) + 32 * (e & 1) + (lane >> 2)) * 512 + h * 256 + (e >> 1) * 64 + (lane & 3) * 16
                         : (128 * h + 16 * e + (lane >> 3)) * 4096 + ((u - 64) >> 1) * 128 + (lane & 7) * 16;
  const auto rs = rsrc(isx ? P.x : P.w, IN_BYTES);
  u32x4 a = __builtin_amdgcn_raw_buffer_load_b128(rs, o, 0, 0);
  u32x4 b = __builtin_amdgcn_raw_buffer_load_b128(rs, o + (isx ? 16 * 512 : 8 * 4096), 0, 0);
  __builtin_amdgcn_sched_barrier(0);
  after_loads();
  __builtin_amdgcn_sched_barrier(0);
  unsigned oa, ob;
  if (isx) {
    const unsigned wi = (lane & 3) * 33 + (lane >> 2);
    sc[wi] = a; sc[wi + 16] = b;
    const unsigned ri = (lane >> 5) * 33 + (lane & 31);
    a = sc[ri]; b = sc[ri + 66];
    oa = ub + ((e & 6) * 2 + (e & 1)) * 1024 + lane * 16; ob = oa + 2048;
  } else {
    sc[lane] = a; sc[lane + 64] = b;
    const unsigned* const sd = (const unsigned*)sc + (lane >> 5) * 256 + (lane & 31);
    unsigned d[8];
#pragma unroll
    for (int j = 0; j < 8; ++j) d[j] = sd[j * 32];
#pragma unroll
    for (int j = 0; j < 4; ++j) {
      a[j] = __builtin_amdgcn_perm(d[2 * j + 1], d[2 * j], 0x05040100u);
      b[j] = __builtin_amdgcn_perm(d[2 * j + 1], d[2 * j], 0x07060302u);
    }
    oa = ub + e * 2048 + lane * 16; ob = oa + 1024;
  }
  const auto rd = rsrc(base + OFF_DATA, NR * SLICE);
  __builtin_amdgcn_raw_buffer_store_b128(a, rd, oa, 0, AUX_SYS);
  __builtin_amdgcn_raw_buffer_store_b128(b, rd, ob, 0, AUX_SYS);
}
__host__ __device__ __forceinline__ unsigned gband(unsigned a, unsigned r) { return (r & 4) ? 31 - a : a; }
__host__ __device__ __forceinline__ unsigned unit_of(unsigned code, unsigned h, unsigned r) {
  return code < 32 ? gband(code, r) * 2 + h : 64 + (8 * (r & 3) + (code - 32)) * 2 + h;
}
__device__ __forceinline__ unsigned fo_of(unsigned ti) { return ti < FO1 ? ti : FO1 + (ti - FO1) / FOG; }
__device__ __forceinline__ unsigned bands_of(unsigned dec, unsigned r) {
  const int d = (int)(((dec >> (4 * (r & 3))) & 15u) ^ 8u) - 8;
  return (r & 4) ? 16 + d : 16 - d;
}

__device__ __forceinline__ unsigned lag_bit(unsigned k) { return 7 * k + (k >= 4 ? 4 : 0); }

template <bool B> struct BoolC { static constexpr bool value = B; };

__global__ void __launch_bounds__(512, 2) fused_gemm_ar(const Params P) {
  __shared__ f32x4 lds[8192];
  const int lane = threadIdx.x & 63;
  const int wave = __builtin_amdgcn_readfirstlane(threadIdx.x >> 6);
  const int bid = blockIdx.x;
  const unsigned rank = P.rank & 7u;
  char* const myws = P.myws;
  char* const myout = P.myout;
  const unsigned task = P.task[bid & 127];
  const unsigned mv = (bid - NGEMM) & 127, k = mv % 7, j0 = mv / 7;
  asm volatile("" :: "s"(P.x), "s"(P.w), "s"(myws), "s"(myout), "s"(rank), "s"(task));
  unsigned* const cntp = (unsigned*)(myws + OFF_CNT) + bid * 8 + wave;

  if (bid < NGEMM) {
    unsigned cnt_raw;
    char* peer_ws;
    {
      u32x4* const scr = (u32x4*)lds + wave * 1024;
      if ((task >> 19) & 1) {
        const unsigned u = (task >> 8) & 127u;
        char* const dws = P.ws[(task >> 16) & 7u];
        stage_piece(P, u, wave, lane, dws, rank * SLICE + u * UNIT, scr, [&]() { cnt_raw = ld_dev(cntp); });
        WAIT_VM(0);
        __builtin_amdgcn_s_barrier();
        if (wave == 0 && lane < 4) st_sys(dws + OFF_FL + (rank * 128 + u) * 128 + lane * 4, rfl(cnt_raw) + 1);
      } else if ((task >> 20) & 1) __builtin_amdgcn_s_sleep(16);
      const unsigned u = task & 127u;
      stage_piece(P, u, wave, lane, myws, rank * SLICE + u * UNIT, scr, [&]() { cnt_raw = ld_dev(cntp); peer_ws = P.ws[lane & 7]; });
      const bool need = lane < 8 ? ((u < 64 ? gband(u >> 1, lane) < 16 + SH : ((unsigned)lane & 3) == ((u - 64) >> 4)) && (unsigned)lane != rank) : lane < 12;
      char* const fp = lane < 8 ? peer_ws + OFF_FS + (rank * 128 + u) * 128 : myws + OFF_FL + (rank * 128 + u) * 128 + (lane & 3) * 4;
      WAIT_VM(0);
      if (!((task >> 19) & 3) && lane < 8 && need)
        st_sys(myws + OFF_SW + (lane * 128 + u) * 128 + wave * 4, rfl(cnt_raw) + 1);
      __builtin_amdgcn_s_barrier();
      if (wave == 0 && need) st_sys(fp, rfl(cnt_raw) + 1);
    }
    const unsigned call = rfl(cnt_raw) + 1;
    if (lane == 0) st_dev(cntp, call);

    unsigned ta_ = ((bid & 7) >> 1) + 4 * (bid >> 5), tb_ = (bid & 1) + 2 * ((bid >> 3) & 3);
#pragma unroll 1
    for (;;) {
    const unsigned ti_ = ta_ < 8 ? (P.inv[(ta_ * 2 + (tb_ >> 2)) & 15] >> (8 * (tb_ & 3))) & 127u : ta_ * 8 + tb_;
    const bool wfirst = ta_ >= 8 || ((P.wfirst[ta_ & 7] >> tb_) & 1);
    const unsigned ue = wfirst ? unit_of(32 + tb_, 0, rank) : unit_of(ta_, 0, rank);
    const unsigned ul = wfirst ? unit_of(ta_, 0, rank) : unit_of(32 + tb_, 0, rank);
    if (SH && ta_ >= 16 - SH && ta_ >= bands_of(wait_word(OFF_DEC, call, myws), rank)) return;
    const auto rdat = rsrc(myws + OFF_DATA, NR * SLICE);
    const unsigned rot = (gband(ta_, rank) >> 2) + (tb_ >> 1);
    const unsigned dbase = wave * SLICE + lane * 16;
    auto ld = [&](unsigned u, int f) {
      return __builtin_amdgcn_raw_buffer_load_b128(rdat, dbase + (f & 3) * 1024, u * UNIT + ((((f >> 2) + rot) & 3) << 12), AUX_SYS);
    };
    f32x16 acc[2][2];
    u32x4* const stash = (u32x4*)lds + wave * 1024 + lane;
    bool ok = wait_flags(myws + OFF_FL + (wave * 128 + ue + ((lane >> 2) & 1)) * 128 + (lane & 3) * 4, call, myws, 0xff);
    auto body = [&](auto WF) {
      constexpr bool W1 = decltype(WF)::value;
      u32x4 r1[16], r2[16];
#pragma unroll
      for (int f = 0; f < 16; ++f) r2[f] = ld(ue + 1, f);
#pragma unroll
      for (int f = 0; f < 16; ++f) r1[f] = ld(ue, f);
      __builtin_amdgcn_sched_barrier(0);
#pragma unroll
      for (int f = 0; f < 16; ++f) stash[f * 64] = r2[f];
      if (ok) ok = wait_flags(myws + OFF_FL + (wave * 128 + ul + ((lane >> 2) & 1)) * 128 + (lane & 3) * 4, call, myws, 0xff);
#pragma unroll
      for (int f = 0; f < 16; ++f) r2[f] = ld(ul, f);
      __builtin_amdgcn_sched_barrier(0);
#pragma unroll
      for (int h = 0; h < 2; ++h)
#pragma unroll
        for (int ks = 0; ks < 8; ++ks) {
#pragma unroll
          for (int i = 0; i < 2; ++i)
#pragma unroll
            for (int j = 0; j < 2; ++j) {
              const bf16x8 wv = __builtin_bit_cast(bf16x8, W1 ? r1[ks * 2 + j] : r2[ks * 2 + j]);
              const bf16x8 xv = __builtin_bit_cast(bf16x8, W1 ? r2[ks * 2 + i] : r1[ks * 2 + i]);
              acc[i][j] = __builtin_amdgcn_mfma_f32_32x32x16_bf16(wv, xv, (h | ks) ? acc[i][j] : f32x16{}, 0, 0, 0);
            }
          if (h == 0) {
#pragma unroll
            for (int t = 0; t < 2; ++t) { r2[ks * 2 + t] = ld(ul + 1, ks * 2 + t); r1[ks * 2 + t] = stash[(ks * 2 + t) * 64]; }
            __builtin_amdgcn_sched_barrier(0);
          }
        }
    };
    if (wfirst) body(BoolC<true>{}); else { asm volatile("; row band first" ::: "memory"); body(BoolC<false>{}); }
    const unsigned tm = ld_dev(myws + OFF_TMO);
    if (!ok) {
#pragma unroll
      for (int i = 0; i < 2; ++i)
#pragma unroll
        for (int j = 0; j < 2; ++j)
#pragma unroll
          for (int r = 0; r < 16; ++r) acc[i][j][r] = __builtin_nanf("");
    }
    {
      const unsigned m = lane & 31, hh = lane >> 5;
#pragma unroll
      for (int i = 0; i < 2; ++i)
#pragma unroll
        for (int g = 0; g < 4; ++g)
#pragma unroll
          for (int j = 0; j < 2; ++j)
            lds[wave * 1024 + ((i * 4 + g) * 2 + j) * 64 + hh * 32 + ((m + 4 * (2 * (g & 1) + hh)) & 31)] =
                f32x4{acc[i][j][4 * g], acc[i][j][4 * g + 1], acc[i][j][4 * g + 2], acc[i][j][4 * g + 3]};
    }
    __syncthreads();
    {
      const unsigned row = 8 * wave + (lane >> 3), c = lane & 7, g = c >> 1, hh = c & 1;
      const unsigned slot = (((row >> 5) * 4 + g) * 2) * 64 + hh * 32 + (((row & 31) + 4 * (c & 3)) & 31);
      f32x4 r0[8], r1[8];
#pragma unroll
      for (int w = 0; w < 8; ++w) { r0[w] = lds[w * 1024 + slot]; r1[w] = lds[w * 1024 + slot + 64]; }
      __builtin_amdgcn_sched_barrier(0);
      f32x4 e0 = r0[0], e1 = r1[0];
#pragma unroll
      for (int w = 1; w < 8; ++w) { e0 += r0[w]; e1 += r1[w]; }
      u32x4 o;
#pragma unroll
      for (int i = 0; i < 4; ++i) o[i] = __builtin_bit_cast(unsigned, __builtin_convertvector(f32x2{e0[i], e1[i]}, bf16x2));
      if (tm == call) o = u32x4{NANPK, NANPK, NANPK, NANPK};
      __builtin_amdgcn_raw_buffer_store_b128(o, rsrc(myout, OUT_BYTES), (64 * gband(ta_, rank) + row) * 4096 + (512 * (rank & 3) + 64 * tb_) * 2 + c * 16, 0, AUX_SYS);
    }
    WAIT_VM(0);
    __builtin_amdgcn_s_barrier();
    if (wave == 0) {
      bool tell = true;
      if (ti_ >= FO1) {
        unsigned old = 0, off = OFF_FC + ((ti_ - FO1) / FOG) * 128;
        asm volatile("" : "+v"(off));
        if (lane == 0) old = __hip_atomic_fetch_add((unsigned*)(myws + off), 1u, __ATOMIC_RELAXED, __HIP_MEMORY_SCOPE_AGENT);
        tell = rfl(old) % FOG == FOG - 1;
      }
      if (tell && lane < 8 && (unsigned)lane != rank) st_sys(peer_ws + OFF_FO + (rank * 128 + fo_of(ti_)) * 128, call);
    }
    if (ti_ >= 8 * SH) return;
    ta_ = 16 + (ti_ >> 3); tb_ = ti_ & 7;
    }
  }

  const u64 t_in = __builtin_amdgcn_s_memrealtime();
  auto list_len = [&](unsigned nr, unsigned np) { return N1 + 8 * (nr - (16 - SH)) + 16 * np - 72; };
  if (bid == NGEMM + NMOVE) {
    if (wave) return;
    char* const peer_ws = P.ws[lane & 7];
    const unsigned call = rfl(ld_dev(cntp)) + 1;
    if (lane == 0) st_dev(cntp, call);
    const unsigned kl = lane < 7 ? lane : 0;
    auto drawn = [&]() { const u64 w = ld_dev64(myws + OFF_LC + kl * 128); return (unsigned)(w >> 32) == call ? (unsigned)w + NW : 0u; };
    auto post = [&](unsigned off, u64 v) {
      __hip_atomic_store((u64*)(myws + OFF_TM2 + lane * 128 + off), v, __ATOMIC_RELAXED, __HIP_MEMORY_SCOPE_AGENT);
      __hip_atomic_store((u64*)(myws + OFF_TM2 + (64 + lane) * 128 + off), v, __ATOMIC_RELAXED, __HIP_MEMORY_SCOPE_AGENT);
    };
    bool alive = true;
    unsigned dec = 0;
    if (SH) {
      unsigned ts = 0;
      for (;;) {
        const bool go = drawn() >= C0;
        const u64 dt = __builtin_amdgcn_s_memrealtime() - t_in;
        if (go && !ts) ts = dt > 100000 ? 100000u : (unsigned)dt + 1;
        if (__all(ts != 0) || !(alive = spin_ok<2>(t_in, call, myws))) break;
      }
      unsigned sum = 0;
#pragma unroll
      for (int i = 0; i < 7; ++i) sum += rdl(ts, i);
      if (lane < 8) __hip_atomic_store((u64*)(peer_ws + OFF_WV + rank * 128), (u64)call << 32 | sum, __ATOMIC_RELAXED, __HIP_MEMORY_SCOPE_SYSTEM);
      u64 v;
      for (;;) {
        v = __hip_atomic_load((const u64*)(myws + OFF_WV + (lane & 7) * 128), __ATOMIC_RELAXED, __HIP_MEMORY_SCOPE_SYSTEM);
        if (__all((unsigned)(v >> 32) == call) || !(alive = alive && spin_ok<2>(t_in, call, myws))) break;
      }
#pragma unroll
      for (int i = 0; i < 4; ++i) {
        const int df = (int)rdl((unsigned)v, i) - (int)rdl((unsigned)v, i + 4);
        int d = (df + (df < 0 ? -SHQ / 2 : SHQ / 2)) / SHQ;
        d = d < -SH ? -SH : d > SH ? SH : d;
        dec |= ((unsigned)d & 15u) << (4 * i);
      }
      if (!alive) dec = 0;
      post(16, call << 16 | dec);
      if (lane == 0) __hip_atomic_store((u64*)(myws + OFF_DEC), (u64)call << 32 | dec, __ATOMIC_RELAXED, __HIP_MEMORY_SCOPE_AGENT);
    }
    const unsigned la_l = list_len(bands_of(dec, rank), bands_of(dec, rank ^ (kl + 1)));
    const auto rff = rsrc(myws + OFF_FF, 7 * 128);
    unsigned todo = 0xffu;
    while (todo) {
      const unsigned left = la_l + 512 - drawn();
      const u32x4 ff = __builtin_amdgcn_raw_buffer_load_b128(rff, (lane >> 2) * 128 + ((lane & 3) == 3 ? 0 : lane & 3) * 16, 0, AUX_SYS);
      if ((todo & 128) && (__any(left <= 512 + TSNAP) || !alive)) {
        unsigned mn = 0xffffu;
#pragma unroll
        for (int i = 0; i < 7; ++i) { const unsigned x = rdl(left, i); mn = x < mn ? x : mn; }
        u64 val = (u64)(call & 15u) << 60 | (u64)(call & 15u) << 28;
#pragma unroll
        for (int i = 0; i < 7; ++i) { const unsigned d = rdl(left, i) - mn; val |= (u64)(d > 127 ? 127u : d) << lag_bit(i); }
        post(8, val);
        todo &= 127u;
      }
      const unsigned have = (unsigned)__ballot(ff[0] >= call && ff[1] >= call && ff[2] >= call && ff[3] >= call);
#pragma unroll
      for (int i = 0; i < 7; ++i)
        if (((todo >> i) & 1) && ((have >> (4 * i)) & 15u) == 15u) {
          if (lane < 8 && (unsigned)lane != rank) st_sys(peer_ws + OFF_FG + ((rank ^ (i + 1)) * 8 + i) * 128, call);
          todo &= ~(1u << i);
        }
      if (!alive) break;
      if (todo) alive = spin_ok<2>(t_in, call, myws);
    }
    return;
  }

  if (wave >= NBW) return;
  const char* const pws = P.lws[k];
  const char* const pout = P.lout[k];
  const unsigned p = (rank ^ (k + 1)) & 7u;
  const unsigned tab = P.tab[lane];
  unsigned cnt_raw = 0;
  if (wave < 2) cnt_raw = ld_dev(cntp);
  typedef __attribute__((address_space(3))) volatile u32x4 lds_u32x4;
  lds_u32x4* const tbl = (lds_u32x4*)lds;
  auto tile_entry = [&](unsigned o, unsigned ti, unsigned half, unsigned fa, unsigned noff) {
    const unsigned t0 = (pick(tab, ti & 63) >> 8) & 63u, t = ti < 64 ? t0 : ti;
    return u32x4{(64 * gband(t >> 3, o) + 32 * half) * 4096 + (512 * (o & 3) + 64 * (t & 7)) * 2, fa, noff, 2u};
  };
  auto entry = [&](unsigned n, unsigned nr, unsigned np) {
    const unsigned nin = N1 + 8 * (nr - (16 - SH));
    const unsigned ui = NPUSH + (n >> 2), u = unit_of(pick(tab, (ui >> 1) & 63) & 63u, ui & 1, rank) & 127u;
    const unsigned y = n - nin, s = y - OWNPOS, z = y < OWNPOS ? y : y - 12;
    const unsigned ti = s < 12 ? T0 + (s >> 1) * 7 + k : (z < 2 * T0 ? z : z + 84) >> 1;
    const u32x4 te = tile_entry(p, ti, y & 1, OFF_FO + (p * 128 + fo_of(ti)) * 128, s < 12 ? OFF_FF + k * 128 + s * 4 : 0u);
    if (n < nin) return u32x4{p * SLICE + u * UNIT + (n & 3) * 4096, OFF_FS + (p * 128 + u) * 128, OFF_FL + (p * 128 + u) * 128 + (n & 3) * 4, 1u};
    return n < list_len(nr, np) ? te : u32x4{0, 0, 0, 0};
  };
  typedef __attribute__((address_space(3))) volatile unsigned lds_u32;
  lds_u32* const box = (lds_u32*)(tbl + TBL);
#pragma unroll 1
  for (unsigned n0 = wave * 64; n0 < TBL; n0 += NBW * 64) {
    u32x4 e = {0, 0, 0, 0};
    if (n0 < (SH ? N1 : N1 + 184)) e = entry(n0 + lane, 16, 16);
    tbl[n0 + lane] = (SH == 0 || n0 + lane < N1) ? e : u32x4{0, 0, 0, 0};
  }
  const unsigned call = rfl(cnt_raw) + 1;
  unsigned fprev = lane == 4 ? (call + 1) << 16 : (call + 1) << 28;
  if (threadIdx.x < 8) box[threadIdx.x] = fprev;
  if (wave < 2 && lane == 0) {
    st_dev(cntp, call);
    if (wave == 0) __hip_atomic_fetch_max((u64*)(myws + OFF_LC + k * 128), (u64)call << 32, __ATOMIC_RELAXED, __HIP_MEMORY_SCOPE_AGENT);
  }
  __syncthreads();
  if (wave > 1) return;

  if (wave == 1) {
    typedef __attribute__((address_space(3))) volatile u64 lds_u64;
    unsigned dec = 0;
    if (SH) {
      while (((dec = rfl(box[4])) ^ call << 16) >> 16) if (!spin_ok<4>(t_in, call, myws)) { dec = 0; break; }
      const unsigned nr = bands_of(dec, rank), np = bands_of(dec, p);
#pragma unroll 1
      for (unsigned n = N1; n < N1 + 16 * SH + 16 * (16 + SH) - 72; n += 64) tbl[n + lane] = entry(n + lane, nr, np);
    }
    const unsigned la = list_len(bands_of(dec, rank), bands_of(dec, p));
    u64 plan;
    while ((((plan = rfl64(*(lds_u64*)(box + 2))) >> 28) & 0xf0000000full) != (call & 15u) * 0x100000001ull)
      if (!spin_ok<4>(t_in, call, myws)) { plan = 0; break; }
    const unsigned kk = (lane % 6) < k ? lane % 6 : lane % 6 + 1, q = (rank ^ (kk + 1)) & 7u;
    const unsigned line = (lane < 6 ? p : q) * 8 + ((p ^ q) - 1);
    const int dk = (int)(plan >> lag_bit(k)) & 127, dq = (int)(plan >> lag_bit(kk)) & 127;
    const int dl = k < kk ? dk - dq : dq - dk;
    int sl = (168 + 14 * 32 - 2 * dl + 2 * (int)((k + kk) % 7) + 1) / 14 - 32;
    sl = sl < 0 ? 0 : sl > 24 ? 24 : sl;
    const int sh = k < kk ? sl : 24 - sl;
    const unsigned len = lane < 6 ? (sh < 12 ? sh : 12) : lane < 12 ? (sh > 12 ? sh - 12 : 0) : 0;
#pragma unroll 1
    for (unsigned e0 = 0; la + e0 < TBL; e0 += 64) {
      const unsigned e = e0 + lane;
      unsigned l = 0, sub = e;
#pragma unroll
      for (int i = 0; i < 12; ++i) { const unsigned x = rdl(len, i); if (l == (unsigned)i && sub >= x) { sub -= x; ++l; } }
      const unsigned ln = pick(line, l), s = (l < 6 ? sub : 11 - sub) & 15u, ti = T0 + (s >> 1) * 7 + (ln & 7);
      const u32x4 te = tile_entry(ln >> 3, ti, s & 1, l < 6 ? OFF_FO + (p * 128 + fo_of(ti)) * 128 : OFF_FG + ln * 128, 0);
      tbl[la + e < TBL ? la + e : TBL - 1] = l < 12 ? te : u32x4{0, OFF_TMO, 0, 3u};
    }
    return;
  }

  auto desc = [](const void* b, unsigned bytes) { const u64 a = (u64)b; return u32x4{rfl((unsigned)a), rfl((unsigned)(a >> 32) & 0xffffu), rfl(bytes), 0x00020000u}; };
  const unsigned vcall = call, vone = 1, vlc = OFF_LC + k * 128;
  const unsigned lo_lin = lane * 16, lo_tile = (lane >> 3) * 4096 + (lane & 7) * 16;
  const unsigned tm_l = OFF_TM2 + mv * 128 + ((lane & 7) >= 2 && (lane & 7) < 6 ? (lane & 7) * 4 : 0);
  u32x4 w0 = {}, w1 = {}, w2 = {}, w3 = {};
  unsigned o0, o1, o2, o3, r0 = 0, r1 = 0, r2 = 0, r3 = 0;
  bool ok = true;
  unsigned kind, noff;
  unsigned nfo;
  auto fetch = [&](unsigned idx) {
    u32x4 e = tbl[idx < TBL ? idx : TBL - 1];
    while (rfl(e[3]) == 0) {
      const unsigned x = ld_sys(myws + tm_l);
      if (lane < 8) box[lane] = x;
      if (!spin_ok<2>(t_in, call, myws)) { e = u32x4{0, OFF_TMO, 0, 3u}; ok = false; break; }
      e = tbl[idx < TBL ? idx : TBL - 1];
    }
    kind = rfl(e[3]); noff = rfl(e[2]);
    const unsigned ts = kind == 2 ? 8 * 4096 : 1024;
    o0 = e[0] + (kind == 2 ? lo_tile : lo_lin); o1 = o0 + ts; o2 = o1 + ts; o3 = o2 + ts;
    nfo = (lane & 7) ? tm_l : e[1];
  };
  auto put = [&](unsigned kd) {
    const u32x4 rd = desc(kd == 2 ? myout : myws + OFF_DATA, kd == 2 ? OUT_BYTES : NR * SLICE);
    asm volatile("s_nop 4\n\tbuffer_store_dwordx4 %0, %4, %8, 0 offen sc0 sc1\n\tbuffer_store_dwordx4 %1, %5, %8, 0 offen sc0 sc1\n\t"
                 "buffer_store_dwordx4 %2, %6, %8, 0 offen sc0 sc1\n\tbuffer_store_dwordx4 %3, %7, %8, 0 offen sc0 sc1\n\ts_nop 1"
                 :: "v"(w0), "v"(w1), "v"(w2), "v"(w3), "v"(r0), "v"(r1), "v"(r2), "v"(r3), "s"(rd) : "memory");
  };
  auto notify = [&](unsigned off) {
    asm volatile("s_mov_b64 exec, 1\n\tglobal_store_dword %0, %1, %2 sc0 sc1\n\ts_mov_b64 exec, -1" :: "v"(off), "v"(vcall), "s"(myws) : "memory");
  };

  fetch(j0);
  {
    u64 bad;
    {
      const unsigned ro = OFF_SW + rank * 128 * 128 + ((rfl(nfo) - OFF_FS) & (128 * 128 - 1)) + (j0 & 3) * 8 + (lane & 1) * 4;
      unsigned left = rfl(ld_sys(myws + nfo)) >= call ? 0u : 64u;
      asm volatile(
          "s_mov_b64 %0, 0\n\ts_cmp_eq_u32 %4, 0\n\ts_cbranch_scc1 1f\n\t"
          "global_load_dword %1, %5, %6 sc0 sc1\n\ts_sleep 6\n\t"
          "global_load_dword %2, %5, %6 sc0 sc1\n\ts_sleep 6\n"
          "0:\n\t"
          "global_load_dword %3, %5, %6 sc0 sc1\n\t"
          "s_waitcnt vmcnt(2)\n\tv_cmp_gt_u32 %0, %7, %1\n\ts_cmp_eq_u64 %0, 0\n\ts_cbranch_scc1 1f\n\tglobal_load_dword %1, %5, %6 sc0 sc1\n\t"
          "s_waitcnt vmcnt(2)\n\tv_cmp_gt_u32 %0, %7, %2\n\ts_cmp_eq_u64 %0, 0\n\ts_cbranch_scc1 1f\n\tglobal_load_dword %2, %5, %6 sc0 sc1\n\t"
          "s_waitcnt vmcnt(2)\n\tv_cmp_gt_u32 %0, %7, %3\n\ts_cmp_eq_u64 %0, 0\n\ts_cbranch_scc1 1f\n\t"
          "s_sub_u32 %4, %4, 1\n\ts_cmp_lg_u32 %4, 0\n\ts_cbranch_scc1 0b\n"
          "1:"
          : "=&s"(bad), "=&v"(r1), "=&v"(r2), "=&v"(r3), "+s"(left) : "v"(ro), "s"(pws), "s"(call) : "memory", "scc");
    }
    if (bad) { ok = wait_flags(myws + nfo, call, myws, 1); WAIT_VM(0); }
  }
  unsigned skind = 0, snoff = 0, pnoff = 0;
  for (;;) {
    u32x4 v0, v1, v2, v3;
    unsigned f, tk;
    asm volatile("s_mov_b64 exec, 1\n\tglobal_atomic_add %0, %1, %2, %3 sc0\n\ts_mov_b64 exec, -1" : "=&v"(tk) : "v"(vlc), "v"(vone), "s"(myws) : "memory");
    const bool first = snoff != 0;
    if (first) { put(skind); pnoff = snoff; }
    {
      const u32x4 rs = desc(kind == 2 ? pout : pws + OFF_DATA, ok ? (kind == 2 ? OUT_BYTES : NR * SLICE) : 0u);
      asm volatile("s_nop 4\n\tbuffer_load_dwordx4 %0, %4, %8, 0 offen sc0 sc1\n\tbuffer_load_dwordx4 %1, %5, %8, 0 offen sc0 sc1\n\t"
                   "buffer_load_dwordx4 %2, %6, %8, 0 offen sc0 sc1\n\tbuffer_load_dwordx4 %3, %7, %8, 0 offen sc0 sc1"
                   : "=&v"(v0), "=&v"(v1), "=&v"(v2), "=&v"(v3) : "v"(o0), "v"(o1), "v"(o2), "v"(o3), "s"(rs) : "memory");
    }
    if (!first && skind) put(skind);
    if (lane < 8) box[lane] = fprev;
    const unsigned ckind = kind, cnoff = noff, q0 = o0, q1 = o1, q2 = o2, q3 = o3;
    if (!skind) WAIT_VM(4);
    asm volatile("s_waitcnt vmcnt(8)" : "+v"(tk) : "v"(r1), "v"(r2), "v"(r3) : "memory");
    fetch(rfl(tk) + NW);
    asm volatile("global_load_dword %0, %1, %2 sc0 sc1" : "=&v"(f) : "v"(nfo), "s"(myws) : "memory");
    asm volatile("" : "+v"(o0), "+v"(o1), "+v"(o2), "+v"(o3));
    if (pnoff) { WAIT_VM(5); notify(pnoff); pnoff = 0; WAIT_VM(1); } else WAIT_VM(0);
    asm volatile("" : "+v"(v0), "+v"(v1), "+v"(v2), "+v"(v3), "+v"(f) :: "memory");
    if (rdl(f, 1) == call) ok = false;
    w0 = v0; w1 = v1; w2 = v2; w3 = v3;
    if (!ok) w0 = w1 = w2 = w3 = u32x4{NANPK, NANPK, NANPK, NANPK};
    r0 = q0; r1 = q1; r2 = q2; r3 = q3; skind = ckind; snoff = cnoff;
    fprev = f;
    if (kind == 3 || rfl(f) < call) {
      if (kind == 3 || rfl(ld_sys(myws + nfo)) < call) {
        put(skind);
        if (snoff) { WAIT_VM(0); notify(snoff); }
        skind = 0; snoff = 0;
        if (kind == 3) break;
        if (ok) ok = wait_flags(myws + nfo, call, myws, 1);
      }
    }
  }
  if (!ok && lane == 0) st_sys(myout + gband(0, rank) * 64 * 4096 + (rank & 3) * 1024, NANPK);
}

}

extern "C" void kernel_launch(int rank, int nranks, ncclComm_t comm, hipStream_t stream,
                              void* const* d_in, const int* in_sizes, int n_in,
                              void* d_out, int out_size, const int* in_dtypes, int out_dtype,
                              void* const* d_out_all, void* const* d_ws_all, size_t ws_bytes) {
  if (nranks != NR || (unsigned)rank >= (unsigned)NR || n_in < 2 || ws_bytes < WS_NEED || out_size != 2048 * 2048 ||
      in_sizes[0] != 2048 * 256 || in_sizes[1] != 256 * 2048) { fprintf(stderr, "unexpected problem shape\n"); return; }
  Params P;
  P.x = (const char*)d_in[0]; P.w = (const char*)d_in[1];
  for (int p = 0; p < NR; ++p) P.ws[p] = (char*)d_ws_all[p];
  for (int k = 0; k < 8; ++k) { const int p = (rank ^ (k + 1)) & 7; P.lws[k] = (char*)d_ws_all[p]; P.lout[k] = (char*)d_out_all[p]; }
  P.myws = P.ws[rank]; P.myout = (char*)d_out_all[rank];
  P.rank = rank; P.pad = 0;
  const unsigned char bo[32] = {32, 0, 33, 1, 2, 34, 3, 35, 4, 36, 5, 37, 6, 38, 7, 39, 8, 9, 10, 11, 12, 13, 14, 15, 16, 17, 18, 19, 20, 21, 22, 23};
  unsigned char ord[64]; int no = 0, na = 0, nb = 0;
  for (int n = 0; n < 16; ++n) {
    if (bo[n] < 32) { P.wfirst[na] = (1u << nb) - 1; for (int b = 0; b < nb; ++b) ord[no++] = na * 8 + b; ++na; }
    else { for (int a = 0; a < na; ++a) ord[no++] = a * 8 + nb; ++nb; }
  }
  for (int i = 0; i < 16; ++i) P.inv[i] = 0;
  for (int i = 0; i < 64; ++i) { P.inv[ord[i] >> 2] |= (unsigned)i << (8 * (ord[i] & 3)); P.tab[i] = bo[i & 31] | (unsigned)ord[i] << 8; }
  int us[128], ps[128], nu = 0;
  for (int pos = 23; pos >= 0; --pos)
    for (int u = 0; u < 128; ++u)
      if ((u < 64 ? (u < 32 ? u >> 1 : 31 - (u >> 1)) : 32 + (((u - 64) >> 1) & 7)) == bo[pos]) { us[nu] = u; ps[nu++] = pos; }
  int i0 = 0, i1 = 16, np = 0;
  for (int bb = 0; bb < 128; ++bb) {
    const int b = (bb + 96) & 127;
    const int i = (b & 7) == 0 ? i0++ : i1++;
    P.task[b] = us[i] | (2 * ps[i] + 1 >= NPUSH && 2 * ps[i] < NPUSH + NCRIT ? 0u : 1u << 20);
    if ((b & 7) != 0 && np < 7 * NPUSH) {
      const unsigned dst = rank ^ ((np + np / 7) % 7 + 1), j = np / 7;
      P.task[b] = us[i] | unit_of(bo[j >> 1], j & 1, dst) << 8 | dst << 16 | 1u << 19; ++np;
    }
  }
  hipLaunchKernelGGL(fused_gemm_ar, dim3(NGEMM + NMOVE + 1), dim3(512), 0, stream, P);
}
```

```cpp
#include <hip/hip_runtime.h>
#include <rccl/rccl.h>
#include <cstdio>
#include <cstdint>

namespace {

typedef unsigned u32x4 __attribute__((ext_vector_type(4)));
typedef float f32x4 __attribute__((ext_vector_type(4)));
typedef float f32x16 __attribute__((ext_vector_type(16)));
typedef __bf16 bf16x8 __attribute__((ext_vector_type(8)));
typedef __bf16 bf16x2 __attribute__((ext_vector_type(2)));
typedef float f32x2 __attribute__((ext_vector_type(2)));
typedef unsigned long long u64;

constexpr unsigned NW = 16;
constexpr unsigned NPUSH = 4;
constexpr unsigned NCRIT = 8;
constexpr unsigned C0 = 2 * NW + NW / 2;
constexpr unsigned IQ = 8;
constexpr unsigned T0A = 44, T0B = 48;
constexpr unsigned GA = 6, GB = 11, GC = 3;
constexpr int MRG = 2;
constexpr unsigned TSNAP = 48, TSNAP2 = 52;
constexpr int NBW = 4;

constexpr int NR = 8;
constexpr unsigned UNIT = 16384;
constexpr unsigned SLICE = 128 * UNIT;
constexpr unsigned OUT_BYTES = 2048u * 2048u * 2u;
constexpr unsigned IN_BYTES = 2048u * 256u * 2u;
constexpr unsigned KB = 1024;
constexpr unsigned OFF_CNT = 0;
constexpr unsigned OFF_TMO = 16 * KB;
constexpr unsigned OFF_LC = 17 * KB;
constexpr unsigned OFF_DEC = 18 * KB + 256;
constexpr unsigned OFF_WV = 19 * KB;
constexpr unsigned OFF_FS = 32 * KB;
constexpr unsigned OFF_FL = 160 * KB;
constexpr unsigned OFF_FO = 288 * KB;
constexpr unsigned OFF_FC = 416 * KB;
constexpr unsigned OFF_FF = 424 * KB;
constexpr unsigned OFF_FG = 432 * KB;
constexpr unsigned OFF_TM2 = 448 * KB;
constexpr unsigned OFF_SW = 512 * KB;
constexpr unsigned OFF_DATA = 2048 * KB;
constexpr size_t WS_NEED = OFF_DATA + (size_t)NR * SLICE;
constexpr int NGEMM = 128, NMOVE = 7 * NW;
constexpr unsigned N1 = (2 * 21 - NPUSH) * 4;
constexpr unsigned DEC0 = 2 | 2 << 2 | 2 << 4 | 2 << 6 | 1 << 8;
constexpr unsigned TBL = 704;
constexpr u64 DEADLINE = 100000000ull;
constexpr int AUX_SYS = 17;
constexpr unsigned NANPK = 0x7fc07fc0u;
static_assert(NGEMM + NMOVE + 1 <= 256 && NMOVE <= 128 && N1 + 32 + 2 * 156 + 12 * GB + 64 < TBL && T0A + 2 * GB + 5 * GA <= 104 && T0B + 7 * GC <= 90, "layout");


struct Params {
  const char* x; const char* w;
  char* myws; char* myout;
  unsigned rank, pad;
  char* ws[NR];
  char* lws[8]; char* lout[8];
  unsigned task[128];
  unsigned job[128];
  unsigned tab[64];
};

__device__ __forceinline__ unsigned ld_sys(const void* p) { return __hip_atomic_load((const unsigned*)p, __ATOMIC_RELAXED, __HIP_MEMORY_SCOPE_SYSTEM); }
__device__ __forceinline__ unsigned ld_dev(const void* p) { return __hip_atomic_load((const unsigned*)p, __ATOMIC_RELAXED, __HIP_MEMORY_SCOPE_AGENT); }
__device__ __forceinline__ u64 ld_dev64(const void* p) { return __hip_atomic_load((const u64*)p, __ATOMIC_RELAXED, __HIP_MEMORY_SCOPE_AGENT); }
__device__ __forceinline__ void st_sys(void* p, unsigned v) { __hip_atomic_store((unsigned*)p, v, __ATOMIC_RELAXED, __HIP_MEMORY_SCOPE_SYSTEM); }
__device__ __forceinline__ void st_dev(void* p, unsigned v) { __hip_atomic_store((unsigned*)p, v, __ATOMIC_RELAXED, __HIP_MEMORY_SCOPE_AGENT); }
__device__ __forceinline__ unsigned rfl(unsigned v) { return __builtin_amdgcn_readfirstlane(v); }
__device__ __forceinline__ u64 rfl64(u64 v) { return (u64)rfl((unsigned)(v >> 32)) << 32 | rfl((unsigned)v); }
__device__ __forceinline__ unsigned rdl(unsigned v, unsigned l) { return __builtin_amdgcn_readlane(v, l); }
__device__ __forceinline__ unsigned pick(unsigned v, unsigned l) { return __builtin_amdgcn_ds_bpermute(l * 4, v); }
#define WAIT_VM(n) asm volatile("s_waitcnt vmcnt(" #n ")" ::: "memory")

template <int NAP>
__device__ __forceinline__ bool spin_ok(u64 t0, unsigned call, char* myws) {
  const u64 dt = __builtin_amdgcn_s_memrealtime() - t0;
  if (dt > 10000ull) {
    if (rfl(ld_dev(myws + OFF_TMO)) == call || dt > DEADLINE) {
      char* const c = myws + OFF_TM2 + (threadIdx.x & 63) * 128;
      st_dev(c, call); st_dev(c + 64 * 128, call); st_dev(myws + OFF_TMO, call);
      return false;
    }
    __builtin_amdgcn_s_sleep(32);
  }
  __builtin_amdgcn_s_sleep(NAP);
  return true;
}
__device__ __forceinline__ bool wait_flags(const void* p, unsigned call, char* myws, u64 mask) {
  const u64 t0 = __builtin_amdgcn_s_memrealtime();
  while ((__ballot(ld_sys(p) >= call) & mask) != mask)
    if (!spin_ok<1>(t0, call, myws)) return false;
  __builtin_amdgcn_fence(__ATOMIC_ACQUIRE, "wavefront");
  return true;
}
__device__ __forceinline__ unsigned wait_word(unsigned off, unsigned call, char* myws) {
  const u64 t0 = __builtin_amdgcn_s_memrealtime();
  for (;;) {
    const u64 v = rfl64(ld_dev64(myws + off));
    if ((unsigned)(v >> 32) == call) return (unsigned)v;
    if (!spin_ok<8>(t0, call, myws)) return DEC0;
  }
}

__device__ __forceinline__ __amdgpu_buffer_rsrc_t rsrc(const void* base, unsigned bytes) {
  return __builtin_amdgcn_make_buffer_rsrc((void*)base, 0, bytes, 0x00020000);
}

template <class F>
__device__ __forceinline__ void stage_piece(const Params& P, unsigned u, unsigned e, unsigned lane, char* base, unsigned ub, u32x4* sc, F&& after_loads) {
  const unsigned h = u & 1;
  const bool isx = u < 64;
  const unsigned o = isx ? (64 * (u >> 1) + 32 * (e & 1) + (lane >> 2)) * 512 + h * 256 + (e >> 1) * 64 + (lane & 3) * 16
                         : (128 * h + 16 * e + (lane >> 3)) * 4096 + ((u - 64) >> 1) * 128 + (lane & 7) * 16;
  const auto rs = rsrc(isx ? P.x : P.w, IN_BYTES);
  u32x4 a = __builtin_amdgcn_raw_buffer_load_b128(rs, o, 0, 0);
  u32x4 b = __builtin_amdgcn_raw_buffer_load_b128(rs, o + (isx ? 16 * 512 : 8 * 4096), 0, 0);
  __builtin_amdgcn_sched_barrier(0);
  after_loads();
  __builtin_amdgcn_sched_barrier(0);
  unsigned oa, ob;
  if (isx) {
    const unsigned wi = (lane & 3) * 33 + (lane >> 2);
    sc[wi] = a; sc[wi + 16] = b;
    const unsigned ri = (lane >> 5) * 33 + (lane & 31);
    a = sc[ri]; b = sc[ri + 66];
    oa = ub + ((e & 6) * 2 + (e & 1)) * 1024 + lane * 16; ob = oa + 2048;
  } else {
    sc[lane] = a; sc[lane + 64] = b;
    const unsigned* const sd = (const unsigned*)sc + (lane >> 5) * 256 + (lane & 31);
    unsigned d[8];
#pragma unroll
    for (int j = 0; j < 8; ++j) d[j] = sd[j * 32];
#pragma unroll
    for (int j = 0; j < 4; ++j) {
      a[j] = __builtin_amdgcn_perm(d[2 * j + 1], d[2 * j], 0x05040100u);
      b[j] = __builtin_amdgcn_perm(d[2 * j + 1], d[2 * j], 0x07060302u);
    }
    oa = ub + e * 2048 + lane * 16; ob = oa + 1024;
  }
  const auto rd = rsrc(base + OFF_DATA, NR * SLICE);
  __builtin_amdgcn_raw_buffer_store_b128(a, rd, oa, 0, AUX_SYS);
  __builtin_amdgcn_raw_buffer_store_b128(b, rd, ob, 0, AUX_SYS);
}
#define HD __host__ __device__ __forceinline__
HD bool wide(unsigned r) { return r < 6; }
HD unsigned col0(unsigned r) { return r < 3 ? 0 : r < 6 ? 13 : 26; }
HD unsigned ncol(unsigned r) { return wide(r) ? 13 : 6; }
HD unsigned nfix(unsigned r) { return wide(r) ? 8 : 15; }
HD unsigned place(unsigned r) { return r < 3 ? r : r < 6 ? r - 3 : 2 * (r - 6); }
HD unsigned ntop(unsigned dec, unsigned r) { return wide(r) ? 9 + ((dec >> (r < 3 ? 0 : 4)) & 3) : 15 + ((dec >> 8) & 3); }
HD unsigned nbot(unsigned dec, unsigned r) { return wide(r) ? 9 + ((dec >> (r < 3 ? 2 : 6)) & 3) : 32 - ntop(dec, r); }
HD unsigned nrow(unsigned dec, unsigned r) { return place(r) == 0 ? ntop(dec, r) : place(r) == 2 ? nbot(dec, r) : 32 - ntop(dec, r) - nbot(dec, r); }
HD unsigned gband(unsigned a, unsigned r, unsigned dec) {
  if (place(r) != 1) return place(r) ? 31 - a : a;
  const unsigned up = 12 - ntop(dec, r);
  return a < 8 ? 12 + a : a - 8 < up ? 19 - a : 12 + a - up;
}
HD unsigned unit_of(unsigned code, unsigned h, unsigned r, unsigned dec) {
  return code < 32 ? gband(code, r, dec) * 2 + h : 64 + (col0(r) + (code - 32)) * 2 + h;
}
HD unsigned code_at(unsigned i, unsigned r, unsigned tabv) {
  return wide(r) ? (i < 16 ? tabv : i < 21 ? 32 + i - 8 : i - 13) : (i < 12 ? tabv : i - 6);
}
HD unsigned tile_at(unsigned ti, unsigned o, unsigned tabv) {
  if (ti < (wide(o) ? 64u : 36u)) return (tabv >> 3) << 4 | (tabv & 7);
  if (!wide(o)) return (6 + (ti - 36) / 6) << 4 | (ti - 36) % 6;
  return ti < 104 ? (ti & 7) << 4 | (ti >> 3) : (8 + (ti - 104) / 13) << 4 | (ti - 104) % 13;
}
HD unsigned fo_of(unsigned ti, unsigned o) {
  const unsigned e = wide(o) ? 104 : 36;
  return ti < 32 ? ti : ti < e ? 24 + (ti >> 2) : 24 + (e >> 2) + (ti - e) / ncol(o);
}
HD unsigned gsize(unsigned o, unsigned t) { return 2 * (wide(o) ? (wide(t) ? GA : GB) : GC); }
HD unsigned tshare(unsigned o) { return wide(o) ? T0A : T0B; }
HD unsigned nshared(unsigned o) { return wide(o) ? 2 * GB + 5 * GA : 7 * GC; }
HD unsigned gfirst(unsigned o, unsigned t) {
  const unsigned i = t < o ? t : t - 1;
  return tshare(o) + (wide(o) ? (wide(t) ? 2 * GB + GA * i : GB * (t - 6)) : GC * i);
}
HD unsigned head_len(unsigned dec, unsigned r, unsigned o) {
  return N1 + 8 * (nrow(dec, r) - nfix(r)) + 2 * tshare(o);
}
HD unsigned late_len(unsigned dec, unsigned o) { return 2 * (nrow(dec, o) * ncol(o) - tshare(o) - nshared(o)); }

__device__ __forceinline__ unsigned lag_bit(unsigned k) { return 7 * k + (k >= 4 ? 4 : 0); }

__device__ __forceinline__ int share_of(u64 plan, unsigned k, unsigned kk, int tot) {
  const int dk = (int)(plan >> lag_bit(k)) & 127, dq = (int)(plan >> lag_bit(kk)) & 127;
  const int dl = k < kk ? dk - dq : dq - dk;
  int sl = (7 * tot + 14 * 32 - 2 * dl + 2 * (int)((k + kk) % 7) + 1) / 14 - 32;
  sl = sl < 0 ? 0 : sl > tot ? tot : sl;
  return k < kk ? sl : tot - sl;
}
__device__ __forceinline__ int firm(int share) { return share > MRG ? share - MRG : 0; }

template <bool B> struct BoolC { static constexpr bool value = B; };

__global__ void __launch_bounds__(512, 2) fused_gemm_ar(const Params P) {
  __shared__ f32x4 lds[8192];
  const int lane = threadIdx.x & 63;
  const int wave = __builtin_amdgcn_readfirstlane(threadIdx.x >> 6);
  const int bid = blockIdx.x;
  const unsigned rank = P.rank & 7u;
  char* const myws = P.myws;
  char* const myout = P.myout;
  const unsigned task = P.task[bid & 127];
  const unsigned mv = (bid - NGEMM) & 127, k = mv % 7, j0 = mv / 7;
  asm volatile("" :: "s"(P.x), "s"(P.w), "s"(myws), "s"(myout), "s"(rank), "s"(task));
  unsigned* const cntp = (unsigned*)(myws + OFF_CNT) + bid * 8 + wave;

  if (bid < NGEMM) {
    unsigned cnt_raw;
    char* peer_ws;
    {
      u32x4* const scr = (u32x4*)lds + wave * 1024;
      if ((task >> 19) & 1) {
        const unsigned u = (task >> 8) & 127u;
        char* const dws = P.ws[(task >> 16) & 7u];
        stage_piece(P, u, wave, lane, dws, rank * SLICE + u * UNIT, scr, [&]() { cnt_raw = ld_dev(cntp); });
        WAIT_VM(0);
        __builtin_amdgcn_s_barrier();
        if (wave == 0 && lane < 4) st_sys(dws + OFF_FL + (rank * 128 + u) * 128 + lane * 4, rfl(cnt_raw) + 1);
      } else if ((task >> 20) & 1) __builtin_amdgcn_s_sleep(16);
      const unsigned u = task & 127u;
      stage_piece(P, u, wave, lane, myws, rank * SLICE + u * UNIT, scr, [&]() { cnt_raw = ld_dev(cntp); peer_ws = P.ws[lane & 7]; });
      const bool need = lane < 8 ? (task >> (24 + lane)) & 1 : lane < 12;
      char* const fp = lane < 8 ? peer_ws + OFF_FS + (rank * 128 + u) * 128 : myws + OFF_FL + (rank * 128 + u) * 128 + (lane & 3) * 4;
      WAIT_VM(0);
      if (!((task >> 19) & 3) && lane < 8 && need)
        st_sys(myws + OFF_SW + (lane * 128 + u) * 128 + wave * 4, rfl(cnt_raw) + 1);
      __builtin_amdgcn_s_barrier();
      if (wave == 0 && need) st_sys(fp, rfl(cnt_raw) + 1);
    }
    const unsigned call = rfl(cnt_raw) + 1;
    if (lane == 0) st_dev(cntp, call);

    unsigned job = P.job[bid];
#pragma unroll 1
    for (;;) {
    if (!((job >> 18) & 1)) return;
    const unsigned ti_ = job & 255u, ta_ = (job >> 8) & 31u, tb_ = (job >> 13) & 15u;
    const bool wfirst = (job >> 17) & 1;
    unsigned dec = DEC0;
    if (ta_ >= nfix(rank)) { dec = wait_word(OFF_DEC, call, myws); if (ta_ >= nrow(dec, rank)) return; }
    const unsigned grow = gband(ta_, rank, dec), gcol = col0(rank) + tb_;
    const unsigned ue = wfirst ? 64 + 2 * gcol : 2 * grow;
    const unsigned ul = wfirst ? 2 * grow : 64 + 2 * gcol;
    const auto rdat = rsrc(myws + OFF_DATA, NR * SLICE);
    const unsigned rot = (grow >> 2) + (gcol >> 1);
    const unsigned dbase = wave * SLICE + lane * 16;
    auto ld = [&](unsigned u, int f) {
      return __builtin_amdgcn_raw_buffer_load_b128(rdat, dbase + (f & 3) * 1024, u * UNIT + ((((f >> 2) + rot) & 3) << 12), AUX_SYS);
    };
    f32x16 acc[2][2];
    u32x4* const stash = (u32x4*)lds + wave * 1024 + lane;
    bool ok = wait_flags(myws + OFF_FL + (wave * 128 + ue + ((lane >> 2) & 1)) * 128 + (lane & 3) * 4, call, myws, 0xff);
    auto body = [&](auto WF) {
      constexpr bool W1 = decltype(WF)::value;
      u32x4 r1[16], r2[16];
#pragma unroll
      for (int f = 0; f < 16; ++f) r2[f] = ld(ue + 1, f);
#pragma unroll
      for (int f = 0; f < 16; ++f) r1[f] = ld(ue, f);
      __builtin_amdgcn_sched_barrier(0);
#pragma unroll
      for (int f = 0; f < 16; ++f) stash[f * 64] = r2[f];
      if (ok) ok = wait_flags(myws + OFF_FL + (wave * 128 + ul + ((lane >> 2) & 1)) * 128 + (lane & 3) * 4, call, myws, 0xff);
#pragma unroll
      for (int f = 0; f < 16; ++f) r2[f] = ld(ul, f);
      __builtin_amdgcn_sched_barrier(0);
#pragma unroll
      for (int h = 0; h < 2; ++h)
#pragma unroll
        for (int ks = 0; ks < 8; ++ks) {
#pragma unroll
          for (int i = 0; i < 2; ++i)
#pragma unroll
            for (int j = 0; j < 2; ++j) {
              const bf16x8 wv = __builtin_bit_cast(bf16x8, W1 ? r1[ks * 2 + j] : r2[ks * 2 + j]);
              const bf16x8 xv = __builtin_bit_cast(bf16x8, W1 ? r2[ks * 2 + i] : r1[ks * 2 + i]);
              acc[i][j] = __builtin_amdgcn_mfma_f32_32x32x16_bf16(wv, xv, (h | ks) ? acc[i][j] : f32x16{}, 0, 0, 0);
            }
          if (h == 0) {
#pragma unroll
            for (int t = 0; t < 2; ++t) { r2[ks * 2 + t] = ld(ul + 1, ks * 2 + t); r1[ks * 2 + t] = stash[(ks * 2 + t) * 64]; }
            __builtin_amdgcn_sched_barrier(0);
          }
        }
    };
    if (wfirst) body(BoolC<true>{}); else { asm volatile("; row band first" ::: "memory"); body(BoolC<false>{}); }
    const unsigned tm = ld_dev(myws + OFF_TMO);
    if (!ok) {
#pragma unroll
      for (int i = 0; i < 2; ++i)
#pragma unroll
        for (int j = 0; j < 2; ++j)
#pragma unroll
          for (int r = 0; r < 16; ++r) acc[i][j][r] = __builtin_nanf("");
    }
    {
      const unsigned m = lane & 31, hh = lane >> 5;
#pragma unroll
      for (int i = 0; i < 2; ++i)
#pragma unroll
        for (int g = 0; g < 4; ++g)
#pragma unroll
          for (int j = 0; j < 2; ++j)
            lds[wave * 1024 + ((i * 4 + g) * 2 + j) * 64 + hh * 32 + ((m + 4 * (2 * (g & 1) + hh)) & 31)] =
                f32x4{acc[i][j][4 * g], acc[i][j][4 * g + 1], acc[i][j][4 * g + 2], acc[i][j][4 * g + 3]};
    }
    __syncthreads();
    {
      const unsigned row = 8 * wave + (lane >> 3), c = lane & 7, g = c >> 1, hh = c & 1;
      const unsigned slot = (((row >> 5) * 4 + g) * 2) * 64 + hh * 32 + (((row & 31) + 4 * (c & 3)) & 31);
      f32x4 r0[8], r1[8];
#pragma unroll
      for (int w = 0; w < 8; ++w) { r0[w] = lds[w * 1024 + slot]; r1[w] = lds[w * 1024 + slot + 64]; }
      __builtin_amdgcn_sched_barrier(0);
      f32x4 e0 = r0[0], e1 = r1[0];
#pragma unroll
      for (int w = 1; w < 8; ++w) { e0 += r0[w]; e1 += r1[w]; }
      u32x4 o;
#pragma unroll
      for (int i = 0; i < 4; ++i) o[i] = __builtin_bit_cast(unsigned, __builtin_convertvector(f32x2{e0[i], e1[i]}, bf16x2));
      if (tm == call) o = u32x4{NANPK, NANPK, NANPK, NANPK};
      __builtin_amdgcn_raw_buffer_store_b128(o, rsrc(myout, OUT_BYTES), (64 * grow + row) * 4096 + gcol * 128 + c * 16, 0, AUX_SYS);
    }
    WAIT_VM(0);
    __builtin_amdgcn_s_barrier();
    if (wave == 0) {
      bool tell = true;
      const unsigned fo = fo_of(ti_, rank);
      if (ti_ >= 32) {
        const unsigned gs = ti_ < (wide(rank) ? 104u : 36u) ? 4 : ncol(rank);
        unsigned old = 0, off = OFF_FC + (fo - 32) * 128;
        asm volatile("" : "+v"(off));
        if (lane == 0) old = __hip_atomic_fetch_add((unsigned*)(myws + off), 1u, __ATOMIC_RELAXED, __HIP_MEMORY_SCOPE_AGENT);
        tell = rfl(old) % gs == gs - 1;
      }
      if (tell && lane < 8 && (unsigned)lane != rank) st_sys(peer_ws + OFF_FO + (rank * 128 + fo) * 128, call);
    }
    if (ti_ >= 28 || !wide(rank)) return;
    const unsigned t2 = tile_at(128 + ti_, rank, 0);
    job = (128 + ti_) | (t2 >> 4) << 8 | (t2 & 15) << 13 | 3u << 17;
    }
  }

  const u64 t_in = __builtin_amdgcn_s_memrealtime();
  if (bid == NGEMM + NMOVE) {
    if (wave) return;
    char* const peer_ws = P.ws[lane & 7];
    const unsigned call = rfl(ld_dev(cntp)) + 1;
    if (lane == 0) st_dev(cntp, call);
    const unsigned kl = lane < 7 ? lane : 0;
    auto drawn = [&]() { const u64 w = ld_dev64(myws + OFF_LC + kl * 128); return (unsigned)(w >> 32) == call ? (unsigned)w + NW : 0u; };
    auto post = [&](unsigned off, u64 v) {
      __hip_atomic_store((u64*)(myws + OFF_TM2 + lane * 128 + off), v, __ATOMIC_RELAXED, __HIP_MEMORY_SCOPE_AGENT);
      __hip_atomic_store((u64*)(myws + OFF_TM2 + (64 + lane) * 128 + off), v, __ATOMIC_RELAXED, __HIP_MEMORY_SCOPE_AGENT);
    };
    bool alive = true;
    unsigned dec = 0;
    {
      unsigned ts = 0;
      for (;;) {
        const bool go = drawn() >= C0;
        const u64 dt = __builtin_amdgcn_s_memrealtime() - t_in;
        if (go && !ts) ts = dt > 100000 ? 100000u : (unsigned)dt + 1;
        if (__all(ts != 0) || !(alive = spin_ok<2>(t_in, call, myws))) break;
      }
      unsigned sum = 0;
#pragma unroll
      for (int i = 0; i < 7; ++i) sum += rdl(ts, i);
      if (lane < 8) __hip_atomic_store((u64*)(peer_ws + OFF_WV + rank * 128), (u64)call << 32 | sum, __ATOMIC_RELAXED, __HIP_MEMORY_SCOPE_SYSTEM);
      u64 v;
      for (;;) {
        v = __hip_atomic_load((const u64*)(myws + OFF_WV + (lane & 7) * 128), __ATOMIC_RELAXED, __HIP_MEMORY_SCOPE_SYSTEM);
        if (__all((unsigned)(v >> 32) == call) || !(alive = alive && spin_ok<2>(t_in, call, myws))) break;
      }
      {
        const unsigned st = (lane >> 4) > 2 ? 2 : lane >> 4, c = lane & 15, cand = st == 2 ? (c & 3) << 8 : c << (4 * st);
        unsigned worst = 0;
#pragma unroll
        for (int i = 0; i < 3; ++i) {
          const unsigned r = st == 2 ? 6 + (i & 1) : 3 * st + i, n = nrow(cand, r);
          const unsigned cost = pick((unsigned)v, r) + IQ * (56 - 2 * ncol(r)) * n;
          worst = (n < 9 || n > (wide(r) ? 12u : 17u) || (st == 2 && c > 2)) ? 0x3ffffffu : cost > worst ? cost : worst;
        }
        unsigned key = (worst > 0x3ffffffu ? 0x3ffffffu : worst) << 4 | c;
#pragma unroll
        for (int d = 1; d < 16; d <<= 1) { const unsigned o = pick(key, lane ^ d); key = o < key ? o : key; }
        dec = (rdl(key, 0) & 15u) | (rdl(key, 16) & 15u) << 4 | (rdl(key, 32) & 3u) << 8;
      }
      if (!alive) dec = DEC0;
      post(16, call << 16 | dec);
      if (lane == 0) __hip_atomic_store((u64*)(myws + OFF_DEC), (u64)call << 32 | dec, __ATOMIC_RELAXED, __HIP_MEMORY_SCOPE_AGENT);
    }
    const unsigned pl = rank ^ (kl + 1), want = call * gsize(pl, rank);
    unsigned known = head_len(dec, rank, pl), rest = 2 * (late_len(dec, pl) + gsize(pl, rank));
#pragma unroll
    for (int i = 0; i < 7; ++i) if ((unsigned)i != kl) rest += gsize(pl, rank ^ (i + 1)) + gsize(rank ^ (i + 1), pl);
    unsigned todo = 0x1ffu;
    while (todo) {
      const unsigned dr = drawn(), ff = ld_dev(myws + OFF_FF + kl * 128);
      if ((todo & 0x180) && (__any(dr + ((todo & 0x80) ? TSNAP : TSNAP2) >= known) || !alive)) {
        const unsigned left = known + rest / 2 + 512 - dr;
        unsigned mn = 0xffffu;
#pragma unroll
        for (int i = 0; i < 7; ++i) { const unsigned x = rdl(left, i); mn = x < mn ? x : mn; }
        u64 val = (u64)(call & 15u) << 60 | (u64)(call & 15u) << 28;
#pragma unroll
        for (int i = 0; i < 7; ++i) { const unsigned d = rdl(left, i) - mn; val |= (u64)(d > 127 ? 127u : d) << lag_bit(i); }
        post((todo & 0x80) ? 8 : 24, val);
        known += late_len(dec, pl) + gsize(pl, rank); rest = 0;
#pragma unroll
        for (int i = 0; i < 7; ++i)
          if ((unsigned)i != kl) {
            const int tot = gsize(pl, rank ^ (i + 1)) + gsize(rank ^ (i + 1), pl), sh = share_of(val, kl, i, tot);
            known += firm(sh); rest += tot - firm(sh) - firm(tot - sh);
          }
        todo &= (todo & 0x80) ? ~0x80u : ~0x100u;
        if (!alive) continue;
      }
      const unsigned have = (unsigned)__ballot((int)(ff - want) >= 0) & todo & 127u;
#pragma unroll
      for (int i = 0; i < 7; ++i)
        if (((have >> i) & 1) && lane < 8 && (unsigned)lane != rank) st_sys(peer_ws + OFF_FG + ((rank ^ (i + 1)) * 8 + rank) * 128, call);
      todo &= ~have;
      if (!alive) break;
      if (todo) alive = spin_ok<2>(t_in, call, myws);
    }
    return;
  }

  if (wave >= NBW) return;
  const char* const pws = P.lws[k];
  const char* const pout = P.lout[k];
  const unsigned p = (rank ^ (k + 1)) & 7u;
  const unsigned tab = P.tab[lane];
  unsigned cnt_raw = 0;
  if (wave < 2) cnt_raw = ld_dev(cntp);
  typedef __attribute__((address_space(3))) volatile u32x4 lds_u32x4;
  lds_u32x4* const tbl = (lds_u32x4*)lds;
  auto tile_entry = [&](unsigned dec, unsigned o, unsigned ti, unsigned half, unsigned fa, unsigned noff) {
    const unsigned t = tile_at(ti, o, (pick(tab, ti & 63) >> 8) & 63u);
    return u32x4{(64 * (gband(t >> 4, o, dec) & 31u) + 32 * half) * 4096 + (col0(o) + (t & 15)) * 128, fa, noff, 2u};
  };
  auto entry = [&](unsigned n, unsigned dec) {
    const unsigned nin = N1 + 8 * (nrow(dec, rank) - nfix(rank));
    const unsigned ui = NPUSH + (n >> 2), u = unit_of(code_at(ui >> 1, rank, pick(tab, (ui >> 1) & 63) & 63u), ui & 1, rank, dec) & 127u;
    const unsigned y = n - nin, ti = (y >> 1) & 255u;
    const u32x4 te = tile_entry(dec, p, ti, y & 1, OFF_FO + (p * 128 + fo_of(ti, p)) * 128, 0);
    if (n < nin) return u32x4{p * SLICE + u * UNIT + (n & 3) * 4096, OFF_FS + (p * 128 + u) * 128, OFF_FL + (p * 128 + u) * 128 + (n & 3) * 4, 1u};
    return n < head_len(dec, rank, p) ? te : u32x4{0, 0, 0, 0};
  };
  typedef __attribute__((address_space(3))) volatile unsigned lds_u32;
  lds_u32* const box = (lds_u32*)(tbl + TBL);
#pragma unroll 1
  for (unsigned n0 = wave * 64; n0 < TBL; n0 += NBW * 64) {
    u32x4 e = {0, 0, 0, 0};
    if (n0 < N1) e = entry(n0 + lane, DEC0);
    tbl[n0 + lane] = n0 + lane < N1 ? e : u32x4{0, 0, 0, 0};
  }
  const unsigned call = rfl(cnt_raw) + 1;
  const unsigned fprev = lane == 4 ? (call + 1) << 16 : (call + 1) << 28;
  if (threadIdx.x < 8) box[threadIdx.x] = fprev;
  if (wave < 2 && lane == 0) {
    st_dev(cntp, call);
    if (wave == 0) __hip_atomic_fetch_max((u64*)(myws + OFF_LC + k * 128), (u64)call << 32, __ATOMIC_RELAXED, __HIP_MEMORY_SCOPE_AGENT);
  }
  __syncthreads();
  if (wave > 1) return;

  if (wave == 1) {
    typedef __attribute__((address_space(3))) volatile u64 lds_u64;
    unsigned dec;
    while (((dec = rfl(box[4])) ^ call << 16) >> 16) if (!spin_ok<4>(t_in, call, myws)) { dec = DEC0; break; }
    unsigned la = head_len(dec, rank, p);
#pragma unroll 1
    for (unsigned n = N1; n < la; n += 64) tbl[n + lane] = entry(n + lane, dec);
    const unsigned l8 = lane < 9 ? lane & 7 : (lane - 9) & 7, q = wide(p) ? (l8 + 6) & 7 : l8, kk = ((rank ^ q) - 1) & 7u;
    const bool pair = q != p && q != rank;
    const int gd = gsize(p, q), tot = gd + gsize(q, p);
    int beg = 0, open = tot;
#pragma unroll 1
    for (int deal = 0; deal < 2; ++deal) {
      u64 plan;
      while ((((plan = rfl64(*(lds_u64*)(box + 2 + 4 * deal))) >> 28) & 0xf0000000full) != (call & 15u) * 0x100000001ull)
        if (!spin_ok<2>(t_in, call, myws)) { plan = 0; break; }
      const int sh = share_of(plan, k, kk, open), end = deal ? beg + sh : firm(sh);
      const int b = lane < 8 ? (beg < gd ? beg : gd) : (beg > gd ? beg : gd), e = lane < 8 ? (end < gd ? end : gd) : (end > gd ? end : gd);
      const unsigned len = lane == 8 ? (deal ? 0u : late_len(dec, p)) : lane > 16 ? 0u : pair ? e - b : lane < 8 && q == rank && !deal ? gd : 0u;
      const unsigned org = lane == 8 ? 2 * (tshare(p) + nshared(p)) : lane < 8 ? 2 * gfirst(p, q) + b : 2 * gfirst(q, p) + (tot - 1 - b);
      unsigned total = 0;
#pragma unroll
      for (int i = 0; i < 17; ++i) total += rdl(len, i);
#pragma unroll 1
      for (unsigned e0 = 0; e0 < total + (deal ? NW : 0); e0 += 64) {
        unsigned l = 0, sub = e0 + lane;
#pragma unroll
        for (int i = 0; i < 17; ++i) { const unsigned x = rdl(len, i); if (l == (unsigned)i && sub >= x) { sub -= x; ++l; } }
        const unsigned ql = pick(q, l), og = pick(org, l), h = l < 9 ? og + sub : og - sub, ti = (h >> 1) & 255u;
        const u32x4 te = tile_entry(dec, l < 9 ? p : ql, ti, h & 1, l < 9 ? OFF_FO + (p * 128 + fo_of(ti, p)) * 128 : OFF_FG + (ql * 8 + p) * 128,
                                    l < 8 && ql == rank ? (OFF_FF + k * 128) | 1u : 0u);
        const unsigned at = la + e0 + lane;
        tbl[at < TBL ? at : TBL - 1] = l < 17 ? te : u32x4{0, OFF_TMO, 0, deal ? 3u : 0u};
      }
      la += total; beg = end; open = tot - end - firm(tot - sh);
    }
    return;
  }

  auto desc = [](const void* b, unsigned bytes) { const u64 a = (u64)b; return u32x4{rfl((unsigned)a), rfl((unsigned)(a >> 32) & 0xffffu), rfl(bytes), 0x00020000u}; };
  const unsigned vcall = call, vone = 1, vlc = OFF_LC + k * 128;
  const unsigned lo_lin = lane * 16, lo_tile = (lane >> 3) * 4096 + (lane & 7) * 16;
  const unsigned tm_l = OFF_TM2 + mv * 128 + ((lane & 7) >= 2 ? (lane & 7) * 4 : 0);
  u32x4 w0 = {}, w1 = {}, w2 = {}, w3 = {};
  unsigned o0, o1, o2, o3, r0 = 0, r1 = 0, r2 = 0, r3 = 0;
  bool ok = true;
  unsigned kind, noff;
  unsigned nfo;
  auto fetch = [&](unsigned idx) {
    u32x4 e = tbl[idx < TBL ? idx : TBL - 1];
    while (rfl(e[3]) == 0) {
      const unsigned x = ld_sys(myws + tm_l);
      if (lane < 8) box[lane] = x;
      if (!spin_ok<2>(t_in, call, myws)) { e = u32x4{0, OFF_TMO, 0, 3u}; ok = false; break; }
      e = tbl[idx < TBL ? idx : TBL - 1];
    }
    kind = rfl(e[3]); noff = rfl(e[2]);
    const unsigned ts = kind == 2 ? 8 * 4096 : 1024;
    o0 = e[0] + (kind == 2 ? lo_tile : lo_lin); o1 = o0 + ts; o2 = o1 + ts; o3 = o2 + ts;
    nfo = (lane & 7) ? tm_l : e[1];
  };
  auto put = [&](unsigned kd) {
    const u32x4 rd = desc(kd == 2 ? myout : myws + OFF_DATA, kd == 2 ? OUT_BYTES : NR * SLICE);
    asm volatile("s_nop 4\n\tbuffer_store_dwordx4 %0, %4, %8, 0 offen sc0 sc1\n\tbuffer_store_dwordx4 %1, %5, %8, 0 offen sc0 sc1\n\t"
                 "buffer_store_dwordx4 %2, %6, %8, 0 offen sc0 sc1\n\tbuffer_store_dwordx4 %3, %7, %8, 0 offen sc0 sc1\n\ts_nop 1"
                 :: "v"(w0), "v"(w1), "v"(w2), "v"(w3), "v"(r0), "v"(r1), "v"(r2), "v"(r3), "s"(rd) : "memory");
  };
  auto notify = [&](unsigned off) {
    if (off & 1) asm volatile("s_mov_b64 exec, 1\n\tglobal_atomic_add %0, %1, %2\n\ts_mov_b64 exec, -1" :: "v"(off - 1), "v"(vone), "s"(myws) : "memory");
    else asm volatile("s_mov_b64 exec, 1\n\tglobal_store_dword %0, %1, %2 sc0 sc1\n\ts_mov_b64 exec, -1" :: "v"(off), "v"(vcall), "s"(myws) : "memory");
  };

  fetch(j0);
  {
    u64 bad;
    {
      const unsigned ro = OFF_SW + rank * 128 * 128 + ((rfl(nfo) - OFF_FS) & (128 * 128 - 1)) + (j0 & 3) * 8 + (lane & 1) * 4;
      unsigned left = rfl(ld_sys(myws + nfo)) >= call ? 0u : 64u;
      asm volatile(
          "s_mov_b64 %0, 0\n\ts_cmp_eq_u32 %4, 0\n\ts_cbranch_scc1 1f\n\t"
          "global_load_dword %1, %5, %6 sc0 sc1\n\ts_sleep 6\n\t"
          "global_load_dword %2, %5, %6 sc0 sc1\n\ts_sleep 6\n"
          "0:\n\t"
          "global_load_dword %3, %5, %6 sc0 sc1\n\t"
          "s_waitcnt vmcnt(2)\n\tv_cmp_gt_u32 %0, %7, %1\n\ts_cmp_eq_u64 %0, 0\n\ts_cbranch_scc1 1f\n\tglobal_load_dword %1, %5, %6 sc0 sc1\n\t"
          "s_waitcnt vmcnt(2)\n\tv_cmp_gt_u32 %0, %7, %2\n\ts_cmp_eq_u64 %0, 0\n\ts_cbranch_scc1 1f\n\tglobal_load_dword %2, %5, %6 sc0 sc1\n\t"
          "s_waitcnt vmcnt(2)\n\tv_cmp_gt_u32 %0, %7, %3\n\ts_cmp_eq_u64 %0, 0\n\ts_cbranch_scc1 1f\n\t"
          "s_sub_u32 %4, %4, 1\n\ts_cmp_lg_u32 %4, 0\n\ts_cbranch_scc1 0b\n"
          "1:"
          : "=&s"(bad), "=&v"(r1), "=&v"(r2), "=&v"(r3), "+s"(left) : "v"(ro), "s"(pws), "s"(call) : "memory", "scc");
    }
    if (bad) { ok = wait_flags(myws + nfo, call, myws, 1); WAIT_VM(0); }
  }
  unsigned skind = 0, snoff = 0, pnoff = 0;
  for (;;) {
    u32x4 v0, v1, v2, v3;
    unsigned f, tk;
    asm volatile("s_mov_b64 exec, 1\n\tglobal_atomic_add %0, %1, %2, %3 sc0\n\ts_mov_b64 exec, -1" : "=&v"(tk) : "v"(vlc), "v"(vone), "s"(myws) : "memory");
    const bool first = snoff != 0;
    if (first) { put(skind); pnoff = snoff; }
    {
      const u32x4 rs = desc(kind == 2 ? pout : pws + OFF_DATA, ok ? (kind == 2 ? OUT_BYTES : NR * SLICE) : 0u);
      asm volatile("s_nop 4\n\tbuffer_load_dwordx4 %0, %4, %8, 0 offen sc0 sc1\n\tbuffer_load_dwordx4 %1, %5, %8, 0 offen sc0 sc1\n\t"
                   "buffer_load_dwordx4 %2, %6, %8, 0 offen sc0 sc1\n\tbuffer_load_dwordx4 %3, %7, %8, 0 offen sc0 sc1"
                   : "=&v"(v0), "=&v"(v1), "=&v"(v2), "=&v"(v3) : "v"(o0), "v"(o1), "v"(o2), "v"(o3), "s"(rs) : "memory");
    }
    if (!first && skind) put(skind);
    const unsigned ckind = kind, cnoff = noff, q0 = o0, q1 = o1, q2 = o2, q3 = o3;
    if (!skind) WAIT_VM(4);
    asm volatile("s_waitcnt vmcnt(8)" : "+v"(tk) : "v"(r1), "v"(r2), "v"(r3) : "memory");
    fetch(rfl(tk) + NW);
    asm volatile("global_load_dword %0, %1, %2 sc0 sc1" : "=&v"(f) : "v"(nfo), "s"(myws) : "memory");
    asm volatile("" : "+v"(o0), "+v"(o1), "+v"(o2), "+v"(o3));
    if (pnoff) { WAIT_VM(5); notify(pnoff); pnoff = 0; WAIT_VM(1); } else WAIT_VM(0);
    asm volatile("" : "+v"(v0), "+v"(v1), "+v"(v2), "+v"(v3), "+v"(f) :: "memory");
    if (rdl(f, 1) == call) ok = false;
    w0 = v0; w1 = v1; w2 = v2; w3 = v3;
    if (!ok) w0 = w1 = w2 = w3 = u32x4{NANPK, NANPK, NANPK, NANPK};
    r0 = q0; r1 = q1; r2 = q2; r3 = q3; skind = ckind; snoff = cnoff;
    box[lane & 7] = f;
    if (kind == 3 || rfl(f) < call) {
      if (kind == 3 || rfl(ld_sys(myws + nfo)) < call) {
        put(skind);
        if (snoff) { WAIT_VM(0); notify(snoff); }
        skind = 0; snoff = 0;
        if (kind == 3) break;
        if (ok) ok = wait_flags(myws + nfo, call, myws, 1);
      }
    }
  }
  if (!ok && lane == 0) st_sys(myout + gband(0, rank, DEC0) * 64 * 4096 + col0(rank) * 128, NANPK);
}

}

extern "C" void kernel_launch(int rank, int nranks, ncclComm_t comm, hipStream_t stream,
                              void* const* d_in, const int* in_sizes, int n_in,
                              void* d_out, int out_size, const int* in_dtypes, int out_dtype,
                              void* const* d_out_all, void* const* d_ws_all, size_t ws_bytes) {
  if (nranks != NR || (unsigned)rank >= (unsigned)NR || n_in < 2 || ws_bytes < WS_NEED || out_size != 2048 * 2048 ||
      in_sizes[0] != 2048 * 256 || in_sizes[1] != 256 * 2048) { fprintf(stderr, "unexpected problem shape\n"); return; }
  Params P;
  P.x = (const char*)d_in[0]; P.w = (const char*)d_in[1];
  for (int p = 0; p < NR; ++p) P.ws[p] = (char*)d_ws_all[p];
  for (int k = 0; k < 8; ++k) { const int p = (rank ^ (k + 1)) & 7; P.lws[k] = (char*)d_ws_all[p]; P.lout[k] = (char*)d_out_all[p]; }
  P.myws = P.ws[rank]; P.myout = (char*)d_out_all[rank];
  P.rank = rank; P.pad = 0;
  const unsigned char bo[16] = {32, 0, 33, 1, 2, 34, 3, 35, 4, 36, 5, 37, 6, 38, 7, 39};
  unsigned char ord[64]; int no = 0, na = 0, nb = 0;
  for (int n = 0; n < 16; ++n) {
    if (bo[n] < 32) { for (int b = 0; b < nb; ++b) ord[no++] = na * 8 + b; ++na; }
    else { for (int a = 0; a < na; ++a) ord[no++] = a * 8 + nb; ++nb; }
  }
  for (int i = 0; i < 64; ++i) P.tab[i] = bo[i & 15] | (unsigned)ord[i] << 8;
  {
    int posn[64];
    for (unsigned i = 0; i < 25; ++i) posn[code_at(i, rank, bo[i & 15]) & 63] = i;
    const unsigned nt = wide(rank) ? 128 : 102;
    int used[8] = {0}; bool done[128] = {false};
    for (int b = 0; b < 128; ++b) P.job[b] = 0;
    for (int pass = 0; pass < 2; ++pass)
      for (unsigned ti = 0; ti < nt; ++ti) {
        const unsigned t = tile_at(ti, rank, ord[ti & 63]), a = t >> 4, b = t & 15;
        int x = (a & 3) * 2 + (b & 1);
        if (done[ti] || (pass == 0 && used[x] == 16)) continue;
        while (used[x] == 16) x = (x + 1) & 7;
        P.job[used[x]++ * 8 + x] = ti | a << 8 | b << 13 | (posn[32 + b] < posn[a] ? 1u : 0u) << 17 | 1u << 18;
        done[ti] = true;
      }
  }
  unsigned mask[128] = {0}; int first[128];
  for (int u = 0; u < 128; ++u) first[u] = 99;
  for (unsigned r = 0; r < NR; ++r)
    for (unsigned dec = 0; dec < 768; ++dec) {
      const unsigned n = nrow(dec, r);
      if (n < 9 || n > (wide(r) ? 12u : 17u)) continue;
      for (unsigned i = 0; i < 2 * (n + ncol(r)); ++i) {
        const unsigned u = unit_of(code_at(i >> 1, r, bo[(i >> 1) & 15]), i & 1, r, dec);
        if (r != (unsigned)rank && i >= NPUSH) { mask[u] |= 1u << r; if ((int)i < first[u]) first[u] = i; }
      }
    }
  int us[128], nu = 0;
  for (int u = 0; u < 128; ++u) if (first[u] == 99) first[u] = 20;
  for (int pos = 98; pos >= 0; --pos) for (int u = 0; u < 128; ++u) if (first[u] == pos) us[nu++] = u;
  int i0 = 0, i1 = 16; unsigned np = 0;
  for (int bb = 0; bb < 128; ++bb) {
    const int b = (bb + 96) & 127;
    const int u = us[(b & 7) == 0 ? i0++ : i1++];
    P.task[b] = u | mask[u] << 24 | (first[u] < (int)(NPUSH + NCRIT) ? 0u : 1u << 20);
    if ((b & 7) != 0 && np < 7 * NPUSH) {
      const unsigned dst = rank ^ ((np + np / 7) % 7 + 1), j = np / 7;
      P.task[b] = u | mask[u] << 24 | unit_of(bo[j >> 1], j & 1, dst, DEC0) << 8 | dst << 16 | 1u << 19; ++np;
    }
  }
  hipLaunchKernelGGL(fused_gemm_ar, dim3(NGEMM + NMOVE + 1), dim3(512), 0, stream, P);
}
```

```cpp
#include <hip/hip_runtime.h>
#include <rccl/rccl.h>
#include <cstdio>
#include <cstdint>

namespace {

typedef unsigned u32x4 __attribute__((ext_vector_type(4)));
typedef float f32x4 __attribute__((ext_vector_type(4)));
typedef float f32x16 __attribute__((ext_vector_type(16)));
typedef __bf16 bf16x8 __attribute__((ext_vector_type(8)));
typedef __bf16 bf16x2 __attribute__((ext_vector_type(2)));
typedef float f32x2 __attribute__((ext_vector_type(2)));
typedef unsigned long long u64;

constexpr unsigned NW = 16;
constexpr unsigned NPUSH = 4;
constexpr unsigned NCRIT = 8;
constexpr unsigned TG = 150;
constexpr unsigned C0 = 2 * NW + NW / 2;
constexpr unsigned IQ = 8;
constexpr unsigned T0A = 44, T0B = 48;
constexpr unsigned GA = 6, GB = 11, GC = 3;
constexpr int MRG = 2;
constexpr unsigned TSNAP = 48, TSNAP2 = 52;
constexpr int NBW = 4;

constexpr int NR = 8;
constexpr unsigned UNIT = 16384;
constexpr unsigned SLICE = 128 * UNIT;
constexpr unsigned OUT_BYTES = 2048u * 2048u * 2u;
constexpr unsigned IN_BYTES = 2048u * 256u * 2u;
constexpr unsigned KB = 1024;
constexpr unsigned OFF_CNT = 0;
constexpr unsigned OFF_TMO = 16 * KB;
constexpr unsigned OFF_LC = 17 * KB;
constexpr unsigned OFF_DEC = 18 * KB + 256;
constexpr unsigned OFF_WV = 19 * KB;
constexpr unsigned OFF_FS = 32 * KB;
constexpr unsigned OFF_FL = 160 * KB;
constexpr unsigned OFF_FO = 288 * KB;
constexpr unsigned OFF_FC = 416 * KB;
constexpr unsigned OFF_FF = 424 * KB;
constexpr unsigned OFF_FG = 432 * KB;
constexpr unsigned OFF_TM2 = 448 * KB;
constexpr unsigned OFF_SW = 512 * KB;
constexpr unsigned OFF_DATA = 2048 * KB;
constexpr size_t WS_NEED = OFF_DATA + (size_t)NR * SLICE;
constexpr int NGEMM = 128, NMOVE = 7 * NW;
constexpr unsigned N1 = (2 * 21 - NPUSH) * 4;
constexpr unsigned DEC0 = 2 | 2 << 2 | 2 << 4 | 2 << 6 | 1 << 8;
constexpr unsigned TBL = 704;
constexpr u64 DEADLINE = 100000000ull;
constexpr int AUX_SYS = 17;
constexpr unsigned NANPK = 0x7fc07fc0u;
static_assert(NW <= 16 && NGEMM + NMOVE + 1 <= 256 && NMOVE <= 128 && N1 + 32 + 2 * 156 + 12 * GB + 64 < TBL && T0A + 2 * GB + 5 * GA <= 104 && T0B + 7 * GC <= 90, "layout");


struct Params {
  const char* x; const char* w;
  char* myws; char* myout;
  unsigned rank, first;
  char* link[8][2];
  unsigned task[128];
  unsigned job[128];
  unsigned tab[64];
};

__device__ __forceinline__ unsigned ld_sys(const void* p) { return __hip_atomic_load((const unsigned*)p, __ATOMIC_RELAXED, __HIP_MEMORY_SCOPE_SYSTEM); }
__device__ __forceinline__ unsigned ld_dev(const void* p) { return __hip_atomic_load((const unsigned*)p, __ATOMIC_RELAXED, __HIP_MEMORY_SCOPE_AGENT); }
__device__ __forceinline__ u64 ld_dev64(const void* p) { return __hip_atomic_load((const u64*)p, __ATOMIC_RELAXED, __HIP_MEMORY_SCOPE_AGENT); }
__device__ __forceinline__ void st_sys(void* p, unsigned v) { __hip_atomic_store((unsigned*)p, v, __ATOMIC_RELAXED, __HIP_MEMORY_SCOPE_SYSTEM); }
__device__ __forceinline__ void st_dev(void* p, unsigned v) { __hip_atomic_store((unsigned*)p, v, __ATOMIC_RELAXED, __HIP_MEMORY_SCOPE_AGENT); }
__device__ __forceinline__ unsigned rfl(unsigned v) { return __builtin_amdgcn_readfirstlane(v); }
__device__ __forceinline__ u64 rfl64(u64 v) { return (u64)rfl((unsigned)(v >> 32)) << 32 | rfl((unsigned)v); }
__device__ __forceinline__ unsigned rdl(unsigned v, unsigned l) { return __builtin_amdgcn_readlane(v, l); }
__device__ __forceinline__ unsigned pick(unsigned v, unsigned l) { return __builtin_amdgcn_ds_bpermute(l * 4, v); }
#define WAIT_VM(n) asm volatile("s_waitcnt vmcnt(" #n ")" ::: "memory")

template <int NAP>
__device__ __forceinline__ bool spin_ok(u64 t0, unsigned call, char* myws) {
  const u64 dt = __builtin_amdgcn_s_memrealtime() - t0;
  if (dt > 10000ull) {
    if (rfl(ld_dev(myws + OFF_TMO)) == call || dt > DEADLINE) {
      char* const c = myws + OFF_TM2 + (threadIdx.x & 63) * 128;
      st_dev(c, call); st_dev(c + 64 * 128, call); st_dev(myws + OFF_TMO, call);
      return false;
    }
    __builtin_amdgcn_s_sleep(32);
  }
  __builtin_amdgcn_s_sleep(NAP);
  return true;
}
__device__ __forceinline__ bool wait_flags(const void* p, unsigned call, char* myws, u64 mask) {
  const u64 t0 = __builtin_amdgcn_s_memrealtime();
  while ((__ballot(ld_sys(p) >= call) & mask) != mask)
    if (!spin_ok<1>(t0, call, myws)) return false;
  __builtin_amdgcn_fence(__ATOMIC_ACQUIRE, "wavefront");
  return true;
}
__device__ __forceinline__ unsigned wait_word(unsigned off, unsigned call, char* myws) {
  const u64 t0 = __builtin_amdgcn_s_memrealtime();
  for (;;) {
    const u64 v = rfl64(ld_dev64(myws + off));
    if ((unsigned)(v >> 32) == call) return (unsigned)v;
    if (!spin_ok<8>(t0, call, myws)) return DEC0;
  }
}

__device__ __forceinline__ __amdgpu_buffer_rsrc_t rsrc(const void* base, unsigned bytes) {
  return __builtin_amdgcn_make_buffer_rsrc((void*)base, 0, bytes, 0x00020000);
}

template <class F>
__device__ __forceinline__ void stage_piece(const Params& P, unsigned u, unsigned e, unsigned lane, char* base, unsigned ub, u32x4* sc, F&& after_loads) {
  const unsigned h = u & 1;
  const bool isx = u < 64;
  const unsigned o = isx ? (64 * (u >> 1) + 32 * (e & 1) + (lane >> 2)) * 512 + h * 256 + (e >> 1) * 64 + (lane & 3) * 16
                         : (128 * h + 16 * e + (lane >> 3)) * 4096 + ((u - 64) >> 1) * 128 + (lane & 7) * 16;
  const auto rs = rsrc(isx ? P.x : P.w, IN_BYTES);
  u32x4 a = __builtin_amdgcn_raw_buffer_load_b128(rs, o, 0, 0);
  u32x4 b = __builtin_amdgcn_raw_buffer_load_b128(rs, o + (isx ? 16 * 512 : 8 * 4096), 0, 0);
  __builtin_amdgcn_sched_barrier(0);
  after_loads();
  __builtin_amdgcn_sched_barrier(0);
  unsigned oa, ob;
  if (isx) {
    const unsigned wi = (lane & 3) * 33 + (lane >> 2);
    sc[wi] = a; sc[wi + 16] = b;
    const unsigned ri = (lane >> 5) * 33 + (lane & 31);
    a = sc[ri]; b = sc[ri + 66];
    oa = ub + ((e & 6) * 2 + (e & 1)) * 1024 + lane * 16; ob = oa + 2048;
  } else {
    sc[lane] = a; sc[lane + 64] = b;
    const unsigned* const sd = (const unsigned*)sc + (lane >> 5) * 256 + (lane & 31);
    unsigned d[8];
#pragma unroll
    for (int j = 0; j < 8; ++j) d[j] = sd[j * 32];
#pragma unroll
    for (int j = 0; j < 4; ++j) {
      a[j] = __builtin_amdgcn_perm(d[2 * j + 1], d[2 * j], 0x05040100u);
      b[j] = __builtin_amdgcn_perm(d[2 * j + 1], d[2 * j], 0x07060302u);
    }
    oa = ub + e * 2048 + lane * 16; ob = oa + 1024;
  }
  const auto rd = rsrc(base + OFF_DATA, NR * SLICE);
  __builtin_amdgcn_raw_buffer_store_b128(a, rd, oa, 0, AUX_SYS);
  __builtin_amdgcn_raw_buffer_store_b128(b, rd, ob, 0, AUX_SYS);
}
#define HD __host__ __device__ __forceinline__
HD bool wide(unsigned r) { return r < 6; }
HD unsigned col0(unsigned r) { return r < 3 ? 0 : r < 6 ? 13 : 26; }
HD unsigned ncol(unsigned r) { return wide(r) ? 13 : 6; }
HD unsigned nfix(unsigned r) { return wide(r) ? 8 : 15; }
HD unsigned place(unsigned r) { return r < 3 ? r : r < 6 ? r - 3 : 2 * (r - 6); }
HD unsigned ntop(unsigned dec, unsigned r) { return wide(r) ? 9 + ((dec >> (r < 3 ? 0 : 4)) & 3) : 15 + ((dec >> 8) & 3); }
HD unsigned nbot(unsigned dec, unsigned r) { return wide(r) ? 9 + ((dec >> (r < 3 ? 2 : 6)) & 3) : 32 - ntop(dec, r); }
HD unsigned nrow(unsigned dec, unsigned r) { return place(r) == 0 ? ntop(dec, r) : place(r) == 2 ? nbot(dec, r) : 32 - ntop(dec, r) - nbot(dec, r); }
HD unsigned gband(unsigned a, unsigned r, unsigned dec) {
  if (place(r) != 1) return place(r) ? 31 - a : a;
  const unsigned up = 12 - ntop(dec, r);
  return a < 8 ? 12 + a : a - 8 < up ? 19 - a : 12 + a - up;
}
HD unsigned unit_of(unsigned code, unsigned h, unsigned r, unsigned dec) {
  return code < 32 ? gband(code, r, dec) * 2 + h : 64 + (col0(r) + (code - 32)) * 2 + h;
}
HD unsigned code_at(unsigned i, unsigned r, unsigned tabv) {
  return wide(r) ? (i < 16 ? tabv : i < 21 ? 32 + i - 8 : i - 13) : (i < 12 ? tabv : i - 6);
}
HD unsigned tile_at(unsigned ti, unsigned o, unsigned tabv) {
  if (ti < (wide(o) ? 64u : 36u)) return (tabv >> 3) << 4 | (tabv & 7);
  if (!wide(o)) return (6 + (ti - 36) / 6) << 4 | (ti - 36) % 6;
  return ti < 104 ? (ti & 7) << 4 | (ti >> 3) : (8 + (ti - 104) / 13) << 4 | (ti - 104) % 13;
}
HD unsigned fo_of(unsigned ti, unsigned o) {
  const unsigned e = wide(o) ? 104 : 36;
  return ti < 32 ? ti : ti < e ? 24 + (ti >> 2) : 24 + (e >> 2) + (ti - e) / ncol(o);
}
HD unsigned gsize(unsigned o, unsigned t) { return 2 * (wide(o) ? (wide(t) ? GA : GB) : GC); }
HD unsigned tshare(unsigned o) { return wide(o) ? T0A : T0B; }
HD unsigned nshared(unsigned o) { return wide(o) ? 2 * GB + 5 * GA : 7 * GC; }
HD unsigned gfirst(unsigned o, unsigned t) {
  const unsigned i = t < o ? t : t - 1;
  return tshare(o) + (wide(o) ? (wide(t) ? 2 * GB + GA * i : GB * (t - 6)) : GC * i);
}
HD unsigned head_len(unsigned dec, unsigned r, unsigned o) {
  return N1 + 8 * (nrow(dec, r) - nfix(r)) + 2 * tshare(o);
}
HD unsigned late_len(unsigned dec, unsigned o) { return 2 * (nrow(dec, o) * ncol(o) - tshare(o) - nshared(o)); }

__device__ __forceinline__ unsigned lag_bit(unsigned k) { return 7 * k + (k >= 4 ? 4 : 0); }

__device__ __forceinline__ int share_of(u64 plan, unsigned k, unsigned kk, int tot) {
  const int dk = (int)(plan >> lag_bit(k)) & 127, dq = (int)(plan >> lag_bit(kk)) & 127;
  const int dl = k < kk ? dk - dq : dq - dk;
  int sl = (7 * tot + 14 * 32 - 2 * dl + 2 * (int)((k + kk) % 7) + 1) / 14 - 32;
  sl = sl < 0 ? 0 : sl > tot ? tot : sl;
  return k < kk ? sl : tot - sl;
}
__device__ __forceinline__ int firm(int share) { return share > MRG ? share - MRG : 0; }

template <bool B> struct BoolC { static constexpr bool value = B; };

__global__ void __launch_bounds__(512, 2) fused_gemm_ar(const Params P) {
  __shared__ f32x4 lds[8192];
  const int lane = threadIdx.x & 63;
  const int wave = __builtin_amdgcn_readfirstlane(threadIdx.x >> 6);
  const int bid = blockIdx.x;
  const unsigned rank = P.rank & 7u;
  char* const myws = P.myws;
  char* const myout = P.myout;
  const unsigned task = P.task[bid & 127];
  const unsigned mv = (bid - NGEMM) & 127, k = mv % 7, j0 = mv / 7;
  const char* const pws = P.link[k][0];
  const char* const pout = P.link[k][1];
  asm volatile("" :: "s"(P.x), "s"(P.w), "s"(myws), "s"(myout), "s"(rank), "s"(P.first), "s"(task), "s"(pws), "s"(pout));
  unsigned* const cntp = (unsigned*)(myws + OFF_CNT) + bid * 8 + wave;

  if (bid < NGEMM) {
    unsigned cnt_raw;
    char* peer_ws;
    {
      u32x4* const scr = (u32x4*)lds + wave * 1024;
      if ((task >> 19) & 1) {
        const unsigned u = (task >> 8) & 127u;
        char* const dws = P.link[(task >> 16) & 7u][0];
        stage_piece(P, u, wave, lane, dws, rank * SLICE + u * UNIT, scr, [&]() { cnt_raw = ld_dev(cntp); });
        WAIT_VM(0);
        __builtin_amdgcn_s_barrier();
        if (wave == 0 && lane < 4) st_sys(dws + OFF_FL + (rank * 128 + u) * 128 + lane * 4, rfl(cnt_raw) + 1);
      } else if ((task >> 20) & 1) __builtin_amdgcn_s_sleep(16);
      const unsigned u = task & 127u;
      stage_piece(P, u, wave, lane, myws, rank * SLICE + u * UNIT, scr, [&]() { cnt_raw = ld_dev(cntp); peer_ws = P.link[((rank ^ lane) - 1) & 7][0]; });
      const bool need = lane < 8 ? (task >> (24 + lane)) & 1 : lane < 12;
      char* const fp = lane < 8 ? peer_ws + OFF_FS + (rank * 128 + u) * 128 : myws + OFF_FL + (rank * 128 + u) * 128 + (lane & 3) * 4;
      WAIT_VM(0);
      if (!((task >> 19) & 3) && lane < 8 && need)
        st_sys(myws + OFF_SW + (lane * 128 + u) * 128 + wave * 4, rfl(cnt_raw) + 1);
      __builtin_amdgcn_s_barrier();
      if (wave == 0 && need) st_sys(fp, rfl(cnt_raw) + 1);
    }
    const unsigned call = rfl(cnt_raw) + 1;
    if (lane == 0) st_dev(cntp, call);

    unsigned job = P.job[bid];
#pragma unroll 1
    for (;;) {
    if (!((job >> 18) & 1)) return;
    const unsigned ti_ = job & 255u, ta_ = (job >> 8) & 31u, tb_ = (job >> 13) & 15u;
    const bool wfirst = (job >> 17) & 1;
    unsigned dec = DEC0;
    if (ta_ >= nfix(rank)) { dec = wait_word(OFF_DEC, call, myws); if (ta_ >= nrow(dec, rank)) return; }
    const unsigned grow = gband(ta_, rank, dec), gcol = col0(rank) + tb_;
    const unsigned ue = wfirst ? 64 + 2 * gcol : 2 * grow;
    const unsigned ul = wfirst ? 2 * grow : 64 + 2 * gcol;
    const auto rdat = rsrc(myws + OFF_DATA, NR * SLICE);
    const unsigned rot = (grow >> 2) + (gcol >> 1);
    const unsigned dbase = wave * SLICE + lane * 16;
    auto ld = [&](unsigned u, int f) {
      return __builtin_amdgcn_raw_buffer_load_b128(rdat, dbase + (f & 3) * 1024, u * UNIT + ((((f >> 2) + rot) & 3) << 12), AUX_SYS);
    };
    f32x16 acc[2][2];
    u32x4* const stash = (u32x4*)lds + wave * 1024 + lane;
    bool ok = wait_flags(myws + OFF_FL + (wave * 128 + ue + ((lane >> 2) & 1)) * 128 + (lane & 3) * 4, call, myws, 0xff);
    auto body = [&](auto WF) {
      constexpr bool W1 = decltype(WF)::value;
      u32x4 r1[16], r2[16];
#pragma unroll
      for (int f = 0; f < 16; ++f) r2[f] = ld(ue + 1, f);
#pragma unroll
      for (int f = 0; f < 16; ++f) r1[f] = ld(ue, f);
      __builtin_amdgcn_sched_barrier(0);
#pragma unroll
      for (int f = 0; f < 16; ++f) stash[f * 64] = r2[f];
      if (ok) ok = wait_flags(myws + OFF_FL + (wave * 128 + ul + ((lane >> 2) & 1)) * 128 + (lane & 3) * 4, call, myws, 0xff);
#pragma unroll
      for (int f = 0; f < 16; ++f) r2[f] = ld(ul, f);
      __builtin_amdgcn_sched_barrier(0);
#pragma unroll
      for (int h = 0; h < 2; ++h)
#pragma unroll
        for (int ks = 0; ks < 8; ++ks) {
#pragma unroll
          for (int i = 0; i < 2; ++i)
#pragma unroll
            for (int j = 0; j < 2; ++j) {
              const bf16x8 wv = __builtin_bit_cast(bf16x8, W1 ? r1[ks * 2 + j] : r2[ks * 2 + j]);
              const bf16x8 xv = __builtin_bit_cast(bf16x8, W1 ? r2[ks * 2 + i] : r1[ks * 2 + i]);
              acc[i][j] = __builtin_amdgcn_mfma_f32_32x32x16_bf16(wv, xv, (h | ks) ? acc[i][j] : f32x16{}, 0, 0, 0);
            }
          if (h == 0) {
#pragma unroll
            for (int t = 0; t < 2; ++t) { r2[ks * 2 + t] = ld(ul + 1, ks * 2 + t); r1[ks * 2 + t] = stash[(ks * 2 + t) * 64]; }
            __builtin_amdgcn_sched_barrier(0);
          }
        }
    };
    if (wfirst) body(BoolC<true>{}); else { asm volatile("; row band first" ::: "memory"); body(BoolC<false>{}); }
    const unsigned tm = ld_dev(myws + OFF_TMO);
    if (!ok) {
#pragma unroll
      for (int i = 0; i < 2; ++i)
#pragma unroll
        for (int j = 0; j < 2; ++j)
#pragma unroll
          for (int r = 0; r < 16; ++r) acc[i][j][r] = __builtin_nanf("");
    }
    {
      const unsigned m = lane & 31, hh = lane >> 5;
#pragma unroll
      for (int i = 0; i < 2; ++i)
#pragma unroll
        for (int g = 0; g < 4; ++g)
#pragma unroll
          for (int j = 0; j < 2; ++j)
            lds[wave * 1024 + ((i * 4 + g) * 2 + j) * 64 + hh * 32 + ((m + 4 * (2 * (g & 1) + hh)) & 31)] =
                f32x4{acc[i][j][4 * g], acc[i][j][4 * g + 1], acc[i][j][4 * g + 2], acc[i][j][4 * g + 3]};
    }
    __syncthreads();
    {
      const unsigned row = 8 * wave + (lane >> 3), c = lane & 7, g = c >> 1, hh = c & 1;
      const unsigned slot = (((row >> 5) * 4 + g) * 2) * 64 + hh * 32 + (((row & 31) + 4 * (c & 3)) & 31);
      f32x4 r0[8], r1[8];
#pragma unroll
      for (int w = 0; w < 8; ++w) { r0[w] = lds[w * 1024 + slot]; r1[w] = lds[w * 1024 + slot + 64]; }
      __builtin_amdgcn_sched_barrier(0);
      f32x4 e0 = r0[0], e1 = r1[0];
#pragma unroll
      for (int w = 1; w < 8; ++w) { e0 += r0[w]; e1 += r1[w]; }
      u32x4 o;
#pragma unroll
      for (int i = 0; i < 4; ++i) o[i] = __builtin_bit_cast(unsigned, __builtin_convertvector(f32x2{e0[i], e1[i]}, bf16x2));
      if (tm == call) o = u32x4{NANPK, NANPK, NANPK, NANPK};
      __builtin_amdgcn_raw_buffer_store_b128(o, rsrc(myout, OUT_BYTES), (64 * grow + row) * 4096 + gcol * 128 + c * 16, 0, AUX_SYS);
    }
    WAIT_VM(0);
    __builtin_amdgcn_s_barrier();
    if (wave == 0) {
      bool tell = true;
      const unsigned fo = fo_of(ti_, rank);
      if (ti_ >= 32) {
        const unsigned gs = ti_ < (wide(rank) ? 104u : 36u) ? 4 : ncol(rank);
        unsigned old = 0, off = OFF_FC + (fo - 32) * 128;
        asm volatile("" : "+v"(off));
        if (lane == 0) old = __hip_atomic_fetch_add((unsigned*)(myws + off), 1u, __ATOMIC_RELAXED, __HIP_MEMORY_SCOPE_AGENT);
        tell = rfl(old) % gs == gs - 1;
      }
      if (tell && lane < 8 && (unsigned)lane != rank) st_sys(peer_ws + OFF_FO + (rank * 128 + fo) * 128, call);
    }
    if (ti_ >= 28 || !wide(rank)) return;
    const unsigned t2 = tile_at(128 + ti_, rank, 0);
    job = (128 + ti_) | (t2 >> 4) << 8 | (t2 & 15) << 13 | 3u << 17;
    }
  }

  const u64 t_in = __builtin_amdgcn_s_memrealtime();
  if (bid == NGEMM + NMOVE) {
    if (wave) return;
    char* const peer_ws = P.link[((rank ^ lane) - 1) & 7][0];
    const unsigned call = rfl(ld_dev(cntp)) + 1;
    if (lane == 0) st_dev(cntp, call);
    const unsigned kl = lane < 7 ? lane : 0;
    auto drawn = [&]() { const u64 w = ld_dev64(myws + OFF_LC + kl * 128); return (unsigned)(w >> 32) == call ? (unsigned)w + NW : 0u; };
    auto post = [&](unsigned off, u64 v) {
      __hip_atomic_store((u64*)(myws + OFF_TM2 + lane * 128 + off), v, __ATOMIC_RELAXED, __HIP_MEMORY_SCOPE_AGENT);
      __hip_atomic_store((u64*)(myws + OFF_TM2 + (64 + lane) * 128 + off), v, __ATOMIC_RELAXED, __HIP_MEMORY_SCOPE_AGENT);
    };
    bool alive = true;
    unsigned dec = 0;
    {
      unsigned ts = 0;
      for (;;) {
        const bool go = drawn() >= C0;
        const u64 dt = __builtin_amdgcn_s_memrealtime() - t_in;
        if (go && !ts) ts = dt > 100000 ? 100000u : (unsigned)dt + 1;
        if (__all(ts != 0) || !(alive = spin_ok<2>(t_in, call, myws))) break;
      }
      unsigned sum = 0;
#pragma unroll
      for (int i = 0; i < 7; ++i) sum += rdl(ts, i);
      if (lane < 8) __hip_atomic_store((u64*)(peer_ws + OFF_WV + rank * 128), (u64)call << 32 | sum, __ATOMIC_RELAXED, __HIP_MEMORY_SCOPE_SYSTEM);
      u64 v;
      for (;;) {
        v = __hip_atomic_load((const u64*)(myws + OFF_WV + (lane & 7) * 128), __ATOMIC_RELAXED, __HIP_MEMORY_SCOPE_SYSTEM);
        if (__all((unsigned)(v >> 32) == call) || !(alive = alive && spin_ok<2>(t_in, call, myws))) break;
      }
      {
        const unsigned st = (lane >> 4) > 2 ? 2 : lane >> 4, c = lane & 15, cand = st == 2 ? (c & 3) << 8 : c << (4 * st);
        unsigned worst = 0;
#pragma unroll
        for (int i = 0; i < 3; ++i) {
          const unsigned r = st == 2 ? 6 + (i & 1) : 3 * st + i, n = nrow(cand, r);
          const unsigned cost = pick((unsigned)v, r) + IQ * (56 - 2 * ncol(r)) * n;
          worst = (n < 9 || n > (wide(r) ? 12u : 17u) || (st == 2 && c > 2)) ? 0x3ffffffu : cost > worst ? cost : worst;
        }
        unsigned key = (worst > 0x3ffffffu ? 0x3ffffffu : worst) << 4 | c;
#pragma unroll
        for (int d = 1; d < 16; d <<= 1) { const unsigned o = pick(key, lane ^ d); key = o < key ? o : key; }
        dec = (rdl(key, 0) & 15u) | (rdl(key, 16) & 15u) << 4 | (rdl(key, 32) & 3u) << 8;
      }
      if (!alive) dec = DEC0;
      post(16, call << 16 | dec);
      if (lane == 0) __hip_atomic_store((u64*)(myws + OFF_DEC), (u64)call << 32 | dec, __ATOMIC_RELAXED, __HIP_MEMORY_SCOPE_AGENT);
    }
    const unsigned pl = rank ^ (kl + 1), want = call * gsize(pl, rank);
    unsigned known = head_len(dec, rank, pl), rest = 2 * (late_len(dec, pl) + gsize(pl, rank));
#pragma unroll
    for (int i = 0; i < 7; ++i) if ((unsigned)i != kl) rest += gsize(pl, rank ^ (i + 1)) + gsize(rank ^ (i + 1), pl);
    unsigned todo = 0x1ffu;
    while (todo) {
      const unsigned dr = drawn(), ff = ld_dev(myws + OFF_FF + kl * 128);
      if ((todo & 0x180) && (__any(dr + ((todo & 0x80) ? TSNAP : TSNAP2) >= known) || !alive)) {
        const unsigned left = known + rest / 2 + 512 - dr;
        unsigned mn = 0xffffu;
#pragma unroll
        for (int i = 0; i < 7; ++i) { const unsigned x = rdl(left, i); mn = x < mn ? x : mn; }
        u64 val = (u64)(call & 15u) << 60 | (u64)(call & 15u) << 28;
#pragma unroll
        for (int i = 0; i < 7; ++i) { const unsigned d = rdl(left, i) - mn; val |= (u64)(d > 127 ? 127u : d) << lag_bit(i); }
        post((todo & 0x80) ? 8 : 24, val);
        known += late_len(dec, pl) + gsize(pl, rank); rest = 0;
#pragma unroll
        for (int i = 0; i < 7; ++i)
          if ((unsigned)i != kl) {
            const int tot = gsize(pl, rank ^ (i + 1)) + gsize(rank ^ (i + 1), pl), sh = share_of(val, kl, i, tot);
            known += firm(sh); rest += tot - firm(sh) - firm(tot - sh);
          }
        todo &= (todo & 0x80) ? ~0x80u : ~0x100u;
        if (!alive) continue;
      }
      const unsigned have = (unsigned)__ballot((int)(ff - want) >= 0) & todo & 127u;
#pragma unroll
      for (int i = 0; i < 7; ++i)
        if (((have >> i) & 1) && lane < 8 && (unsigned)lane != rank) st_sys(peer_ws + OFF_FG + ((rank ^ (i + 1)) * 8 + rank) * 128, call);
      todo &= ~have;
      if (!alive) break;
      if (todo) alive = spin_ok<2>(t_in, call, myws);
    }
    return;
  }

  if (wave > NBW) return;
  const unsigned p = (rank ^ (k + 1)) & 7u;
  typedef __attribute__((address_space(3))) volatile u32x4 lds_u32x4;
  typedef __attribute__((address_space(3))) volatile unsigned lds_u32;
  lds_u32x4* const tbl = (lds_u32x4*)lds;
  lds_u32* const box = (lds_u32*)(tbl + TBL);
  auto unit_entry = [&](unsigned u, unsigned q) {
    return u32x4{p * SLICE + u * UNIT + q * 4096, OFF_FS + (p * 128 + u) * 128, OFF_FL + (p * 128 + u) * 128 + q * 4, 1u};
  };

  if (wave) {
    const unsigned tab = P.tab[lane];
    unsigned cnt_raw = 0;
    if (wave == 1) cnt_raw = ld_dev(cntp);
    auto in_entry = [&](unsigned n, unsigned dec) {
      const unsigned ui = NPUSH + (n >> 2);
      return unit_entry(unit_of(code_at(ui >> 1, rank, pick(tab, (ui >> 1) & 63) & 63u), ui & 1, rank, dec) & 127u, n & 3);
    };
    auto tile_entry = [&](unsigned dec, unsigned o, unsigned ti, unsigned half, unsigned fa, unsigned noff) {
      const unsigned t = tile_at(ti, o, (pick(tab, ti & 63) >> 8) & 63u);
      return u32x4{(64 * (gband(t >> 4, o, dec) & 31u) + 32 * half) * 4096 + (col0(o) + (t & 15)) * 128, fa, noff, 2u};
    };
    auto entry = [&](unsigned n, unsigned dec) {
      const unsigned nin = N1 + 8 * (nrow(dec, rank) - nfix(rank)), y = n - nin, ti = (y >> 1) & 255u;
      const u32x4 ie = in_entry(n, dec), te = tile_entry(dec, p, ti, y & 1, OFF_FO + (p * 128 + fo_of(ti, p)) * 128, 0);
      return n < nin ? ie : n < head_len(dec, rank, p) ? te : u32x4{0, 0, 0, 0};
    };
    {
      const unsigned n = threadIdx.x - 64;
      const u32x4 e = in_entry(n, DEC0);
      tbl[n] = n < N1 ? e : u32x4{0, 0, 0, 0};
#pragma unroll
      for (unsigned n0 = NBW * 64; n0 < TBL; n0 += NBW * 64) if (n0 + n < TBL) tbl[n0 + n] = u32x4{0, 0, 0, 0};
    }
    const unsigned call = rfl(cnt_raw) + 1;
    if (wave == 1 && lane < 8) box[lane] = lane == 4 ? (call + 1) << 16 : (call + 1) << 28;
    if (wave == 1 && lane == 0) st_dev(cntp, call);
    asm volatile("s_waitcnt lgkmcnt(0)\n\ts_barrier" ::: "memory");
    if (wave > 1) return;

    typedef __attribute__((address_space(3))) volatile u64 lds_u64;
    unsigned dec;
    while (((dec = rfl(box[4])) ^ call << 16) >> 16) if (!spin_ok<4>(t_in, call, myws)) { dec = DEC0; break; }
    unsigned la = head_len(dec, rank, p);
#pragma unroll 1
    for (unsigned n = N1; n < la; n += 64) tbl[n + lane] = entry(n + lane, dec);
    const unsigned l8 = lane < 9 ? lane & 7 : (lane - 9) & 7, q = wide(p) ? (l8 + 6) & 7 : l8, kk = ((rank ^ q) - 1) & 7u;
    const bool pair = q != p && q != rank;
    const int gd = gsize(p, q), tot = gd + gsize(q, p);
    int beg = 0, open = tot;
#pragma unroll 1
    for (int deal = 0; deal < 2; ++deal) {
      u64 plan;
      while ((((plan = rfl64(*(lds_u64*)(box + 2 + 4 * deal))) >> 28) & 0xf0000000full) != (call & 15u) * 0x100000001ull)
        if (!spin_ok<2>(t_in, call, myws)) { plan = 0; break; }
      const int sh = share_of(plan, k, kk, open), end = deal ? beg + sh : firm(sh);
      const int b = lane < 8 ? (beg < gd ? beg : gd) : (beg > gd ? beg : gd), e = lane < 8 ? (end < gd ? end : gd) : (end > gd ? end : gd);
      const unsigned len = lane == 8 ? (deal ? 0u : late_len(dec, p)) : lane > 16 ? 0u : pair ? e - b : lane < 8 && q == rank && !deal ? gd : 0u;
      const unsigned org = lane == 8 ? 2 * (tshare(p) + nshared(p)) : lane < 8 ? 2 * gfirst(p, q) + b : 2 * gfirst(q, p) + (tot - 1 - b);
      unsigned total = 0;
#pragma unroll
      for (int i = 0; i < 17; ++i) total += rdl(len, i);
#pragma unroll 1
      for (unsigned e0 = 0; e0 < total + (deal ? NW : 0); e0 += 64) {
        unsigned l = 0, sub = e0 + lane;
#pragma unroll
        for (int i = 0; i < 17; ++i) { const unsigned x = rdl(len, i); if (l == (unsigned)i && sub >= x) { sub -= x; ++l; } }
        const unsigned ql = pick(q, l), og = pick(org, l), h = l < 9 ? og + sub : og - sub, ti = (h >> 1) & 255u;
        const u32x4 te = tile_entry(dec, l < 9 ? p : ql, ti, h & 1, l < 9 ? OFF_FO + (p * 128 + fo_of(ti, p)) * 128 : OFF_FG + (ql * 8 + p) * 128,
                                    l < 8 && ql == rank ? (OFF_FF + k * 128) | 1u : 0u);
        const unsigned at = la + e0 + lane;
        tbl[at < TBL ? at : TBL - 1] = l < 17 ? te : u32x4{0, OFF_TMO, 0, deal ? 3u : 0u};
      }
      la += total; beg = end; open = tot - end - firm(tot - sh);
    }
    return;
  }

  const unsigned u1 = (P.first >> (8 * (j0 >> 2))) & 127u;
  const unsigned cnt_raw = ld_dev(cntp), fs1 = ld_sys(myws + OFF_FS + (p * 128 + u1) * 128);
  const unsigned call = rfl(cnt_raw) + 1;
  asm volatile("" :: "s"(call) : "memory");
  const u64 t_ld = __builtin_amdgcn_s_memrealtime();
  if (lane == 0) {
    st_dev(cntp, call);
    __hip_atomic_fetch_max((u64*)(myws + OFF_LC + k * 128), (u64)call << 32, __ATOMIC_RELAXED, __HIP_MEMORY_SCOPE_AGENT);
  }
  auto desc = [](const void* b, unsigned bytes) { const u64 a = (u64)b; return u32x4{rfl((unsigned)a), rfl((unsigned)(a >> 32) & 0xffffu), rfl(bytes), 0x00020000u}; };
  const unsigned vcall = call, vone = 1, vlc = OFF_LC + k * 128;
  const unsigned lo_lin = lane * 16, lo_tile = (lane >> 3) * 4096 + (lane & 7) * 16;
  const unsigned tm_l = OFF_TM2 + mv * 128 + ((lane & 7) >= 2 ? (lane & 7) * 4 : 0);
  u32x4 w0 = {}, w1 = {}, w2 = {}, w3 = {};
  unsigned o0, o1, o2, o3, r0 = 0, r1 = 0, r2 = 0, r3 = 0;
  bool ok = true;
  unsigned kind, noff;
  unsigned nfo;
  auto take = [&](const u32x4& e) {
    kind = rfl(e[3]); noff = rfl(e[2]);
    const unsigned ts = kind == 2 ? 8 * 4096 : 1024;
    o0 = e[0] + (kind == 2 ? lo_tile : lo_lin); o1 = o0 + ts; o2 = o1 + ts; o3 = o2 + ts;
    nfo = (lane & 7) ? tm_l : e[1];
  };
  auto fetch = [&](unsigned idx) {
    u32x4 e = tbl[idx < TBL ? idx : TBL - 1];
    while (rfl(e[3]) == 0) {
      const unsigned x = ld_sys(myws + tm_l);
      box[lane & 7] = x;
      if (!spin_ok<2>(t_in, call, myws)) { e = u32x4{0, OFF_TMO, 0, 3u}; ok = false; break; }
      e = tbl[idx < TBL ? idx : TBL - 1];
    }
    take(e);
  };
  auto put = [&](unsigned kd) {
    const u32x4 rd = desc(kd == 2 ? myout : myws + OFF_DATA, kd == 2 ? OUT_BYTES : NR * SLICE);
    asm volatile("s_nop 4\n\tbuffer_store_dwordx4 %0, %4, %8, 0 offen sc0 sc1\n\tbuffer_store_dwordx4 %1, %5, %8, 0 offen sc0 sc1\n\t"
                 "buffer_store_dwordx4 %2, %6, %8, 0 offen sc0 sc1\n\tbuffer_store_dwordx4 %3, %7, %8, 0 offen sc0 sc1\n\ts_nop 1"
                 :: "v"(w0), "v"(w1), "v"(w2), "v"(w3), "v"(r0), "v"(r1), "v"(r2), "v"(r3), "s"(rd) : "memory");
  };
  auto notify = [&](unsigned off) {
    if (off & 1) asm volatile("s_mov_b64 exec, 1\n\tglobal_atomic_add %0, %1, %2\n\ts_mov_b64 exec, -1" :: "v"(off - 1), "v"(vone), "s"(myws) : "memory");
    else asm volatile("s_mov_b64 exec, 1\n\tglobal_store_dword %0, %1, %2 sc0 sc1\n\ts_mov_b64 exec, -1" :: "v"(off), "v"(vcall), "s"(myws) : "memory");
  };

  take(unit_entry(u1, j0 & 3));
  asm volatile("" : "+v"(o0), "+v"(o1), "+v"(o2), "+v"(o3), "+v"(w0), "+v"(w1), "+v"(w2), "+v"(w3));
  {
    u64 bad;
    {
      const unsigned ro = OFF_SW + (rank * 128 + u1) * 128 + (j0 & 3) * 8 + (lane & 1) * 4;
      unsigned left = rfl(fs1) >= call ? 0u : 64u;
      asm volatile(
          "s_mov_b64 %0, 0\n\ts_cmp_eq_u32 %4, 0\n\ts_cbranch_scc1 2f\n\t"
          "global_load_dword %1, %5, %6 sc0 sc1\n\ts_sleep 6\n\t"
          "global_load_dword %2, %5, %6 sc0 sc1\n\ts_sleep 6\n"
          "0:\n\t"
          "global_load_dword %3, %5, %6 sc0 sc1\n\t"
          "s_waitcnt vmcnt(2)\n\tv_cmp_gt_u32 %0, %7, %1\n\ts_cmp_eq_u64 %0, 0\n\ts_cbranch_scc1 1f\n\tglobal_load_dword %1, %5, %6 sc0 sc1\n\t"
          "s_waitcnt vmcnt(2)\n\tv_cmp_gt_u32 %0, %7, %2\n\ts_cmp_eq_u64 %0, 0\n\ts_cbranch_scc1 1f\n\tglobal_load_dword %2, %5, %6 sc0 sc1\n\t"
          "s_waitcnt vmcnt(2)\n\tv_cmp_gt_u32 %0, %7, %3\n\ts_cmp_eq_u64 %0, 0\n\ts_cbranch_scc1 1f\n\t"
          "s_sub_u32 %4, %4, 1\n\ts_cmp_lg_u32 %4, 0\n\ts_cbranch_scc1 0b\n"
          "2:\n\ts_waitcnt vmcnt(0)\n"
          "1:"
          : "=&s"(bad), "=&v"(r1), "=&v"(r2), "=&v"(r3), "+s"(left) : "v"(ro), "s"(pws), "s"(call) : "memory", "scc");
    }
    if (bad) ok = wait_flags(myws + nfo, call, myws, 1);
    while (__builtin_amdgcn_s_memrealtime() - t_ld < TG) __builtin_amdgcn_s_sleep(1);
  }
  bool joined = false;
  unsigned skind = 0, snoff = 0, pnoff = 0;
  for (;;) {
    u32x4 v0, v1, v2, v3;
    unsigned f, tk;
    asm volatile("s_mov_b64 exec, 1\n\tglobal_atomic_add %0, %1, %2, %3 sc0\n\ts_mov_b64 exec, -1" : "=&v"(tk) : "v"(vlc), "v"(vone), "s"(myws) : "memory");
    const bool first = snoff != 0;
    if (first) { put(skind); pnoff = snoff; }
    {
      const u32x4 rs = desc(kind == 2 ? pout : pws + OFF_DATA, ok ? (kind == 2 ? OUT_BYTES : NR * SLICE) : 0u);
      asm volatile("s_nop 4\n\tbuffer_load_dwordx4 %0, %4, %8, 0 offen sc0 sc1\n\tbuffer_load_dwordx4 %1, %5, %8, 0 offen sc0 sc1\n\t"
                   "buffer_load_dwordx4 %2, %6, %8, 0 offen sc0 sc1\n\tbuffer_load_dwordx4 %3, %7, %8, 0 offen sc0 sc1"
                   : "=&v"(v0), "=&v"(v1), "=&v"(v2), "=&v"(v3) : "v"(o0), "v"(o1), "v"(o2), "v"(o3), "s"(rs) : "memory");
    }
    if (!first && skind) put(skind);
    const unsigned ckind = kind, cnoff = noff, q0 = o0, q1 = o1, q2 = o2, q3 = o3;
    if (!joined) { __builtin_amdgcn_s_barrier(); joined = true; }
    if (!skind) WAIT_VM(4);
    asm volatile("s_waitcnt vmcnt(8)" : "+v"(tk) : "v"(r1), "v"(r2), "v"(r3) : "memory");
    fetch(rfl(tk) + NW);
    asm volatile("global_load_dword %0, %1, %2 sc0 sc1" : "=&v"(f) : "v"(nfo), "s"(myws) : "memory");
    asm volatile("" : "+v"(o0), "+v"(o1), "+v"(o2), "+v"(o3));
    if (pnoff) { WAIT_VM(5); notify(pnoff); pnoff = 0; WAIT_VM(1); } else WAIT_VM(0);
    asm volatile("" : "+v"(v0), "+v"(v1), "+v"(v2), "+v"(v3), "+v"(f) :: "memory");
    if (rdl(f, 1) == call) ok = false;
    w0 = v0; w1 = v1; w2 = v2; w3 = v3;
    if (!ok) w0 = w1 = w2 = w3 = u32x4{NANPK, NANPK, NANPK, NANPK};
    r0 = q0; r1 = q1; r2 = q2; r3 = q3; skind = ckind; snoff = cnoff;
    box[lane & 7] = f;
    if (kind == 3 || rfl(f) < call) {
      if (kind == 3 || rfl(ld_sys(myws + nfo)) < call) {
        put(skind);
        if (snoff) { WAIT_VM(0); notify(snoff); }
        skind = 0; snoff = 0;
        if (kind == 3) break;
        if (ok) ok = wait_flags(myws + nfo, call, myws, 1);
      }
    }
  }
  if (!ok && lane == 0) st_sys(myout + gband(0, rank, DEC0) * 64 * 4096 + col0(rank) * 128, NANPK);
}

}

extern "C" void kernel_launch(int rank, int nranks, ncclComm_t comm, hipStream_t stream,
                              void* const* d_in, const int* in_sizes, int n_in,
                              void* d_out, int out_size, const int* in_dtypes, int out_dtype,
                              void* const* d_out_all, void* const* d_ws_all, size_t ws_bytes) {
  if (nranks != NR || (unsigned)rank >= (unsigned)NR || n_in < 2 || ws_bytes < WS_NEED || out_size != 2048 * 2048 ||
      in_sizes[0] != 2048 * 256 || in_sizes[1] != 256 * 2048) { fprintf(stderr, "unexpected problem shape\n"); return; }
  Params P;
  P.x = (const char*)d_in[0]; P.w = (const char*)d_in[1];
  for (int k = 0; k < 8; ++k) { const int p = (rank ^ (k + 1)) & 7; P.link[k][0] = (char*)d_ws_all[p]; P.link[k][1] = (char*)d_out_all[p]; }
  P.myws = (char*)d_ws_all[rank]; P.myout = (char*)d_out_all[rank];
  P.rank = rank; P.first = 0;
  const unsigned char bo[16] = {32, 0, 33, 1, 2, 34, 3, 35, 4, 36, 5, 37, 6, 38, 7, 39};
  unsigned char ord[64]; int no = 0, na = 0, nb = 0;
  for (int n = 0; n < 16; ++n) {
    if (bo[n] < 32) { for (int b = 0; b < nb; ++b) ord[no++] = na * 8 + b; ++na; }
    else { for (int a = 0; a < na; ++a) ord[no++] = a * 8 + nb; ++nb; }
  }
  for (unsigned i = 0; i < 4; ++i) P.first |= unit_of(bo[(NPUSH + i) >> 1], (NPUSH + i) & 1, rank, DEC0) << (8 * i);
  for (int i = 0; i < 64; ++i) P.tab[i] = bo[i & 15] | (unsigned)ord[i] << 8;
  {
    int posn[64];
    for (unsigned i = 0; i < 25; ++i) posn[code_at(i, rank, bo[i & 15]) & 63] = i;
    const unsigned nt = wide(rank) ? 128 : 102;
    int used[8] = {0}; bool done[128] = {false};
    for (int b = 0; b < 128; ++b) P.job[b] = 0;
    for (int pass = 0; pass < 2; ++pass)
      for (unsigned ti = 0; ti < nt; ++ti) {
        const unsigned t = tile_at(ti, rank, ord[ti & 63]), a = t >> 4, b = t & 15;
        int x = (a & 3) * 2 + (b & 1);
        if (done[ti] || (pass == 0 && used[x] == 16)) continue;
        while (used[x] == 16) x = (x + 1) & 7;
        P.job[used[x]++ * 8 + x] = ti | a << 8 | b << 13 | (posn[32 + b] < posn[a] ? 1u : 0u) << 17 | 1u << 18;
        done[ti] = true;
      }
  }
  unsigned mask[128] = {0}; int first[128];
  for (int u = 0; u < 128; ++u) first[u] = 99;
  for (unsigned r = 0; r < NR; ++r)
    for (unsigned dec = 0; dec < 768; ++dec) {
      const unsigned n = nrow(dec, r);
      if (n < 9 || n > (wide(r) ? 12u : 17u)) continue;
      for (unsigned i = 0; i < 2 * (n + ncol(r)); ++i) {
        const unsigned u = unit_of(code_at(i >> 1, r, bo[(i >> 1) & 15]), i & 1, r, dec);
        if (r != (unsigned)rank && i >= NPUSH) { mask[u] |= 1u << r; if ((int)i < first[u]) first[u] = i; }
      }
    }
  int us[128], nu = 0;
  for (int u = 0; u < 128; ++u) if (first[u] == 99) first[u] = 20;
  for (int pos = 98; pos >= 0; --pos) for (int u = 0; u < 128; ++u) if (first[u] == pos) us[nu++] = u;
  int i0 = 0, i1 = 16; unsigned np = 0;
  for (int bb = 0; bb < 128; ++bb) {
    const int b = (bb + 96) & 127;
    const int u = us[(b & 7) == 0 ? i0++ : i1++];
    P.task[b] = u | mask[u] << 24 | (first[u] < (int)(NPUSH + NCRIT) ? 0u : 1u << 20);
    if ((b & 7) != 0 && np < 7 * NPUSH) {
      const unsigned dst = rank ^ ((np + np / 7) % 7 + 1), j = np / 7;
      P.task[b] = u | mask[u] << 24 | unit_of(bo[j >> 1], j & 1, dst, DEC0) << 8 | ((rank ^ dst) - 1) << 16 | 1u << 19; ++np;
    }
  }
  hipLaunchKernelGGL(fused_gemm_ar, dim3(NGEMM + NMOVE + 1), dim3(512), 0, stream, P);
}
```

```cpp
#include <hip/hip_runtime.h>
#include <rccl/rccl.h>
#include <cstdio>
#include <cstdint>

namespace {

typedef unsigned u32x4 __attribute__((ext_vector_type(4)));
typedef float f32x4 __attribute__((ext_vector_type(4)));
typedef float f32x16 __attribute__((ext_vector_type(16)));
typedef __bf16 bf16x8 __attribute__((ext_vector_type(8)));
typedef __bf16 bf16x2 __attribute__((ext_vector_type(2)));
typedef float f32x2 __attribute__((ext_vector_type(2)));
typedef unsigned long long u64;

constexpr unsigned NW = 16;
constexpr unsigned NPUSH = 4;
constexpr unsigned NCRIT = 8;
constexpr unsigned TG = 150;
constexpr unsigned C0 = 2 * NW + NW / 2;
constexpr unsigned IQ = 8;
constexpr unsigned T0A = 44, T0B = 48;
constexpr unsigned GA = 6, GB = 11, GC = 3;
constexpr int MRG = 2;
constexpr unsigned TSNAP = 48, TSNAP2 = 52;
constexpr int NBW = 4;

constexpr int NR = 8;
constexpr unsigned UNIT = 16384;
constexpr unsigned SLICE = 128 * UNIT;
constexpr unsigned OUT_BYTES = 2048u * 2048u * 2u;
constexpr unsigned IN_BYTES = 2048u * 256u * 2u;
constexpr unsigned KB = 1024;
constexpr unsigned OFF_CNT = 0;
constexpr unsigned OFF_TMO = 16 * KB;
constexpr unsigned OFF_LC = 17 * KB;
constexpr unsigned OFF_DEC = 18 * KB + 256;
constexpr unsigned OFF_WV = 19 * KB;
constexpr unsigned OFF_FS = 32 * KB;
constexpr unsigned OFF_FL = 160 * KB;
constexpr unsigned OFF_FO = 288 * KB;
constexpr unsigned OFF_FC = 416 * KB;
constexpr unsigned OFF_FF = 424 * KB;
constexpr unsigned OFF_FG = 432 * KB;
constexpr unsigned OFF_TM2 = 448 * KB;
constexpr unsigned OFF_SW = 512 * KB;
constexpr unsigned OFF_DATA = 2048 * KB;
constexpr size_t WS_NEED = OFF_DATA + (size_t)NR * SLICE;
constexpr int NGEMM = 128, NMOVE = 7 * NW;
constexpr unsigned N1 = (2 * 21 - NPUSH) * 4;
constexpr unsigned DEC0 = 2 | 2 << 2 | 2 << 4 | 2 << 6 | 1 << 8;
constexpr unsigned TBL = 704;
constexpr u64 DEADLINE = 100000000ull;
constexpr int AUX_SYS = 17;
constexpr unsigned NANPK = 0x7fc07fc0u;
static_assert(NW <= 16 && NGEMM + NMOVE + 1 <= 256 && NMOVE <= 128 && N1 + 32 + 2 * 156 + 12 * GB + 64 < TBL && T0A + 2 * GB + 5 * GA <= 104 && T0B + 7 * GC <= 90, "layout");


struct Params {
  const char* x; const char* w;
  char* myws; char* myout;
  unsigned rank, first;
  char* link[8][2];
  unsigned task[128];
  unsigned job[128];
  unsigned tab[64];
};

__device__ __forceinline__ unsigned ld_sys(const void* p) { return __hip_atomic_load((const unsigned*)p, __ATOMIC_RELAXED, __HIP_MEMORY_SCOPE_SYSTEM); }
__device__ __forceinline__ unsigned ld_dev(const void* p) { return __hip_atomic_load((const unsigned*)p, __ATOMIC_RELAXED, __HIP_MEMORY_SCOPE_AGENT); }
__device__ __forceinline__ u64 ld_dev64(const void* p) { return __hip_atomic_load((const u64*)p, __ATOMIC_RELAXED, __HIP_MEMORY_SCOPE_AGENT); }
__device__ __forceinline__ void st_sys(void* p, unsigned v) { __hip_atomic_store((unsigned*)p, v, __ATOMIC_RELAXED, __HIP_MEMORY_SCOPE_SYSTEM); }
__device__ __forceinline__ void st_dev(void* p, unsigned v) { __hip_atomic_store((unsigned*)p, v, __ATOMIC_RELAXED, __HIP_MEMORY_SCOPE_AGENT); }
__device__ __forceinline__ unsigned rfl(unsigned v) { return __builtin_amdgcn_readfirstlane(v); }
__device__ __forceinline__ u64 rfl64(u64 v) { return (u64)rfl((unsigned)(v >> 32)) << 32 | rfl((unsigned)v); }
__device__ __forceinline__ unsigned rdl(unsigned v, unsigned l) { return __builtin_amdgcn_readlane(v, l); }
__device__ __forceinline__ unsigned pick(unsigned v, unsigned l) { return __builtin_amdgcn_ds_bpermute(l * 4, v); }
#define WAIT_VM(n) asm volatile("s_waitcnt vmcnt(" #n ")" ::: "memory")

template <int NAP>
__device__ __forceinline__ bool spin_ok(u64 t0, unsigned call, char* myws) {
  const u64 dt = __builtin_amdgcn_s_memrealtime() - t0;
  if (dt > 10000ull) {
    if (rfl(ld_dev(myws + OFF_TMO)) == call || dt > DEADLINE) {
      char* const c = myws + OFF_TM2 + (threadIdx.x & 63) * 128;
      st_dev(c, call); st_dev(c + 64 * 128, call); st_dev(myws + OFF_TMO, call);
      return false;
    }
    __builtin_amdgcn_s_sleep(32);
  }
  __builtin_amdgcn_s_sleep(NAP);
  return true;
}
__device__ __forceinline__ bool wait_flags(const void* p, unsigned call, char* myws, u64 mask) {
  const u64 t0 = __builtin_amdgcn_s_memrealtime();
  while ((__ballot(ld_sys(p) >= call) & mask) != mask)
    if (!spin_ok<1>(t0, call, myws)) return false;
  __builtin_amdgcn_fence(__ATOMIC_ACQUIRE, "wavefront");
  return true;
}
__device__ __forceinline__ unsigned wait_word(unsigned off, unsigned call, char* myws) {
  const u64 t0 = __builtin_amdgcn_s_memrealtime();
  for (;;) {
    const u64 v = rfl64(ld_dev64(myws + off));
    if ((unsigned)(v >> 32) == call) return (unsigned)v;
    if (!spin_ok<8>(t0, call, myws)) return DEC0;
  }
}

__device__ __forceinline__ __amdgpu_buffer_rsrc_t rsrc(const void* base, unsigned bytes) {
  return __builtin_amdgcn_make_buffer_rsrc((void*)base, 0, bytes, 0x00020000);
}

template <class F>
__device__ __forceinline__ void stage_piece(const char* px, const char* pw, unsigned u, unsigned e, unsigned lane, char* base, unsigned ub, u32x4* sc, F&& after_loads) {
  const unsigned h = u & 1;
  const bool isx = u < 64;
  const unsigned o = isx ? (64 * (u >> 1) + 32 * (e & 1) + (lane >> 2)) * 512 + h * 256 + (e >> 1) * 64 + (lane & 3) * 16
                         : (128 * h + 16 * e + (lane >> 3)) * 4096 + ((u - 64) >> 1) * 128 + (lane & 7) * 16;
  const auto rs = rsrc(isx ? px : pw, IN_BYTES);
  u32x4 a = __builtin_amdgcn_raw_buffer_load_b128(rs, o, 0, 0);
  u32x4 b = __builtin_amdgcn_raw_buffer_load_b128(rs, o + (isx ? 16 * 512 : 8 * 4096), 0, 0);
  __builtin_amdgcn_sched_barrier(0);
  after_loads();
  __builtin_amdgcn_sched_barrier(0);
  unsigned oa, ob;
  if (isx) {
    const unsigned wi = (lane & 3) * 33 + (lane >> 2);
    sc[wi] = a; sc[wi + 16] = b;
    asm volatile("" ::: "memory");
    const unsigned ri = (lane >> 5) * 33 + (lane & 31);
    a = sc[ri]; b = sc[ri + 66];
    oa = ub + ((e & 6) * 2 + (e & 1)) * 1024 + lane * 16; ob = oa + 2048;
  } else {
    sc[lane] = a; sc[lane + 64] = b;
    asm volatile("" ::: "memory");
    const unsigned* const sd = (const unsigned*)sc + (lane >> 5) * 256 + (lane & 31);
    unsigned d[8];
#pragma unroll
    for (int j = 0; j < 8; ++j) d[j] = sd[j * 32];
#pragma unroll
    for (int j = 0; j < 4; ++j) {
      a[j] = __builtin_amdgcn_perm(d[2 * j + 1], d[2 * j], 0x05040100u);
      b[j] = __builtin_amdgcn_perm(d[2 * j + 1], d[2 * j], 0x07060302u);
    }
    oa = ub + e * 2048 + lane * 16; ob = oa + 1024;
  }
  const auto rd = rsrc(base + OFF_DATA, NR * SLICE);
  __builtin_amdgcn_raw_buffer_store_b128(a, rd, oa, 0, AUX_SYS);
  __builtin_amdgcn_raw_buffer_store_b128(b, rd, ob, 0, AUX_SYS);
}
#define HD __host__ __device__ __forceinline__
HD bool wide(unsigned r) { return r < 6; }
HD unsigned col0(unsigned r) { return r < 3 ? 0 : r < 6 ? 13 : 26; }
HD unsigned ncol(unsigned r) { return wide(r) ? 13 : 6; }
HD unsigned nfix(unsigned r) { return wide(r) ? 8 : 15; }
HD unsigned place(unsigned r) { return r < 3 ? r : r < 6 ? r - 3 : 2 * (r - 6); }
HD unsigned ntop(unsigned dec, unsigned r) { return wide(r) ? 9 + ((dec >> (r < 3 ? 0 : 4)) & 3) : 15 + ((dec >> 8) & 3); }
HD unsigned nbot(unsigned dec, unsigned r) { return wide(r) ? 9 + ((dec >> (r < 3 ? 2 : 6)) & 3) : 32 - ntop(dec, r); }
HD unsigned nrow(unsigned dec, unsigned r) { return place(r) == 0 ? ntop(dec, r) : place(r) == 2 ? nbot(dec, r) : 32 - ntop(dec, r) - nbot(dec, r); }
HD unsigned gband(unsigned a, unsigned r, unsigned dec) {
  if (place(r) != 1) return place(r) ? 31 - a : a;
  const unsigned up = 12 - ntop(dec, r);
  return a < 8 ? 12 + a : a - 8 < up ? 19 - a : 12 + a - up;
}
HD unsigned unit_of(unsigned code, unsigned h, unsigned r, unsigned dec) {
  return code < 32 ? gband(code, r, dec) * 2 + h : 64 + (col0(r) + (code - 32)) * 2 + h;
}
HD unsigned code_at(unsigned i, unsigned r, unsigned tabv) {
  return wide(r) ? (i < 16 ? tabv : i < 21 ? 32 + i - 8 : i - 13) : (i < 12 ? tabv : i - 6);
}
HD unsigned tile_at(unsigned ti, unsigned o, unsigned tabv) {
  if (ti < (wide(o) ? 64u : 36u)) return (tabv >> 3) << 4 | (tabv & 7);
  if (!wide(o)) return (6 + (ti - 36) / 6) << 4 | (ti - 36) % 6;
  return ti < 104 ? (ti & 7) << 4 | (ti >> 3) : (8 + (ti - 104) / 13) << 4 | (ti - 104) % 13;
}
HD unsigned fo_of(unsigned ti, unsigned o) {
  const unsigned e = wide(o) ? 104 : 36;
  return ti < 32 ? ti : ti < e ? 24 + (ti >> 2) : 24 + (e >> 2) + (ti - e) / ncol(o);
}
HD unsigned gsize(unsigned o, unsigned t) { return 2 * (wide(o) ? (wide(t) ? GA : GB) : GC); }
HD unsigned tshare(unsigned o) { return wide(o) ? T0A : T0B; }
HD unsigned nshared(unsigned o) { return wide(o) ? 2 * GB + 5 * GA : 7 * GC; }
HD unsigned gfirst(unsigned o, unsigned t) {
  const unsigned i = t < o ? t : t - 1;
  return tshare(o) + (wide(o) ? (wide(t) ? 2 * GB + GA * i : GB * (t - 6)) : GC * i);
}
HD unsigned head_len(unsigned dec, unsigned r, unsigned o) {
  return N1 + 8 * (nrow(dec, r) - nfix(r)) + 2 * tshare(o);
}
HD unsigned late_len(unsigned dec, unsigned o) { return 2 * (nrow(dec, o) * ncol(o) - tshare(o) - nshared(o)); }

__device__ __forceinline__ unsigned lag_bit(unsigned k) { return 7 * k + (k >= 4 ? 4 : 0); }

__device__ __forceinline__ int share_of(u64 plan, unsigned k, unsigned kk, int tot) {
  const int dk = (int)(plan >> lag_bit(k)) & 127, dq = (int)(plan >> lag_bit(kk)) & 127;
  const int dl = k < kk ? dk - dq : dq - dk;
  int sl = (7 * tot + 14 * 32 - 2 * dl + 2 * (int)((k + kk) % 7) + 1) / 14 - 32;
  sl = sl < 0 ? 0 : sl > tot ? tot : sl;
  return k < kk ? sl : tot - sl;
}
__device__ __forceinline__ int firm(int share) { return share > MRG ? share - MRG : 0; }

template <bool B> struct BoolC { static constexpr bool value = B; };

__global__ void __launch_bounds__(512, 2) fused_gemm_ar(const Params P) {
  __shared__ f32x4 lds[8192];
  const int lane = threadIdx.x & 63;
  const int wave = __builtin_amdgcn_readfirstlane(threadIdx.x >> 6);
  const int bid = blockIdx.x;
  const unsigned rank = P.rank & 7u;
  char* const myws = P.myws;
  char* const myout = P.myout;
  const unsigned task = P.task[bid & 127];
  const unsigned mv = (bid - NGEMM) & 127, k = mv % 7, j0 = mv / 7;
  const char* const pws = P.link[k][0];
  const char* const pout = P.link[k][1];
  const char* px = P.x; const char* pw = P.w;
  asm volatile("" : "+s"(px), "+s"(pw) : "s"(myws), "s"(myout), "s"(rank), "s"(P.first), "s"(task), "s"(pws), "s"(pout));
  unsigned* const cntp = (unsigned*)(myws + OFF_CNT) + bid * 8 + wave;

  if (__builtin_expect(bid < NGEMM, 1)) {
    unsigned cnt_raw;
    char* peer_ws;
    {
      u32x4* const scr = (u32x4*)lds + wave * 1024;
      if ((task >> 19) & 1) {
        const unsigned u = (task >> 8) & 127u;
        char* const dws = P.link[(task >> 16) & 7u][0];
        stage_piece(px, pw, u, wave, lane, dws, rank * SLICE + u * UNIT, scr, [&]() { cnt_raw = ld_dev(cntp); });
        WAIT_VM(0);
        __builtin_amdgcn_s_barrier();
        if (wave == 0 && lane < 4) st_sys(dws + OFF_FL + (rank * 128 + u) * 128 + lane * 4, rfl(cnt_raw) + 1);
      } else if ((task >> 20) & 1) __builtin_amdgcn_s_sleep(16);
      const unsigned u = task & 127u;
      stage_piece(px, pw, u, wave, lane, myws, rank * SLICE + u * UNIT, scr, [&]() { cnt_raw = ld_dev(cntp); peer_ws = P.link[((rank ^ lane) - 1) & 7][0]; });
      const bool need = lane < 8 ? (task >> (24 + lane)) & 1 : lane < 12;
      char* const fp = lane < 8 ? peer_ws + OFF_FS + (rank * 128 + u) * 128 : myws + OFF_FL + (rank * 128 + u) * 128 + (lane & 3) * 4;
      WAIT_VM(0);
      if (!((task >> 19) & 3) && lane < 8 && need)
        st_sys(myws + OFF_SW + (lane * 128 + u) * 128 + wave * 4, rfl(cnt_raw) + 1);
      __builtin_amdgcn_s_barrier();
      if (wave == 0 && need) st_sys(fp, rfl(cnt_raw) + 1);
    }
    const unsigned call = rfl(cnt_raw) + 1;
    if (lane == 0) st_dev(cntp, call);

    unsigned job = P.job[bid];
#pragma unroll 1
    for (;;) {
    if (!((job >> 18) & 1)) return;
    const unsigned ti_ = job & 255u, ta_ = (job >> 8) & 31u, tb_ = (job >> 13) & 15u;
    const bool wfirst = (job >> 17) & 1;
    unsigned dec = DEC0;
    if (ta_ >= nfix(rank)) { dec = wait_word(OFF_DEC, call, myws); if (ta_ >= nrow(dec, rank)) return; }
    const unsigned grow = gband(ta_, rank, dec), gcol = col0(rank) + tb_;
    const unsigned ue = wfirst ? 64 + 2 * gcol : 2 * grow;
    const unsigned ul = wfirst ? 2 * grow : 64 + 2 * gcol;
    const auto rdat = rsrc(myws + OFF_DATA, NR * SLICE);
    const unsigned rot = (grow >> 2) + (gcol >> 1);
    const unsigned dbase = wave * SLICE + lane * 16;
    auto ld = [&](unsigned u, int f) {
      return __builtin_amdgcn_raw_buffer_load_b128(rdat, dbase + (f & 3) * 1024, u * UNIT + ((((f >> 2) + rot) & 3) << 12), AUX_SYS);
    };
    f32x16 acc[2][2];
    u32x4* const stash = (u32x4*)lds + wave * 1024 + lane;
    bool ok = wait_flags(myws + OFF_FL + (wave * 128 + ue + ((lane >> 2) & 1)) * 128 + (lane & 3) * 4, call, myws, 0xff);
    auto body = [&](auto WF) {
      constexpr bool W1 = decltype(WF)::value;
      u32x4 r1[16], r2[16];
#pragma unroll
      for (int f = 0; f < 16; ++f) r2[f] = ld(ue + 1, f);
#pragma unroll
      for (int f = 0; f < 16; ++f) r1[f] = ld(ue, f);
      __builtin_amdgcn_sched_barrier(0);
#pragma unroll
      for (int f = 0; f < 16; ++f) stash[f * 64] = r2[f];
      if (ok) ok = wait_flags(myws + OFF_FL + (wave * 128 + ul + ((lane >> 2) & 1)) * 128 + (lane & 3) * 4, call, myws, 0xff);
#pragma unroll
      for (int f = 0; f < 16; ++f) r2[f] = ld(ul, f);
      __builtin_amdgcn_sched_barrier(0);
#pragma unroll
      for (int h = 0; h < 2; ++h)
#pragma unroll
        for (int ks = 0; ks < 8; ++ks) {
#pragma unroll
          for (int i = 0; i < 2; ++i)
#pragma unroll
            for (int j = 0; j < 2; ++j) {
              const bf16x8 wv = __builtin_bit_cast(bf16x8, W1 ? r1[ks * 2 + j] : r2[ks * 2 + j]);
              const bf16x8 xv = __builtin_bit_cast(bf16x8, W1 ? r2[ks * 2 + i] : r1[ks * 2 + i]);
              acc[i][j] = __builtin_amdgcn_mfma_f32_32x32x16_bf16(wv, xv, (h | ks) ? acc[i][j] : f32x16{}, 0, 0, 0);
            }
          if (h == 0) {
#pragma unroll
            for (int t = 0; t < 2; ++t) { r2[ks * 2 + t] = ld(ul + 1, ks * 2 + t); r1[ks * 2 + t] = stash[(ks * 2 + t) * 64]; }
            __builtin_amdgcn_sched_barrier(0);
          }
        }
    };
    if (wfirst) body(BoolC<true>{}); else { asm volatile("; row band first" ::: "memory"); body(BoolC<false>{}); }
    const unsigned tm = ld_dev(myws + OFF_TMO);
    if (!ok) {
#pragma unroll
      for (int i = 0; i < 2; ++i)
#pragma unroll
        for (int j = 0; j < 2; ++j)
#pragma unroll
          for (int r = 0; r < 16; ++r) acc[i][j][r] = __builtin_nanf("");
    }
    {
      const unsigned m = lane & 31, hh = lane >> 5;
#pragma unroll
      for (int i = 0; i < 2; ++i)
#pragma unroll
        for (int g = 0; g < 4; ++g)
#pragma unroll
          for (int j = 0; j < 2; ++j)
            lds[wave * 1024 + ((i * 4 + g) * 2 + j) * 64 + hh * 32 + ((m + 4 * (2 * (g & 1) + hh)) & 31)] =
                f32x4{acc[i][j][4 * g], acc[i][j][4 * g + 1], acc[i][j][4 * g + 2], acc[i][j][4 * g + 3]};
    }
    __syncthreads();
    {
      const unsigned row = 8 * wave + (lane >> 3), c = lane & 7, g = c >> 1, hh = c & 1;
      const unsigned slot = (((row >> 5) * 4 + g) * 2) * 64 + hh * 32 + (((row & 31) + 4 * (c & 3)) & 31);
      f32x4 r0[8], r1[8];
#pragma unroll
      for (int w = 0; w < 8; ++w) { r0[w] = lds[w * 1024 + slot]; r1[w] = lds[w * 1024 + slot + 64]; }
      __builtin_amdgcn_sched_barrier(0);
      f32x4 e0 = r0[0], e1 = r1[0];
#pragma unroll
      for (int w = 1; w < 8; ++w) { e0 += r0[w]; e1 += r1[w]; }
      u32x4 o;
#pragma unroll
      for (int i = 0; i < 4; ++i) o[i] = __builtin_bit_cast(unsigned, __builtin_convertvector(f32x2{e0[i], e1[i]}, bf16x2));
      if (tm == call) o = u32x4{NANPK, NANPK, NANPK, NANPK};
      __builtin_amdgcn_raw_buffer_store_b128(o, rsrc(myout, OUT_BYTES), (64 * grow + row) * 4096 + gcol * 128 + c * 16, 0, AUX_SYS);
    }
    WAIT_VM(0);
    __builtin_amdgcn_s_barrier();
    if (wave == 0) {
      bool tell = true;
      const unsigned fo = fo_of(ti_, rank);
      if (ti_ >= 32) {
        const unsigned gs = ti_ < (wide(rank) ? 104u : 36u) ? 4 : ncol(rank);
        unsigned old = 0, off = OFF_FC + (fo - 32) * 128;
        asm volatile("" : "+v"(off));
        if (lane == 0) old = __hip_atomic_fetch_add((unsigned*)(myws + off), 1u, __ATOMIC_RELAXED, __HIP_MEMORY_SCOPE_AGENT);
        tell = rfl(old) % gs == gs - 1;
      }
      if (tell && lane < 8 && (unsigned)lane != rank) st_sys(peer_ws + OFF_FO + (rank * 128 + fo) * 128, call);
    }
    if (ti_ >= 28 || !wide(rank)) return;
    const unsigned t2 = tile_at(128 + ti_, rank, 0);
    job = (128 + ti_) | (t2 >> 4) << 8 | (t2 & 15) << 13 | 3u << 17;
    }
  }

  const u64 t_in = __builtin_amdgcn_s_memrealtime();
  if (bid == NGEMM + NMOVE) {
    if (wave) return;
    char* const peer_ws = P.link[((rank ^ lane) - 1) & 7][0];
    const unsigned call = rfl(ld_dev(cntp)) + 1;
    if (lane == 0) st_dev(cntp, call);
    const unsigned kl = lane < 7 ? lane : 0;
    auto drawn = [&]() { const u64 w = ld_dev64(myws + OFF_LC + kl * 128); return (unsigned)(w >> 32) == call ? (unsigned)w + NW : 0u; };
    auto post = [&](unsigned off, u64 v) {
      __hip_atomic_store((u64*)(myws + OFF_TM2 + lane * 128 + off), v, __ATOMIC_RELAXED, __HIP_MEMORY_SCOPE_AGENT);
      __hip_atomic_store((u64*)(myws + OFF_TM2 + (64 + lane) * 128 + off), v, __ATOMIC_RELAXED, __HIP_MEMORY_SCOPE_AGENT);
    };
    bool alive = true;
    unsigned dec = 0;
    {
      unsigned ts = 0;
      for (;;) {
        const bool go = drawn() >= C0;
        const u64 dt = __builtin_amdgcn_s_memrealtime() - t_in;
        if (go && !ts) ts = dt > 100000 ? 100000u : (unsigned)dt + 1;
        if (__all(ts != 0) || !(alive = spin_ok<2>(t_in, call, myws))) break;
      }
      unsigned sum = 0;
#pragma unroll
      for (int i = 0; i < 7; ++i) sum += rdl(ts, i);
      if (lane < 8) __hip_atomic_store((u64*)(peer_ws + OFF_WV + rank * 128), (u64)call << 32 | sum, __ATOMIC_RELAXED, __HIP_MEMORY_SCOPE_SYSTEM);
      u64 v;
      for (;;) {
        v = __hip_atomic_load((const u64*)(myws + OFF_WV + (lane & 7) * 128), __ATOMIC_RELAXED, __HIP_MEMORY_SCOPE_SYSTEM);
        if (__all((unsigned)(v >> 32) == call) || !(alive = alive && spin_ok<2>(t_in, call, myws))) break;
      }
      {
        const unsigned st = (lane >> 4) > 2 ? 2 : lane >> 4, c = lane & 15, cand = st == 2 ? (c & 3) << 8 : c << (4 * st);
        unsigned worst = 0;
#pragma unroll
        for (int i = 0; i < 3; ++i) {
          const unsigned r = st == 2 ? 6 + (i & 1) : 3 * st + i, n = nrow(cand, r);
          const unsigned cost = pick((unsigned)v, r) + IQ * (56 - 2 * ncol(r)) * n;
          worst = (n < 9 || n > (wide(r) ? 12u : 17u) || (st == 2 && c > 2)) ? 0x3ffffffu : cost > worst ? cost : worst;
        }
        unsigned key = (worst > 0x3ffffffu ? 0x3ffffffu : worst) << 4 | c;
#pragma unroll
        for (int d = 1; d < 16; d <<= 1) { const unsigned o = pick(key, lane ^ d); key = o < key ? o : key; }
        dec = (rdl(key, 0) & 15u) | (rdl(key, 16) & 15u) << 4 | (rdl(key, 32) & 3u) << 8;
      }
      if (!alive) dec = DEC0;
      post(16, call << 16 | dec);
      if (lane == 0) __hip_atomic_store((u64*)(myws + OFF_DEC), (u64)call << 32 | dec, __ATOMIC_RELAXED, __HIP_MEMORY_SCOPE_AGENT);
    }
    const unsigned pl = rank ^ (kl + 1), want = call * gsize(pl, rank);
    unsigned known = head_len(dec, rank, pl), rest = 2 * (late_len(dec, pl) + gsize(pl, rank));
#pragma unroll
    for (int i = 0; i < 7; ++i) if ((unsigned)i != kl) rest += gsize(pl, rank ^ (i + 1)) + gsize(rank ^ (i + 1), pl);
    unsigned todo = 0x1ffu;
    while (todo) {
      const unsigned dr = drawn(), ff = ld_dev(myws + OFF_FF + kl * 128);
      if ((todo & 0x180) && (__any(dr + ((todo & 0x80) ? TSNAP : TSNAP2) >= known) || !alive)) {
        const unsigned left = known + rest / 2 + 512 - dr;
        unsigned mn = 0xffffu;
#pragma unroll
        for (int i = 0; i < 7; ++i) { const unsigned x = rdl(left, i); mn = x < mn ? x : mn; }
        u64 val = (u64)(call & 15u) << 60 | (u64)(call & 15u) << 28;
#pragma unroll
        for (int i = 0; i < 7; ++i) { const unsigned d = rdl(left, i) - mn; val |= (u64)(d > 127 ? 127u : d) << lag_bit(i); }
        post((todo & 0x80) ? 8 : 24, val);
        known += late_len(dec, pl) + gsize(pl, rank); rest = 0;
#pragma unroll
        for (int i = 0; i < 7; ++i)
          if ((unsigned)i != kl) {
            const int tot = gsize(pl, rank ^ (i + 1)) + gsize(rank ^ (i + 1), pl), sh = share_of(val, kl, i, tot);
            known += firm(sh); rest += tot - firm(sh) - firm(tot - sh);
          }
        todo &= (todo & 0x80) ? ~0x80u : ~0x100u;
        if (!alive) continue;
      }
      const unsigned have = (unsigned)__ballot((int)(ff - want) >= 0) & todo & 127u;
#pragma unroll
      for (int i = 0; i < 7; ++i)
        if (((have >> i) & 1) && lane < 8 && (unsigned)lane != rank) st_sys(peer_ws + OFF_FG + ((rank ^ (i + 1)) * 8 + rank) * 128, call);
      todo &= ~have;
      if (!alive) break;
      if (todo) alive = spin_ok<2>(t_in, call, myws);
    }
    return;
  }

  if (wave > NBW) return;
  const unsigned p = (rank ^ (k + 1)) & 7u;
  typedef __attribute__((address_space(3))) volatile u32x4 lds_u32x4;
  typedef __attribute__((address_space(3))) volatile unsigned lds_u32;
  lds_u32x4* const tbl = (lds_u32x4*)lds;
  lds_u32* const box = (lds_u32*)(tbl + TBL);
  auto unit_entry = [&](unsigned u, unsigned q) {
    return u32x4{p * SLICE + u * UNIT + q * 4096, OFF_FS + (p * 128 + u) * 128, OFF_FL + (p * 128 + u) * 128 + q * 4, 1u};
  };

  if (wave) {
    const unsigned tab = P.tab[lane];
    unsigned cnt_raw = 0;
    if (wave == 1) cnt_raw = ld_dev(cntp);
    auto in_entry = [&](unsigned n, unsigned dec) {
      const unsigned ui = NPUSH + (n >> 2);
      return unit_entry(unit_of(code_at(ui >> 1, rank, pick(tab, (ui >> 1) & 63) & 63u), ui & 1, rank, dec) & 127u, n & 3);
    };
    auto tile_entry = [&](unsigned dec, unsigned o, unsigned ti, unsigned half, unsigned fa, unsigned noff) {
      const unsigned t = tile_at(ti, o, (pick(tab, ti & 63) >> 8) & 63u);
      return u32x4{(64 * (gband(t >> 4, o, dec) & 31u) + 32 * half) * 4096 + (col0(o) + (t & 15)) * 128, fa, noff, 2u};
    };
    auto entry = [&](unsigned n, unsigned dec) {
      const unsigned nin = N1 + 8 * (nrow(dec, rank) - nfix(rank)), y = n - nin, ti = (y >> 1) & 255u;
      const u32x4 ie = in_entry(n, dec), te = tile_entry(dec, p, ti, y & 1, OFF_FO + (p * 128 + fo_of(ti, p)) * 128, 0);
      return n < nin ? ie : n < head_len(dec, rank, p) ? te : u32x4{0, 0, 0, 0};
    };
    {
      const unsigned n = threadIdx.x - 64;
      const u32x4 e = in_entry(n, DEC0);
      tbl[n] = n < N1 ? e : u32x4{0, 0, 0, 0};
#pragma unroll
      for (unsigned n0 = NBW * 64; n0 < TBL; n0 += NBW * 64) if (n0 + n < TBL) tbl[n0 + n] = u32x4{0, 0, 0, 0};
    }
    const unsigned call = rfl(cnt_raw) + 1;
    if (wave == 1 && lane < 8) box[lane] = lane == 4 ? (call + 1) << 16 : (call + 1) << 28;
    if (wave == 1 && lane == 0) st_dev(cntp, call);
    asm volatile("s_waitcnt lgkmcnt(0)\n\ts_barrier" ::: "memory");
    if (wave > 1) return;

    typedef __attribute__((address_space(3))) volatile u64 lds_u64;
    unsigned dec;
    while (((dec = rfl(box[4])) ^ call << 16) >> 16) if (!spin_ok<4>(t_in, call, myws)) { dec = DEC0; break; }
    unsigned la = head_len(dec, rank, p);
#pragma unroll 1
    for (unsigned n = N1; n < la; n += 64) tbl[n + lane] = entry(n + lane, dec);
    const unsigned l8 = lane < 9 ? lane & 7 : (lane - 9) & 7, q = wide(p) ? (l8 + 6) & 7 : l8, kk = ((rank ^ q) - 1) & 7u;
    const bool pair = q != p && q != rank;
    const int gd = gsize(p, q), tot = gd + gsize(q, p);
    int beg = 0, open = tot;
#pragma unroll 1
    for (int deal = 0; deal < 2; ++deal) {
      u64 plan;
      while ((((plan = rfl64(*(lds_u64*)(box + 2 + 4 * deal))) >> 28) & 0xf0000000full) != (call & 15u) * 0x100000001ull)
        if (!spin_ok<2>(t_in, call, myws)) { plan = 0; break; }
      const int sh = share_of(plan, k, kk, open), end = deal ? beg + sh : firm(sh);
      const int b = lane < 8 ? (beg < gd ? beg : gd) : (beg > gd ? beg : gd), e = lane < 8 ? (end < gd ? end : gd) : (end > gd ? end : gd);
      const unsigned len = lane == 8 ? (deal ? 0u : late_len(dec, p)) : lane > 16 ? 0u : pair ? e - b : lane < 8 && q == rank && !deal ? gd : 0u;
      const unsigned org = lane == 8 ? 2 * (tshare(p) + nshared(p)) : lane < 8 ? 2 * gfirst(p, q) + b : 2 * gfirst(q, p) + (tot - 1 - b);
      unsigned total = 0;
#pragma unroll
      for (int i = 0; i < 17; ++i) total += rdl(len, i);
#pragma unroll 1
      for (unsigned e0 = 0; e0 < total + (deal ? NW : 0); e0 += 64) {
        unsigned l = 0, sub = e0 + lane;
#pragma unroll
        for (int i = 0; i < 17; ++i) { const unsigned x = rdl(len, i); if (l == (unsigned)i && sub >= x) { sub -= x; ++l; } }
        const unsigned ql = pick(q, l), og = pick(org, l), h = l < 9 ? og + sub : og - sub, ti = (h >> 1) & 255u;
        const u32x4 te = tile_entry(dec, l < 9 ? p : ql, ti, h & 1, l < 9 ? OFF_FO + (p * 128 + fo_of(ti, p)) * 128 : OFF_FG + (ql * 8 + p) * 128,
                                    l < 8 && ql == rank ? (OFF_FF + k * 128) | 1u : 0u);
        const unsigned at = la + e0 + lane;
        tbl[at < TBL ? at : TBL - 1] = l < 17 ? te : u32x4{0, OFF_TMO, 0, deal ? 3u : 0u};
      }
      la += total; beg = end; open = tot - end - firm(tot - sh);
    }
    return;
  }

  const unsigned u1 = (P.first >> (8 * (j0 >> 2))) & 127u;
  const unsigned cnt_raw = ld_dev(cntp), fs1 = ld_sys(myws + OFF_FS + (p * 128 + u1) * 128);
  const unsigned call = rfl(cnt_raw) + 1;
  asm volatile("" :: "s"(call) : "memory");
  const u64 t_ld = __builtin_amdgcn_s_memrealtime();
  if (lane == 0) {
    st_dev(cntp, call);
    __hip_atomic_fetch_max((u64*)(myws + OFF_LC + k * 128), (u64)call << 32, __ATOMIC_RELAXED, __HIP_MEMORY_SCOPE_AGENT);
  }
  auto desc = [](const void* b, unsigned bytes) { const u64 a = (u64)b; return u32x4{rfl((unsigned)a), rfl((unsigned)(a >> 32) & 0xffffu), rfl(bytes), 0x00020000u}; };
  const unsigned vcall = call, vone = 1, vlc = OFF_LC + k * 128;
  const unsigned lo_lin = lane * 16, lo_tile = (lane >> 3) * 4096 + (lane & 7) * 16;
  const unsigned tm_l = OFF_TM2 + mv * 128 + ((lane & 7) >= 2 ? (lane & 7) * 4 : 0);
  u32x4 w0 = {}, w1 = {}, w2 = {}, w3 = {};
  unsigned o0, o1, o2, o3, r0 = 0, r1 = 0, r2 = 0, r3 = 0;
  bool ok = true;
  unsigned kind, noff;
  unsigned nfo;
  auto take = [&](const u32x4& e) {
    kind = rfl(e[3]); noff = rfl(e[2]);
    const unsigned ts = kind == 2 ? 8 * 4096 : 1024;
    o0 = e[0] + (kind == 2 ? lo_tile : lo_lin); o1 = o0 + ts; o2 = o1 + ts; o3 = o2 + ts;
    nfo = (lane & 7) ? tm_l : e[1];
  };
  auto fetch = [&](unsigned idx) {
    u32x4 e = tbl[idx < TBL ? idx : TBL - 1];
    while (rfl(e[3]) == 0) {
      const unsigned x = ld_sys(myws + tm_l);
      box[lane & 7] = x;
      if (!spin_ok<2>(t_in, call, myws)) { e = u32x4{0, OFF_TMO, 0, 3u}; ok = false; break; }
      e = tbl[idx < TBL ? idx : TBL - 1];
    }
    take(e);
  };
  auto put = [&](unsigned kd) {
    const u32x4 rd = desc(kd == 2 ? myout : myws + OFF_DATA, kd == 2 ? OUT_BYTES : NR * SLICE);
    asm volatile("s_nop 4\n\tbuffer_store_dwordx4 %0, %4, %8, 0 offen sc0 sc1\n\tbuffer_store_dwordx4 %1, %5, %8, 0 offen sc0 sc1\n\t"
                 "buffer_store_dwordx4 %2, %6, %8, 0 offen sc0 sc1\n\tbuffer_store_dwordx4 %3, %7, %8, 0 offen sc0 sc1\n\ts_nop 1"
                 :: "v"(w0), "v"(w1), "v"(w2), "v"(w3), "v"(r0), "v"(r1), "v"(r2), "v"(r3), "s"(rd) : "memory");
  };
  auto notify = [&](unsigned off) {
    if (off & 1) asm volatile("s_mov_b64 exec, 1\n\tglobal_atomic_add %0, %1, %2\n\ts_mov_b64 exec, -1" :: "v"(off - 1), "v"(vone), "s"(myws) : "memory");
    else asm volatile("s_mov_b64 exec, 1\n\tglobal_store_dword %0, %1, %2 sc0 sc1\n\ts_mov_b64 exec, -1" :: "v"(off), "v"(vcall), "s"(myws) : "memory");
  };

  take(unit_entry(u1, j0 & 3));
  asm volatile("" : "+v"(o0), "+v"(o1), "+v"(o2), "+v"(o3), "+v"(w0), "+v"(w1), "+v"(w2), "+v"(w3));
  {
    u64 bad;
    {
      const unsigned ro = OFF_SW + (rank * 128 + u1) * 128 + (j0 & 3) * 8 + (lane & 1) * 4;
      unsigned left = rfl(fs1) >= call ? 0u : 64u;
      asm volatile(
          "s_mov_b64 %0, 0\n\ts_cmp_eq_u32 %4, 0\n\ts_cbranch_scc1 2f\n\t"
          "global_load_dword %1, %5, %6 sc0 sc1\n\ts_sleep 6\n\t"
          "global_load_dword %2, %5, %6 sc0 sc1\n\ts_sleep 6\n"
          "0:\n\t"
          "global_load_dword %3, %5, %6 sc0 sc1\n\t"
          "s_waitcnt vmcnt(2)\n\tv_cmp_gt_u32 %0, %7, %1\n\ts_cmp_eq_u64 %0, 0\n\ts_cbranch_scc1 1f\n\tglobal_load_dword %1, %5, %6 sc0 sc1\n\t"
          "s_waitcnt vmcnt(2)\n\tv_cmp_gt_u32 %0, %7, %2\n\ts_cmp_eq_u64 %0, 0\n\ts_cbranch_scc1 1f\n\tglobal_load_dword %2, %5, %6 sc0 sc1\n\t"
          "s_waitcnt vmcnt(2)\n\tv_cmp_gt_u32 %0, %7, %3\n\ts_cmp_eq_u64 %0, 0\n\ts_cbranch_scc1 1f\n\t"
          "s_sub_u32 %4, %4, 1\n\ts_cmp_lg_u32 %4, 0\n\ts_cbranch_scc1 0b\n"
          "2:\n\ts_waitcnt vmcnt(0)\n"
          "1:"
          : "=&s"(bad), "=&v"(r1), "=&v"(r2), "=&v"(r3), "+s"(left) : "v"(ro), "s"(pws), "s"(call) : "memory", "scc");
    }
    if (bad) ok = wait_flags(myws + nfo, call, myws, 1);
    while (__builtin_amdgcn_s_memrealtime() - t_ld < TG) __builtin_amdgcn_s_sleep(1);
  }
  bool joined = false;
  unsigned skind = 0, snoff = 0, pnoff = 0;
  for (;;) {
    u32x4 v0, v1, v2, v3;
    unsigned f, tk;
    asm volatile("s_mov_b64 exec, 1\n\tglobal_atomic_add %0, %1, %2, %3 sc0\n\ts_mov_b64 exec, -1" : "=&v"(tk) : "v"(vlc), "v"(vone), "s"(myws) : "memory");
    const bool first = snoff != 0;
    if (first) { put(skind); pnoff = snoff; }
    {
      const u32x4 rs = desc(kind == 2 ? pout : pws + OFF_DATA, ok ? (kind == 2 ? OUT_BYTES : NR * SLICE) : 0u);
      asm volatile("s_nop 4\n\tbuffer_load_dwordx4 %0, %4, %8, 0 offen sc0 sc1\n\tbuffer_load_dwordx4 %1, %5, %8, 0 offen sc0 sc1\n\t"
                   "buffer_load_dwordx4 %2, %6, %8, 0 offen sc0 sc1\n\tbuffer_load_dwordx4 %3, %7, %8, 0 offen sc0 sc1"
                   : "=&v"(v0), "=&v"(v1), "=&v"(v2), "=&v"(v3) : "v"(o0), "v"(o1), "v"(o2), "v"(o3), "s"(rs) : "memory");
    }
    if (!first && skind) put(skind);
    const unsigned ckind = kind, cnoff = noff, q0 = o0, q1 = o1, q2 = o2, q3 = o3;
    if (!joined) { __builtin_amdgcn_s_barrier(); joined = true; }
    if (!skind) WAIT_VM(4);
    asm volatile("s_waitcnt vmcnt(8)" : "+v"(tk) : "v"(r1), "v"(r2), "v"(r3) : "memory");
    fetch(rfl(tk) + NW);
    asm volatile("global_load_dword %0, %1, %2 sc0 sc1" : "=&v"(f) : "v"(nfo), "s"(myws) : "memory");
    asm volatile("" : "+v"(o0), "+v"(o1), "+v"(o2), "+v"(o3));
    if (pnoff) { WAIT_VM(5); notify(pnoff); pnoff = 0; WAIT_VM(1); } else WAIT_VM(0);
    asm volatile("" : "+v"(v0), "+v"(v1), "+v"(v2), "+v"(v3), "+v"(f) :: "memory");
    if (rdl(f, 1) == call) ok = false;
    w0 = v0; w1 = v1; w2 = v2; w3 = v3;
    if (!ok) w0 = w1 = w2 = w3 = u32x4{NANPK, NANPK, NANPK, NANPK};
    r0 = q0; r1 = q1; r2 = q2; r3 = q3; skind = ckind; snoff = cnoff;
    box[lane & 7] = f;
    if (kind == 3 || rfl(f) < call) {
      if (kind == 3 || rfl(ld_sys(myws + nfo)) < call) {
        put(skind);
        if (snoff) { WAIT_VM(0); notify(snoff); }
        skind = 0; snoff = 0;
        if (kind == 3) break;
        if (ok) ok = wait_flags(myws + nfo, call, myws, 1);
      }
    }
  }
  if (!ok && lane == 0) st_sys(myout + gband(0, rank, DEC0) * 64 * 4096 + col0(rank) * 128, NANPK);
}

}

extern "C" void kernel_launch(int rank, int nranks, ncclComm_t comm, hipStream_t stream,
                              void* const* d_in, const int* in_sizes, int n_in,
                              void* d_out, int out_size, const int* in_dtypes, int out_dtype,
                              void* const* d_out_all, void* const* d_ws_all, size_t ws_bytes) {
  if (nranks != NR || (unsigned)rank >= (unsigned)NR || n_in < 2 || ws_bytes < WS_NEED || out_size != 2048 * 2048 ||
      in_sizes[0] != 2048 * 256 || in_sizes[1] != 256 * 2048) { fprintf(stderr, "unexpected problem shape\n"); return; }
  Params P;
  P.x = (const char*)d_in[0]; P.w = (const char*)d_in[1];
  for (int k = 0; k < 8; ++k) { const int p = (rank ^ (k + 1)) & 7; P.link[k][0] = (char*)d_ws_all[p]; P.link[k][1] = (char*)d_out_all[p]; }
  P.myws = (char*)d_ws_all[rank]; P.myout = (char*)d_out_all[rank];
  P.rank = rank; P.first = 0;
  const unsigned char bo[16] = {32, 0, 33, 1, 2, 34, 3, 35, 4, 36, 5, 37, 6, 38, 7, 39};
  unsigned char ord[64]; int no = 0, na = 0, nb = 0;
  for (int n = 0; n < 16; ++n) {
    if (bo[n] < 32) { for (int b = 0; b < nb; ++b) ord[no++] = na * 8 + b; ++na; }
    else { for (int a = 0; a < na; ++a) ord[no++] = a * 8 + nb; ++nb; }
  }
  for (unsigned i = 0; i < 4; ++i) P.first |= unit_of(bo[(NPUSH + i) >> 1], (NPUSH + i) & 1, rank, DEC0) << (8 * i);
  for (int i = 0; i < 64; ++i) P.tab[i] = bo[i & 15] | (unsigned)ord[i] << 8;
  {
    int posn[64];
    for (unsigned i = 0; i < 25; ++i) posn[code_at(i, rank, bo[i & 15]) & 63] = i;
    const unsigned nt = wide(rank) ? 128 : 102;
    int used[8] = {0}; bool done[128] = {false};
    for (int b = 0; b < 128; ++b) P.job[b] = 0;
    for (int pass = 0; pass < 2; ++pass)
      for (unsigned ti = 0; ti < nt; ++ti) {
        const unsigned t = tile_at(ti, rank, ord[ti & 63]), a = t >> 4, b = t & 15;
        int x = (a & 3) * 2 + (b & 1);
        if (done[ti] || (pass == 0 && used[x] == 16)) continue;
        while (used[x] == 16) x = (x + 1) & 7;
        P.job[used[x]++ * 8 + x] = ti | a << 8 | b << 13 | (posn[32 + b] < posn[a] ? 1u : 0u) << 17 | 1u << 18;
        done[ti] = true;
      }
  }
  unsigned mask[128] = {0}; int first[128];
  for (int u = 0; u < 128; ++u) first[u] = 99;
  for (unsigned r = 0; r < NR; ++r)
    for (unsigned dec = 0; dec < 768; ++dec) {
      const unsigned n = nrow(dec, r);
      if (n < 9 || n > (wide(r) ? 12u : 17u)) continue;
      for (unsigned i = 0; i < 2 * (n + ncol(r)); ++i) {
        const unsigned u = unit_of(code_at(i >> 1, r, bo[(i >> 1) & 15]), i & 1, r, dec);
        if (r != (unsigned)rank && i >= NPUSH) { mask[u] |= 1u << r; if ((int)i < first[u]) first[u] = i; }
      }
    }
  int us[128], nu = 0;
  for (int u = 0; u < 128; ++u) if (first[u] == 99) first[u] = 20;
  for (int pos = 98; pos >= 0; --pos) for (int u = 0; u < 128; ++u) if (first[u] == pos) us[nu++] = u;
  int i0 = 0, i1 = 16; unsigned np = 0;
  for (int bb = 0; bb < 128; ++bb) {
    const int b = (bb + 96) & 127;
    const int u = us[(b & 7) == 0 ? i0++ : i1++];
    P.task[b] = u | mask[u] << 24 | (first[u] < (int)(NPUSH + NCRIT) ? 0u : 1u << 20);
    if ((b & 7) != 0 && np < 7 * NPUSH) {
      const unsigned dst = rank ^ ((np + np / 7) % 7 + 1), j = np / 7;
      P.task[b] = u | mask[u] << 24 | unit_of(bo[j >> 1], j & 1, dst, DEC0) << 8 | ((rank ^ dst) - 1) << 16 | 1u << 19; ++np;
    }
  }
  hipLaunchKernelGGL(fused_gemm_ar, dim3(NGEMM + NMOVE + 1), dim3(512), 0, stream, P);
}
```
